```python
import numpy as np

kernel_name = "gpu_gemm_a2a_m4096_k4096_n16384_bf16_gelu"


ABSMAX_RTOL = 0.008

COMPUTE_DTYPE = "bfloat16"
EPILOGUE = "gelu"
IN_QUANT = None
OUT_QUANT = None
IN_DTYPES = {"x": "bfloat16", "w_mat": "bfloat16"}
WORLD_SIZE = 8

def _to_bf16(x):
    u = np.ascontiguousarray(x, dtype=np.float32).view(np.uint32)
    return ((u + 0x7FFF + ((u >> 16) & 1)) & 0xFFFF0000).view(np.float32)

def setup_inputs():

    x = _to_bf16(np.random.randn(4096, 4096).astype(np.float32))
    w_mat = _to_bf16((np.random.randn(4096, 16384) / np.sqrt(4096)).astype(np.float32))
    return {"x": x, "w_mat": w_mat}

def _epilogue(y):
    c = 0.7978845608028654
    return (0.5 * y * (1.0 + np.tanh(c * (y + 0.044715 * y ** 3)))).astype(np.float32)

def reference(x, w_mat):
    y = (x @ w_mat).astype(np.float32)
    return _epilogue(y)
```

```cpp
#include <hip/hip_runtime.h>
#include <rccl/rccl.h>
#include <cstdio>

#define HIP_CHECK(expr) do { hipError_t _e = (expr); if (_e != hipSuccess) { \
    fprintf(stderr, "HIP error %s at %s:%d\n", hipGetErrorString(_e), __FILE__, __LINE__); \
    return; } } while (0)

#define LAS __attribute__((address_space(3)))
typedef short bf16x8 __attribute__((ext_vector_type(8)));
typedef short bf16x4 __attribute__((ext_vector_type(4)));
typedef float f32x4 __attribute__((ext_vector_type(4)));
typedef float f32x2 __attribute__((ext_vector_type(2)));
typedef unsigned u32x4 __attribute__((ext_vector_type(4)));
typedef __bf16 bfx2 __attribute__((ext_vector_type(2)));

#define SX 1
#define TAU_O 16
#define TAU_X2 4
#define XB 4
#define AUXA 17
#define NSA 5
#define NSB 5
constexpr int NR = 8;
constexpr int KDIM = 4096, NDIM = 16384, MLOC = 512, NSH = 2048;
constexpr int NKT = 64, SLOT = 16384, NJ = 2 * NKT, NWG = 256;
constexpr int RING_BYTES = (NSA + NSB) * SLOT;
constexpr int LDS_BYTES = RING_BYTES;
constexpr int RR = 4 - SX;
constexpr unsigned OUT_PER_DEST = RR * 16;
constexpr int NREM_XCD = 14 * RR;
constexpr size_t WS_DONE = 0, WS_TMO = 128, WS_OUTCNT = 4096, WS_XCNT = 8192, WS_XLOC = 16384, WS_STAMP = 65536, WS_XBUF = 1u << 20;
constexpr size_t XTILE = 128 * KDIM * 2;
constexpr size_t XPAR = (size_t)NR * 2 * XTILE;
constexpr unsigned NSLOT_O = SX == 0 ? 40 : 37, FRAME_O = NSLOT_O * TAU_O, FRAME_X = NWG * TAU_X2 / 2;
constexpr unsigned long long DEADLINE = 40000000ull;

struct Params { const char* x; const char* w; char* out[NR]; char* ws[NR]; int rank; unsigned desc[NWG]; };
struct Tile { int src; int rt; int dest; int ct; };
__device__ __forceinline__ Tile tile_of(unsigned d, int rank) {
  Tile r; r.src = (rank + (int)(d & 7)) & 7; r.rt = (d >> 3) & 3; r.dest = (rank + (int)((d >> 5) & 7)) & 7; r.ct = (d >> 8) & 15; return r;
}
static inline unsigned tile_code(int so, int rt, int dof, int ct) { return (unsigned)(so | rt << 3 | dof << 5 | ct << 8); }
static void build_desc(unsigned* desc) {
  int cnt[8] = {0}, seen[8] = {0};
  for (int x = 0; x < 8; ++x) {
    unsigned first[64], second[64]; int n1 = 0, n2 = 0;
    const int late = NREM_XCD > 32 ? NREM_XCD - 32 : 0, nfull = late / RR, nsingle = late % RR;
    int state[8][2] = {};
    for (int k = 0; k < nfull; ++k) { const int d = 1 + (nfull * x + k) % 7; state[d][state[d][0] ? 1 : 0] = RR; }
    if (nsingle) { const int d = 1 + 3 + (x & 3); state[d][state[d][1] ? 0 : 1] = nsingle; }
    for (int h = 0; h < 2; ++h) for (int d = 1; d < 8; ++d) for (int rt = 0; rt < RR; ++rt) {
      const unsigned c = tile_code(0, rt, d, 2 * x + h);
      if (rt >= RR - state[d][h]) second[n2++] = c; else first[n1++] = c;
    }
    for (int rs = 0; rs < 4 + 7 * SX; ++rs) for (int h = 0; h < 2; ++h) {
      const unsigned c = rs < 4 ? tile_code(0, rs, 0, 2 * x + h) : tile_code(1 + (rs - 4) / (SX ? SX : 1), RR + (rs - 4) % (SX ? SX : 1), 0, 2 * x + h);
      if (n1 < 32) first[n1++] = c; else second[n2++] = c;
    }
    for (int i = 0; i < 32; ++i) { desc[i * 8 + x] = first[i] | second[i] << 12; cnt[(first[i] >> 5) & 7]++; }
  }
  for (int b = 0; b < NWG; ++b) { const int d = (desc[b] >> 5) & 7; const int k = seen[d]++; desc[b] |= (unsigned)k << 24; (void)cnt; }
}

__device__ __forceinline__ unsigned cvtpk(float lo, float hi) { f32x2 v = {lo, hi}; bfx2 b = __builtin_convertvector(v, bfx2); return __builtin_bit_cast(unsigned, b); }
__device__ __forceinline__ float gelu1(float y) {
  const float k0 = -2.0f * 0.7978845608028654f * 1.4426950408889634f, k1 = k0 * 0.044715f;
  const float z = y * __builtin_fmaf(y * y, k1, k0);
  return y * __builtin_amdgcn_rcpf(1.0f + __builtin_amdgcn_exp2f(z));
}
__device__ __forceinline__ unsigned rfl(unsigned v) { return (unsigned)__builtin_amdgcn_readfirstlane((int)v); }
__device__ __forceinline__ unsigned now32() { return (unsigned)__builtin_amdgcn_s_memrealtime(); }
__device__ __forceinline__ unsigned next_slot(unsigned t, unsigned slot, unsigned frame) { return t - (t - slot) % frame + frame; }
template <int N> __device__ __forceinline__ void wait_vm() { asm volatile("s_waitcnt vmcnt(%0)" :: "i"(N) : "memory"); }
__device__ __forceinline__ void st16_sys(char* p, u32x4 v) { asm volatile("global_store_dwordx4 %0, %1, off sc0 sc1\n\ts_nop 1" :: "v"(p), "v"(v) : "memory"); }
#define BAR() do { asm volatile("" ::: "memory"); __builtin_amdgcn_s_barrier(); asm volatile("" ::: "memory"); } while (0)
#define SCHED() __builtin_amdgcn_sched_barrier(0)
#define TS(k) do {} while (0)

__device__ __forceinline__ bool wait_ge(const unsigned* flag, unsigned want, unsigned* tmo, unsigned call, bool active) {
  const unsigned long long t0 = __builtin_amdgcn_s_memrealtime();
  for (;;) {
    const bool ok = !active || __hip_atomic_load(flag, __ATOMIC_RELAXED, __HIP_MEMORY_SCOPE_SYSTEM) >= want;
    if (__all(ok)) return true;
    if (__hip_atomic_load(tmo, __ATOMIC_RELAXED, __HIP_MEMORY_SCOPE_AGENT) == call || __builtin_amdgcn_s_memrealtime() - t0 > DEADLINE) {
      __hip_atomic_store(tmo, call, __ATOMIC_RELAXED, __HIP_MEMORY_SCOPE_AGENT);
      return false;
    }
    __builtin_amdgcn_s_sleep(2);
  }
}

__global__ void __launch_bounds__(512, 2) gemm_a2a_kernel(Params p) {
  extern __shared__ __attribute__((aligned(16))) char smem[];
  LAS char* lds = (LAS char*)smem;
  const int tid = threadIdx.x, wid = (int)rfl(tid >> 6), lane = tid & 63;
  const int b = blockIdx.x, rank = p.rank;
  const unsigned desc = p.desc[b];
  const Tile T0 = tile_of(desc, rank), T1 = tile_of(desc >> 12, rank);
  const unsigned oslot = (desc >> 24) * TAU_O + ((desc >> 5) & 7) * (TAU_O / 8);
  constexpr int J0 = NKT - NSA;

  char* myws = p.ws[rank];
  unsigned* tmo = (unsigned*)(myws + WS_TMO);
  const unsigned peer = (unsigned)(rank + 1 + lane) & 7;
  if (wid == 7) {
    if (T1.src == rank) return;
    const unsigned call = rfl(__hip_atomic_load((unsigned*)(myws + WS_DONE), __ATOMIC_RELAXED, __HIP_MEMORY_SCOPE_AGENT)) / NWG + 1;
    const unsigned* fl = (const unsigned*)(myws + WS_XCNT + T1.src * 512);
    BAR();
    int it = 0;
#pragma unroll
    for (int c = 0; c < 4; ++c) {
      for (; it < J0 + 16 * c - 5; ++it) BAR();
      const unsigned seen = __hip_atomic_load(fl + c * 32, __ATOMIC_RELAXED, __HIP_MEMORY_SCOPE_SYSTEM);
      for (; it < J0 + 16 * c; ++it) BAR();
      if (rfl(seen) < call) { wait_ge(fl + c * 32, call, tmo, call, true); wait_vm<0>(); }
      if (c == 0) TS(6);
    }
  } else if (wid == 6) {
    const int g = b >> 4;
    u32x4 xv[4];
#pragma unroll
    for (int m = 0; m < XB; ++m)
      xv[m] = *(const u32x4*)(p.x + RR * XTILE + (size_t)((b & 15) * 8 + (lane >> 3)) * 8192 + (16 * m + g) * 128 + ((lane & 7) ^ ((4 * (b & 1) + (lane >> 4)) & 7)) * 16);
    const unsigned call = rfl(__hip_atomic_load((unsigned*)(myws + WS_DONE), __ATOMIC_RELAXED, __HIP_MEMORY_SCOPE_AGENT)) / NWG + 1;
    char* pws = p.ws[0];
#pragma unroll
    for (int q = 1; q < NR; ++q) pws = peer == (unsigned)q ? p.ws[q] : pws;
    const size_t so = WS_XBUF + (call & 1) * XPAR + (size_t)rank * 2 * XTILE + (size_t)g * 16384 + (b & 15) * 1024 + lane * 16;
    __builtin_amdgcn_s_waitcnt(0x0F70);
    BAR();
    int it = 0;
    unsigned tn = next_slot(now32(), (unsigned)b * TAU_X2 / 2, FRAME_X);
#pragma unroll
    for (int m = 0; m < XB; ++m) {
      const int ch = m;
      while ((int)(now32() - tn) < 0 && it < 24 + 16 * m) { BAR(); ++it; }
      tn += FRAME_X;
      const int i0 = it;
#pragma unroll
      for (int q = 0; q < 7; ++q) st16_sys(p.ws[(rank + 1 + q) & 7] + so + m * 16 * 16384, xv[m]);
      if (m == 0) TS(9);
      for (; it < i0 + 5; ++it) BAR();
      wait_vm<0>();
      unsigned tick; unsigned long long sv;
      asm volatile("s_mov_b64 %1, exec\n\ts_mov_b64 exec, 1\n\tglobal_atomic_add %0, %2, %3, off sc0 sc1\n\ts_mov_b64 exec, %1"
                   : "=&v"(tick), "=&s"(sv) : "v"(myws + WS_XLOC + ch * 128), "v"(1u) : "memory");
      for (; it < i0 + 8; ++it) BAR();
      asm volatile("s_waitcnt vmcnt(0)" : "+v"(tick) :: "memory");
      if (rfl(tick) == call * (64 * XB) - 1) {
        if (lane < 7) __hip_atomic_store((unsigned*)(pws + WS_XCNT + rank * 512 + ch * 128), call, __ATOMIC_RELAXED, __HIP_MEMORY_SCOPE_SYSTEM);
      }
    }
    TS(5);
  } else if (wid >= 4) {
    const int pw = wid - 4;
    unsigned done0;
    asm volatile("global_load_dword %0, %1, off sc1" : "=v"(done0) : "v"(p.ws[rank] + WS_DONE) : "memory");
    auto srdA = __builtin_amdgcn_make_buffer_rsrc((void*)(p.x + (size_t)T0.rt * XTILE), 0, (int)XTILE, 0x00020000);
    const auto srdB = __builtin_amdgcn_make_buffer_rsrc((void*)p.w, 0, KDIM * NDIM * 2, 0x00020000);
    unsigned voffA[2], voffB[2];
#pragma unroll
    for (int i = 0; i < 2; ++i) {
      voffA[i] = (unsigned)(lane >> 3) * 8192u + (unsigned)(((lane & 7) ^ ((4 * i + (lane >> 4)) & 7)) * 16);
      voffB[i] = (unsigned)(lane >> 4) * 32768u + (unsigned)(((lane & 15) ^ (((lane >> 4) << 2) | (i << 1))) * 16);
    }
    unsigned sA = (unsigned)(64 * pw) * 8192u, stepK = 128u, stepI = 65536u;
    const unsigned sB0 = (unsigned)(32 * pw) * 32768u + (unsigned)(T0.dest * 16 + T0.ct) * 256u, sB1 = (unsigned)(32 * pw) * 32768u + (unsigned)(T1.dest * 16 + T1.ct) * 256u;
    auto issueA = [&](int j, int s) {
      const unsigned so = sA + (unsigned)(j & 63) * stepK;
      LAS char* d = lds + s * SLOT + pw * 8192;
#pragma unroll
      for (int i = 0; i < 8; ++i) __builtin_amdgcn_raw_ptr_buffer_load_lds(srdA, (LAS void*)(d + i * 1024), 16, voffA[i & 1], so + i * stepI, 0, AUXA);
    };
    auto issueB = [&](int j, int s) {
      const unsigned so = ((j >> 6) ? sB1 : sB0) + (unsigned)(j & 63) * (64u * 32768u);
      LAS char* d = lds + (NSA + s) * SLOT + pw * 8192;
#pragma unroll
      for (int i = 0; i < 8; ++i) __builtin_amdgcn_raw_ptr_buffer_load_lds(srdB, (LAS void*)(d + i * 1024), 16, voffB[(i >> 1) & 1], so + i * (4u * 32768u), 0, 0);
    };
    static_assert(NSA <= NSB, "ring order");
#pragma unroll
    for (int i = -NSB; i < 0; ++i) { if (i + NSA >= 0) issueA(i + NSA, i + NSA); issueB(i + NSB, i + NSB); }
    constexpr int NW = (NSA == NSB) ? 16 * (NSA - 2) : 8 + 16 * (NSA - 2);
    static_assert(NW <= 63, "vmcnt range");
    asm volatile("s_waitcnt vmcnt(%1)" : "+v"(done0) : "i"(NW + 16 > 63 ? 63 : NW + 16) : "memory");
    BAR();
    int sa = 0, sb = 0;
    for (int j = 0; j < NJ - NSB; ++j) {
      wait_vm<NW>(); BAR();
      if (j == J0) {
        const char* a1 = p.x + (size_t)T1.rt * XTILE;
        if (T1.src != rank) {
          a1 = p.ws[rank] + WS_XBUF + ((rfl(done0) / NWG + 1) & 1) * XPAR + (size_t)(T1.src * 2 + T1.rt - RR) * XTILE;
          sA = (unsigned)pw * 8192u; stepK = 16384u; stepI = 1024u; voffA[0] = voffA[1] = (unsigned)lane * 16u;
        }
        srdA = __builtin_amdgcn_make_buffer_rsrc((void*)a1, 0, (int)XTILE, 0x00020000);
      }
      issueA(j + NSA, sa); issueB(j + NSB, sb);
      sa = sa + 1 == NSA ? 0 : sa + 1; sb = sb + 1 == NSB ? 0 : sb + 1;
    }
#pragma unroll
    for (int j = NJ - NSB; j < NJ; ++j) {
      int cnt = 0;
      const int iA = j + 1 - NSA;
      if (NSA != NSB && iA + NSB < NJ) cnt += 8;
      for (int i = iA + 1; i <= j - 1; ++i) { if (i + NSA < NJ) cnt += 8; if (i + NSB < NJ) cnt += 8; }
      if (j + 1 >= NJ) cnt = 0;
      switch (cnt) {
        case 8: wait_vm<8>(); break; case 16: wait_vm<16>(); break; case 24: wait_vm<24>(); break; case 32: wait_vm<32>(); break;
        case 40: wait_vm<40>(); break; case 48: wait_vm<48>(); break; default: wait_vm<0>(); break;
      }
      BAR();
      if (j + NSA < NJ) { issueA(j + NSA, sa); sa = sa + 1 == NSA ? 0 : sa + 1; }
    }
  } else {
    const int wr = wid >> 1, wc = wid & 1, fr = lane & 15, fq = lane >> 4;
    const unsigned baseA0 = (unsigned)(wr * 64 + fr) * 128u + (unsigned)((fq ^ ((fr >> 1) & 7)) * 16);
    unsigned baseB[4];
    { const int q = (lane & 15) >> 2, pp = lane & 3; const int s = (q << 2) | ((fq & 1) << 1);
#pragma unroll
      for (int n = 0; n < 4; ++n) baseB[n] = (unsigned)(NSA * SLOT) + (unsigned)(8 * fq + q) * 256u + (unsigned)(((8 * wc + 2 * n) ^ s) * 16 + 8 * pp); }
    f32x4 acc[4][4];
    bf16x8 FA[2][4], FB[2][4];
    u32x4 O[8];
    auto readA = [&](bf16x8 (&F)[4], unsigned addr) {
#pragma unroll
      for (int m = 0; m < 4; ++m) F[m] = *(const LAS bf16x8*)(lds + addr + m * 2048);
    };
    auto readB = [&](bf16x8 (&F)[4], unsigned so, int ks) {
#pragma unroll
      for (int n = 0; n < 4; ++n) {
        bf16x4 lo = __builtin_amdgcn_ds_read_tr16_b64_v4i16((LAS bf16x4*)(lds + baseB[n] + so + ks * 8192));
        bf16x4 hi = __builtin_amdgcn_ds_read_tr16_b64_v4i16((LAS bf16x4*)(lds + baseB[n] + so + ks * 8192 + 1024));
        F[n] = __builtin_shufflevector(lo, hi, 0, 1, 2, 3, 4, 5, 6, 7);
      }
    };
    auto mma = [&](bf16x8 (&A)[4], bf16x8 (&B)[4]) {
#pragma unroll
      for (int i = 0; i < 16; ++i) {
        const int n = i >> 2, m = (n & 1) ? 3 - (i & 3) : (i & 3);
        acc[m][n] = __builtin_amdgcn_mfma_f32_16x16x32_bf16(B[n], A[m], acc[m][n], 0, 0, 0);
      }
    };
    auto interleave = [&]() {
#pragma unroll
      for (int i = 0; i < 12; ++i) { __builtin_amdgcn_sched_group_barrier(0x8, 1, 0); __builtin_amdgcn_sched_group_barrier(0x100, 1, 0); }
      __builtin_amdgcn_sched_group_barrier(0x8, 4, 0);
    };
    auto out_ptr = [&](const Tile& T) {
      return p.out[T.dest & 7] + (size_t)(T.src * MLOC + T.rt * 128 + wr * 64 + fr) * (NSH * 2) + (T.ct * 128 + wc * 64 + 8 * ((fq >> 1) + 2 * (fq & 1))) * 2;
    };
    auto pack = [&]() {
#pragma unroll
      for (int m = 0; m < 4; ++m)
#pragma unroll
        for (int q = 0; q < 2; ++q) {
          const unsigned x0 = cvtpk(gelu1(acc[m][2 * q][0]), gelu1(acc[m][2 * q][1])), x1 = cvtpk(gelu1(acc[m][2 * q][2]), gelu1(acc[m][2 * q][3]));
          const unsigned y0 = cvtpk(gelu1(acc[m][2 * q + 1][0]), gelu1(acc[m][2 * q + 1][1])), y1 = cvtpk(gelu1(acc[m][2 * q + 1][2]), gelu1(acc[m][2 * q + 1][3]));
          const auto r0 = __builtin_amdgcn_permlane16_swap(x0, y0, false, false);
          const auto r1 = __builtin_amdgcn_permlane16_swap(x1, y1, false, false);
          O[m * 2 + q] = (u32x4){r0[0], r1[0], r0[1], r1[1]};
        }
    };
    auto zero = [&]() {
#pragma unroll
      for (int m = 0; m < 4; ++m)
#pragma unroll
        for (int n = 0; n < 4; ++n) acc[m][n] = (f32x4){0.f, 0.f, 0.f, 0.f};
    };
    unsigned soA = 0, soB = 0;
#define HALF1() do { SCHED(); readA(FA[1], (baseA0 ^ 64u) + soA); readB(FB[1], soB, 1); mma(FA[0], FB[0]); interleave(); SCHED(); } while (0)
#define HALF2() do { __builtin_amdgcn_s_waitcnt(0xC07F); __builtin_amdgcn_s_barrier(); SCHED(); \
      soA = soA + SLOT == NSA * SLOT ? 0 : soA + SLOT; soB = soB + SLOT == NSB * SLOT ? 0 : soB + SLOT; \
      readA(FA[0], baseA0 + soA); readB(FB[0], soB, 0); mma(FA[1], FB[1]); interleave(); SCHED(); } while (0)
    unsigned done0, tmov;
    asm volatile("global_load_dword %0, %1, off sc1" : "=v"(done0) : "v"(p.ws[rank] + WS_DONE) : "memory");
    if (wid == 0) TS(0);
    BAR();
    if (wid == 0) TS(1);
    readA(FA[0], baseA0); readB(FB[0], 0, 0);
    zero();
    for (int kt = 0; kt < NKT; ++kt) { HALF1(); HALF2(); }
    if (wid == 0) TS(2);
    pack();
    zero();
    char* o0 = out_ptr(T0);
    auto burst = [&](int m) {
      if (m == 0) { st16_sys(o0, O[0]); st16_sys(o0 + 64, O[1]); }
      else if (m == 1) { st16_sys(o0 + 16 * 4096, O[2]); st16_sys(o0 + 16 * 4096 + 64, O[3]); }
      else if (m == 2) { st16_sys(o0 + 32 * 4096, O[4]); st16_sys(o0 + 32 * 4096 + 64, O[5]); }
      else { st16_sys(o0 + 48 * 4096, O[6]); st16_sys(o0 + 48 * 4096 + 64, O[7]); }
    };
    int nb = 0;
    unsigned tn = next_slot(now32(), oslot, FRAME_O);
    for (int kt = 0; kt < NKT; ++kt) {
      unsigned long long t64;
      __builtin_amdgcn_s_waitcnt(0xC07F);
      asm volatile("s_memrealtime %0" : "=s"(t64));
      HALF1();
      asm volatile("s_waitcnt lgkmcnt(0)" : "+s"(t64) :: "memory");
      const unsigned tnow = (unsigned)t64;
      if ((int)(tnow - tn) >= 0 && nb < 4) { burst(nb); ++nb; tn += FRAME_O; }
      HALF2();
    }
    if (wid == 0) TS(3);
    for (; nb < 4; ++nb) burst(nb);
    asm volatile("global_load_dword %0, %1, off sc1" : "=v"(tmov) : "v"(p.ws[rank] + WS_TMO) : "memory");
    pack();
    SCHED();
    asm volatile("s_waitcnt vmcnt(0)" : "+v"(done0), "+v"(tmov) :: "memory");
    const unsigned call = rfl(done0) / NWG + 1;
    if (rfl(tmov) == call) {
#pragma unroll
      for (int i = 0; i < 8; ++i) O[i] = (u32x4){0x7fc07fc0u, 0x7fc07fc0u, 0x7fc07fc0u, 0x7fc07fc0u};
    }
    char* o1 = out_ptr(T1);
#pragma unroll
    for (int i = 0; i < 8; ++i) st16_sys(o1 + (i >> 1) * 16 * 4096 + (i & 1) * 64, O[i]);
    wait_vm<0>();
    if (wid == 0) TS(4);
    BAR();
    if (wid == 0) {
      if (lane == 0) {
        if (T0.dest != rank) __hip_atomic_fetch_add((unsigned*)(p.ws[T0.dest & 7] + WS_OUTCNT + rank * 128), 1u, __ATOMIC_RELAXED, __HIP_MEMORY_SCOPE_SYSTEM);
        if (T1.dest != rank) __hip_atomic_fetch_add((unsigned*)(p.ws[T1.dest & 7] + WS_OUTCNT + rank * 128), 1u, __ATOMIC_RELAXED, __HIP_MEMORY_SCOPE_SYSTEM);
        __hip_atomic_fetch_add((unsigned*)(myws + WS_DONE), 1u, __ATOMIC_RELAXED, __HIP_MEMORY_SCOPE_AGENT);
      }
      if (b == NWG - 1) {
        const bool ok = wait_ge((const unsigned*)(myws + WS_OUTCNT + peer * 128), call * OUT_PER_DEST, tmo, call, lane < 7);
        if (!ok && lane == 0) *(unsigned short*)(p.out[rank] + (size_t)rank * MLOC * NSH * 2) = 0x7fc0;
        TS(8);
      }
    }
  }
}

extern "C" void kernel_launch(int rank, int nranks, ncclComm_t comm, hipStream_t stream,
                              void* const* d_in, const int* in_sizes, int n_in,
                              void* d_out, int out_size, const int* in_dtypes, int out_dtype,
                              void* const* d_out_all, void* const* d_ws_all, size_t ws_bytes) {
  if (nranks != NR || ws_bytes < WS_XBUF + 2 * XPAR || (unsigned)rank >= (unsigned)NR) { fprintf(stderr, "unsupported configuration\n"); return; }
  HIP_CHECK(hipFuncSetAttribute((const void*)gemm_a2a_kernel, hipFuncAttributeMaxDynamicSharedMemorySize, LDS_BYTES));
  Params p;
  p.x = (const char*)d_in[0]; p.w = (const char*)d_in[1]; p.rank = rank;
  for (int i = 0; i < NR; ++i) { p.out[i] = (char*)d_out_all[i]; p.ws[i] = (char*)d_ws_all[i]; }
  build_desc(p.desc);
  gemm_a2a_kernel<<<NWG, 512, LDS_BYTES, stream>>>(p);
  HIP_CHECK(hipGetLastError());
}
```

```cpp
#include <hip/hip_runtime.h>
#include <rccl/rccl.h>
#include <cstdio>

#define HIP_CHECK(expr) do { hipError_t _e = (expr); if (_e != hipSuccess) { \
    fprintf(stderr, "HIP error %s at %s:%d\n", hipGetErrorString(_e), __FILE__, __LINE__); \
    return; } } while (0)

#define LAS __attribute__((address_space(3)))
typedef short bf16x8 __attribute__((ext_vector_type(8)));
typedef short bf16x4 __attribute__((ext_vector_type(4)));
typedef float f32x4 __attribute__((ext_vector_type(4)));
typedef float f32x2 __attribute__((ext_vector_type(2)));
typedef unsigned u32x4 __attribute__((ext_vector_type(4)));
typedef __bf16 bfx2 __attribute__((ext_vector_type(2)));

#define SX 1
#define TAU_O 16
#define TAU_X2 4
#define XB 4
#define ROT 1
#define AUXA 17
#define NSA 5
#define NSB 5
constexpr int NR = 8;
constexpr int KDIM = 4096, NDIM = 16384, MLOC = 512, NSH = 2048;
constexpr int NKT = 64, SLOT = 16384, NJ = 2 * NKT, NWG = 256;
constexpr int RING_BYTES = (NSA + NSB) * SLOT;
constexpr int LDS_BYTES = RING_BYTES;
constexpr int RR = 4 - SX;
constexpr unsigned OUT_PER_DEST = RR * 16;
constexpr int NREM_XCD = 14 * RR;
constexpr size_t WS_DONE = 0, WS_TMO = 128, WS_OUTCNT = 4096, WS_XCNT = 8192, WS_XLOC = 16384, WS_STAMP = 65536, WS_XBUF = 1u << 20;
constexpr size_t XTILE = 128 * KDIM * 2;
constexpr size_t XPAR = (size_t)NR * 2 * XTILE;
constexpr unsigned NSLOT_O = SX == 0 ? 40 : 37, FRAME_O = NSLOT_O * TAU_O, FRAME_X = NWG * TAU_X2 / 2;
constexpr unsigned long long DEADLINE = 40000000ull;

struct Params { const char* x; const char* w; char* out[NR]; char* ws[NR]; int rank; unsigned desc[NWG]; };
struct Tile { int src; int rt; int dest; int ct; };
__device__ __forceinline__ Tile tile_of(unsigned d, int rank) {
  Tile r; r.src = (rank + (int)(d & 7)) & 7; r.rt = (d >> 3) & 3; r.dest = (rank + (int)((d >> 5) & 7)) & 7; r.ct = (d >> 8) & 15; return r;
}
__device__ __forceinline__ unsigned rot_of(unsigned d) {
  const unsigned so = d & 7, rt = (d >> 3) & 3, dof = (d >> 5) & 7, h = (d >> 8) & 1, xcd = (d >> 9) & 7;
  return (rt + so + dof + (dof ? 3 * h : 2 * h) + xcd) & 3;
}
static inline unsigned tile_code(int so, int rt, int dof, int ct) { return (unsigned)(so | rt << 3 | dof << 5 | ct << 8); }
static void build_desc(unsigned* desc) {
  int cnt[8] = {0}, seen[8] = {0};
  for (int x = 0; x < 8; ++x) {
    unsigned first[64], second[64]; int n1 = 0, n2 = 0;
    const int late = NREM_XCD > 32 ? NREM_XCD - 32 : 0, nfull = late / RR, nsingle = late % RR;
    int state[8][2] = {};
    for (int k = 0; k < nfull; ++k) { const int d = 1 + (nfull * x + k) % 7; state[d][state[d][0] ? 1 : 0] = RR; }
    if (nsingle) { const int d = 1 + 3 + (x & 3); state[d][state[d][1] ? 0 : 1] = nsingle; }
    for (int h = 0; h < 2; ++h) for (int d = 1; d < 8; ++d) for (int rt = 0; rt < RR; ++rt) {
      const unsigned c = tile_code(0, rt, d, 2 * x + h);
      if (rt >= RR - state[d][h]) second[n2++] = c; else first[n1++] = c;
    }
    for (int rs = 0; rs < 4 + 7 * SX; ++rs) for (int h = 0; h < 2; ++h) {
      const unsigned c = rs < 4 ? tile_code(0, rs, 0, 2 * x + h) : tile_code(1 + (rs - 4) / (SX ? SX : 1), RR + (rs - 4) % (SX ? SX : 1), 0, 2 * x + h);
      if (n1 < 32) first[n1++] = c; else second[n2++] = c;
    }
    for (int i = 0; i < 32; ++i) { desc[i * 8 + x] = first[i] | second[i] << 12; cnt[(first[i] >> 5) & 7]++; }
  }
  for (int b = 0; b < NWG; ++b) { const int d = (desc[b] >> 5) & 7; const int k = seen[d]++; desc[b] |= (unsigned)k << 24; (void)cnt; }
}

__device__ __forceinline__ unsigned cvtpk(float lo, float hi) { f32x2 v = {lo, hi}; bfx2 b = __builtin_convertvector(v, bfx2); return __builtin_bit_cast(unsigned, b); }
__device__ __forceinline__ float gelu1(float y) {
  const float k0 = -2.0f * 0.7978845608028654f * 1.4426950408889634f, k1 = k0 * 0.044715f;
  const float z = y * __builtin_fmaf(y * y, k1, k0);
  return y * __builtin_amdgcn_rcpf(1.0f + __builtin_amdgcn_exp2f(z));
}
__device__ __forceinline__ unsigned rfl(unsigned v) { return (unsigned)__builtin_amdgcn_readfirstlane((int)v); }
__device__ __forceinline__ unsigned now32() { return (unsigned)__builtin_amdgcn_s_memrealtime(); }
__device__ __forceinline__ unsigned next_slot(unsigned t, unsigned slot, unsigned frame) { return t - (t - slot) % frame + frame; }
template <int N> __device__ __forceinline__ void wait_vm() { asm volatile("s_waitcnt vmcnt(%0)" :: "i"(N) : "memory"); }
__device__ __forceinline__ void st16_sys(char* p, u32x4 v) { asm volatile("global_store_dwordx4 %0, %1, off sc0 sc1\n\ts_nop 1" :: "v"(p), "v"(v) : "memory"); }
#define BAR() do { asm volatile("" ::: "memory"); __builtin_amdgcn_s_barrier(); asm volatile("" ::: "memory"); } while (0)
#define SCHED() __builtin_amdgcn_sched_barrier(0)
#define TS(k) do {} while (0)

__device__ __forceinline__ bool wait_ge(const unsigned* flag, unsigned want, unsigned* tmo, unsigned call, bool active) {
  const unsigned long long t0 = __builtin_amdgcn_s_memrealtime();
  for (;;) {
    const bool ok = !active || __hip_atomic_load(flag, __ATOMIC_RELAXED, __HIP_MEMORY_SCOPE_SYSTEM) >= want;
    if (__all(ok)) return true;
    if (__hip_atomic_load(tmo, __ATOMIC_RELAXED, __HIP_MEMORY_SCOPE_AGENT) == call || __builtin_amdgcn_s_memrealtime() - t0 > DEADLINE) {
      __hip_atomic_store(tmo, call, __ATOMIC_RELAXED, __HIP_MEMORY_SCOPE_AGENT);
      return false;
    }
    __builtin_amdgcn_s_sleep(2);
  }
}

__global__ void __launch_bounds__(512, 2) gemm_a2a_kernel(Params p) {
  extern __shared__ __attribute__((aligned(16))) char smem[];
  LAS char* lds = (LAS char*)smem;
  const int tid = threadIdx.x, wid = (int)rfl(tid >> 6), lane = tid & 63;
  const int b = blockIdx.x, rank = p.rank;
  const unsigned desc = p.desc[b];
  const Tile T0 = tile_of(desc, rank), T1 = tile_of(desc >> 12, rank);
  const unsigned oslot = (desc >> 24) * TAU_O + ((desc >> 5) & 7) * (TAU_O / 8);
  constexpr int J0 = NKT - NSA;

  char* myws = p.ws[rank];
  unsigned* tmo = (unsigned*)(myws + WS_TMO);
  const unsigned peer = (unsigned)(rank + 1 + lane) & 7;
  if (wid == 7) {
    if (T1.src == rank) return;
    const unsigned call = rfl(__hip_atomic_load((unsigned*)(myws + WS_DONE), __ATOMIC_RELAXED, __HIP_MEMORY_SCOPE_AGENT)) / NWG + 1;
    const unsigned* fl = (const unsigned*)(myws + WS_XCNT + T1.src * 512);
    BAR();
    int it = 0;
#pragma unroll
    for (int c = 0; c < 4; ++c) {
      for (; it < J0 + 16 * c - 5; ++it) BAR();
      const unsigned seen = __hip_atomic_load(fl + c * 32, __ATOMIC_RELAXED, __HIP_MEMORY_SCOPE_SYSTEM);
      for (; it < J0 + 16 * c; ++it) BAR();
      if (rfl(seen) < call) { wait_ge(fl + c * 32, call, tmo, call, true); wait_vm<0>(); }
      if (c == 0) TS(6);
    }
  } else if (wid == 6) {
    const int g = b >> 4;
    u32x4 xv[4];
#pragma unroll
    for (int m = 0; m < XB; ++m)
      xv[m] = *(const u32x4*)(p.x + RR * XTILE + (size_t)((b & 15) * 8 + (lane >> 3)) * 8192 + (16 * m + g) * 128 + ((lane & 7) ^ ((4 * (b & 1) + (lane >> 4)) & 7)) * 16);
    const unsigned call = rfl(__hip_atomic_load((unsigned*)(myws + WS_DONE), __ATOMIC_RELAXED, __HIP_MEMORY_SCOPE_AGENT)) / NWG + 1;
    char* pws = p.ws[0];
#pragma unroll
    for (int q = 1; q < NR; ++q) pws = peer == (unsigned)q ? p.ws[q] : pws;
    const size_t so = WS_XBUF + (call & 1) * XPAR + (size_t)rank * 2 * XTILE + (size_t)g * 16384 + (b & 15) * 1024 + lane * 16;
    __builtin_amdgcn_s_waitcnt(0x0F70);
    BAR();
    int it = 0;
    unsigned tn = next_slot(now32(), (unsigned)b * TAU_X2 / 2, FRAME_X);
#pragma unroll
    for (int m = 0; m < XB; ++m) {
      const int ch = m;
      while ((int)(now32() - tn) < 0 && it < 24 + 16 * m) { BAR(); ++it; }
      tn += FRAME_X;
      const int i0 = it;
#pragma unroll
      for (int q = 0; q < 7; ++q) st16_sys(p.ws[(rank + 1 + q) & 7] + so + m * 16 * 16384, xv[m]);
      if (m == 0) TS(9);
      for (; it < i0 + 5; ++it) BAR();
      wait_vm<0>();
      unsigned tick; unsigned long long sv;
      asm volatile("s_mov_b64 %1, exec\n\ts_mov_b64 exec, 1\n\tglobal_atomic_add %0, %2, %3, off sc0 sc1\n\ts_mov_b64 exec, %1"
                   : "=&v"(tick), "=&s"(sv) : "v"(myws + WS_XLOC + ch * 128), "v"(1u) : "memory");
      for (; it < i0 + 8; ++it) BAR();
      asm volatile("s_waitcnt vmcnt(0)" : "+v"(tick) :: "memory");
      if (rfl(tick) == call * (64 * XB) - 1) {
        if (lane < 7) __hip_atomic_store((unsigned*)(pws + WS_XCNT + rank * 512 + ch * 128), call, __ATOMIC_RELAXED, __HIP_MEMORY_SCOPE_SYSTEM);
      }
    }
    TS(5);
  } else if (wid >= 4) {
    const int pw = wid - 4;
    unsigned done0;
    asm volatile("global_load_dword %0, %1, off sc1" : "=v"(done0) : "v"(p.ws[rank] + WS_DONE) : "memory");
    auto srdA = __builtin_amdgcn_make_buffer_rsrc((void*)(p.x + (size_t)T0.rt * XTILE), 0, (int)XTILE, 0x00020000);
    const auto srdB = __builtin_amdgcn_make_buffer_rsrc((void*)p.w, 0, KDIM * NDIM * 2, 0x00020000);
    unsigned voffA[2], voffB[2];
#pragma unroll
    for (int i = 0; i < 2; ++i) {
      voffA[i] = (unsigned)(lane >> 3) * 8192u + (unsigned)(((lane & 7) ^ ((4 * i + (lane >> 4)) & 7)) * 16);
      voffB[i] = (unsigned)(lane >> 4) * 32768u + (unsigned)(((lane & 15) ^ (((lane >> 4) << 2) | (i << 1))) * 16);
    }
    unsigned sA = (unsigned)(64 * pw) * 8192u, stepK = 128u, stepI = 65536u;
    const unsigned sB0 = (unsigned)(32 * pw) * 32768u + (unsigned)(T0.dest * 16 + T0.ct) * 256u, sB1 = (unsigned)(32 * pw) * 32768u + (unsigned)(T1.dest * 16 + T1.ct) * 256u;
    const unsigned rot0 = ROT ? rot_of(desc) : 0, rot1 = ROT ? rot_of(desc >> 12) : 0;
    auto kperm = [&](int j) { return ((unsigned)j & 60u) | (((unsigned)j + ((j >> 6) ? rot1 : rot0)) & 3u); };
    auto issueA = [&](int j, int s) {
      const unsigned so = sA + kperm(j) * stepK;
      LAS char* d = lds + s * SLOT + pw * 8192;
#pragma unroll
      for (int i = 0; i < 8; ++i) __builtin_amdgcn_raw_ptr_buffer_load_lds(srdA, (LAS void*)(d + i * 1024), 16, voffA[i & 1], so + i * stepI, 0, AUXA);
    };
    auto issueB = [&](int j, int s) {
      const unsigned so = ((j >> 6) ? sB1 : sB0) + kperm(j) * (64u * 32768u);
      LAS char* d = lds + (NSA + s) * SLOT + pw * 8192;
#pragma unroll
      for (int i = 0; i < 8; ++i) __builtin_amdgcn_raw_ptr_buffer_load_lds(srdB, (LAS void*)(d + i * 1024), 16, voffB[(i >> 1) & 1], so + i * (4u * 32768u), 0, 0);
    };
    static_assert(NSA <= NSB, "ring order");
#pragma unroll
    for (int i = -NSB; i < 0; ++i) { if (i + NSA >= 0) issueA(i + NSA, i + NSA); issueB(i + NSB, i + NSB); }
    constexpr int NW = (NSA == NSB) ? 16 * (NSA - 2) : 8 + 16 * (NSA - 2);
    static_assert(NW <= 63, "vmcnt range");
    asm volatile("s_waitcnt vmcnt(%1)" : "+v"(done0) : "i"(NW + 16 > 63 ? 63 : NW + 16) : "memory");
    BAR();
    int sa = 0, sb = 0;
    for (int j = 0; j < NJ - NSB; ++j) {
      wait_vm<NW>(); BAR();
      if (j == J0) {
        const char* a1 = p.x + (size_t)T1.rt * XTILE;
        if (T1.src != rank) {
          a1 = p.ws[rank] + WS_XBUF + ((rfl(done0) / NWG + 1) & 1) * XPAR + (size_t)(T1.src * 2 + T1.rt - RR) * XTILE;
          sA = (unsigned)pw * 8192u; stepK = 16384u; stepI = 1024u; voffA[0] = voffA[1] = (unsigned)lane * 16u;
        }
        srdA = __builtin_amdgcn_make_buffer_rsrc((void*)a1, 0, (int)XTILE, 0x00020000);
      }
      issueA(j + NSA, sa); issueB(j + NSB, sb);
      sa = sa + 1 == NSA ? 0 : sa + 1; sb = sb + 1 == NSB ? 0 : sb + 1;
    }
#pragma unroll
    for (int j = NJ - NSB; j < NJ; ++j) {
      int cnt = 0;
      const int iA = j + 1 - NSA;
      if (NSA != NSB && iA + NSB < NJ) cnt += 8;
      for (int i = iA + 1; i <= j - 1; ++i) { if (i + NSA < NJ) cnt += 8; if (i + NSB < NJ) cnt += 8; }
      if (j + 1 >= NJ) cnt = 0;
      switch (cnt) {
        case 8: wait_vm<8>(); break; case 16: wait_vm<16>(); break; case 24: wait_vm<24>(); break; case 32: wait_vm<32>(); break;
        case 40: wait_vm<40>(); break; case 48: wait_vm<48>(); break; default: wait_vm<0>(); break;
      }
      BAR();
      if (j + NSA < NJ) { issueA(j + NSA, sa); sa = sa + 1 == NSA ? 0 : sa + 1; }
    }
  } else {
    const int wr = wid >> 1, wc = wid & 1, fr = lane & 15, fq = lane >> 4;
    const unsigned baseA0 = (unsigned)(wr * 64 + fr) * 128u + (unsigned)((fq ^ ((fr >> 1) & 7)) * 16);
    unsigned baseB[4];
    { const int q = (lane & 15) >> 2, pp = lane & 3; const int s = (q << 2) | ((fq & 1) << 1);
#pragma unroll
      for (int n = 0; n < 4; ++n) baseB[n] = (unsigned)(NSA * SLOT) + (unsigned)(8 * fq + q) * 256u + (unsigned)(((8 * wc + 2 * n) ^ s) * 16 + 8 * pp); }
    f32x4 acc[4][4];
    bf16x8 FA[2][4], FB[2][4];
    u32x4 O[8];
    auto readA = [&](bf16x8 (&F)[4], unsigned addr) {
#pragma unroll
      for (int m = 0; m < 4; ++m) F[m] = *(const LAS bf16x8*)(lds + addr + m * 2048);
    };
    auto readB = [&](bf16x8 (&F)[4], unsigned so, int ks) {
#pragma unroll
      for (int n = 0; n < 4; ++n) {
        bf16x4 lo = __builtin_amdgcn_ds_read_tr16_b64_v4i16((LAS bf16x4*)(lds + baseB[n] + so + ks * 8192));
        bf16x4 hi = __builtin_amdgcn_ds_read_tr16_b64_v4i16((LAS bf16x4*)(lds + baseB[n] + so + ks * 8192 + 1024));
        F[n] = __builtin_shufflevector(lo, hi, 0, 1, 2, 3, 4, 5, 6, 7);
      }
    };
    auto mma = [&](bf16x8 (&A)[4], bf16x8 (&B)[4]) {
#pragma unroll
      for (int i = 0; i < 16; ++i) {
        const int n = i >> 2, m = (n & 1) ? 3 - (i & 3) : (i & 3);
        acc[m][n] = __builtin_amdgcn_mfma_f32_16x16x32_bf16(B[n], A[m], acc[m][n], 0, 0, 0);
      }
    };
    auto interleave = [&]() {
#pragma unroll
      for (int i = 0; i < 12; ++i) { __builtin_amdgcn_sched_group_barrier(0x8, 1, 0); __builtin_amdgcn_sched_group_barrier(0x100, 1, 0); }
      __builtin_amdgcn_sched_group_barrier(0x8, 4, 0);
    };
    auto out_ptr = [&](const Tile& T) {
      return p.out[T.dest & 7] + (size_t)(T.src * MLOC + T.rt * 128 + wr * 64 + fr) * (NSH * 2) + (T.ct * 128 + wc * 64 + 8 * ((fq >> 1) + 2 * (fq & 1))) * 2;
    };
    auto pack = [&]() {
#pragma unroll
      for (int m = 0; m < 4; ++m)
#pragma unroll
        for (int q = 0; q < 2; ++q) {
          const unsigned x0 = cvtpk(gelu1(acc[m][2 * q][0]), gelu1(acc[m][2 * q][1])), x1 = cvtpk(gelu1(acc[m][2 * q][2]), gelu1(acc[m][2 * q][3]));
          const unsigned y0 = cvtpk(gelu1(acc[m][2 * q + 1][0]), gelu1(acc[m][2 * q + 1][1])), y1 = cvtpk(gelu1(acc[m][2 * q + 1][2]), gelu1(acc[m][2 * q + 1][3]));
          const auto r0 = __builtin_amdgcn_permlane16_swap(x0, y0, false, false);
          const auto r1 = __builtin_amdgcn_permlane16_swap(x1, y1, false, false);
          O[m * 2 + q] = (u32x4){r0[0], r1[0], r0[1], r1[1]};
        }
    };
    auto zero = [&]() {
#pragma unroll
      for (int m = 0; m < 4; ++m)
#pragma unroll
        for (int n = 0; n < 4; ++n) acc[m][n] = (f32x4){0.f, 0.f, 0.f, 0.f};
    };
    unsigned soA = 0, soB = 0;
#define HALF1() do { SCHED(); readA(FA[1], (baseA0 ^ 64u) + soA); readB(FB[1], soB, 1); mma(FA[0], FB[0]); interleave(); SCHED(); } while (0)
#define HALF2() do { __builtin_amdgcn_s_waitcnt(0xC07F); __builtin_amdgcn_s_barrier(); SCHED(); \
      soA = soA + SLOT == NSA * SLOT ? 0 : soA + SLOT; soB = soB + SLOT == NSB * SLOT ? 0 : soB + SLOT; \
      readA(FA[0], baseA0 + soA); readB(FB[0], soB, 0); mma(FA[1], FB[1]); interleave(); SCHED(); } while (0)
    unsigned done0, tmov;
    asm volatile("global_load_dword %0, %1, off sc1" : "=v"(done0) : "v"(p.ws[rank] + WS_DONE) : "memory");
    if (wid == 0) TS(0);
    BAR();
    if (wid == 0) TS(1);
    readA(FA[0], baseA0); readB(FB[0], 0, 0);
    zero();
    for (int kt = 0; kt < NKT; ++kt) { HALF1(); HALF2(); }
    if (wid == 0) TS(2);
    pack();
    zero();
    char* o0 = out_ptr(T0);
    auto burst = [&](int m) {
      if (m == 0) { st16_sys(o0, O[0]); st16_sys(o0 + 64, O[1]); }
      else if (m == 1) { st16_sys(o0 + 16 * 4096, O[2]); st16_sys(o0 + 16 * 4096 + 64, O[3]); }
      else if (m == 2) { st16_sys(o0 + 32 * 4096, O[4]); st16_sys(o0 + 32 * 4096 + 64, O[5]); }
      else { st16_sys(o0 + 48 * 4096, O[6]); st16_sys(o0 + 48 * 4096 + 64, O[7]); }
    };
    int nb = 0;
    unsigned tn = next_slot(now32(), oslot, FRAME_O);
    for (int kt = 0; kt < NKT; ++kt) {
      unsigned long long t64;
      __builtin_amdgcn_s_waitcnt(0xC07F);
      asm volatile("s_memrealtime %0" : "=s"(t64));
      HALF1();
      asm volatile("s_waitcnt lgkmcnt(0)" : "+s"(t64) :: "memory");
      const unsigned tnow = (unsigned)t64;
      if ((int)(tnow - tn) >= 0 && nb < 4) { burst(nb); ++nb; tn += FRAME_O; }
      HALF2();
    }
    if (wid == 0) TS(3);
    for (; nb < 4; ++nb) burst(nb);
    asm volatile("global_load_dword %0, %1, off sc1" : "=v"(tmov) : "v"(p.ws[rank] + WS_TMO) : "memory");
    pack();
    SCHED();
    asm volatile("s_waitcnt vmcnt(0)" : "+v"(done0), "+v"(tmov) :: "memory");
    const unsigned call = rfl(done0) / NWG + 1;
    if (rfl(tmov) == call) {
#pragma unroll
      for (int i = 0; i < 8; ++i) O[i] = (u32x4){0x7fc07fc0u, 0x7fc07fc0u, 0x7fc07fc0u, 0x7fc07fc0u};
    }
    char* o1 = out_ptr(T1);
#pragma unroll
    for (int i = 0; i < 8; ++i) st16_sys(o1 + (i >> 1) * 16 * 4096 + (i & 1) * 64, O[i]);
    wait_vm<0>();
    if (wid == 0) TS(4);
    BAR();
    if (wid == 0) {
      if (lane == 0) {
        if (T0.dest != rank) __hip_atomic_fetch_add((unsigned*)(p.ws[T0.dest & 7] + WS_OUTCNT + rank * 128), 1u, __ATOMIC_RELAXED, __HIP_MEMORY_SCOPE_SYSTEM);
        if (T1.dest != rank) __hip_atomic_fetch_add((unsigned*)(p.ws[T1.dest & 7] + WS_OUTCNT + rank * 128), 1u, __ATOMIC_RELAXED, __HIP_MEMORY_SCOPE_SYSTEM);
        __hip_atomic_fetch_add((unsigned*)(myws + WS_DONE), 1u, __ATOMIC_RELAXED, __HIP_MEMORY_SCOPE_AGENT);
      }
      if (b == NWG - 1) {
        const bool ok = wait_ge((const unsigned*)(myws + WS_OUTCNT + peer * 128), call * OUT_PER_DEST, tmo, call, lane < 7);
        if (!ok && lane == 0) *(unsigned short*)(p.out[rank] + (size_t)rank * MLOC * NSH * 2) = 0x7fc0;
        TS(8);
      }
    }
  }
}

extern "C" void kernel_launch(int rank, int nranks, ncclComm_t comm, hipStream_t stream,
                              void* const* d_in, const int* in_sizes, int n_in,
                              void* d_out, int out_size, const int* in_dtypes, int out_dtype,
                              void* const* d_out_all, void* const* d_ws_all, size_t ws_bytes) {
  if (nranks != NR || ws_bytes < WS_XBUF + 2 * XPAR || (unsigned)rank >= (unsigned)NR) { fprintf(stderr, "unsupported configuration\n"); return; }
  HIP_CHECK(hipFuncSetAttribute((const void*)gemm_a2a_kernel, hipFuncAttributeMaxDynamicSharedMemorySize, LDS_BYTES));
  Params p;
  p.x = (const char*)d_in[0]; p.w = (const char*)d_in[1]; p.rank = rank;
  for (int i = 0; i < NR; ++i) { p.out[i] = (char*)d_out_all[i]; p.ws[i] = (char*)d_ws_all[i]; }
  build_desc(p.desc);
  gemm_a2a_kernel<<<NWG, 512, LDS_BYTES, stream>>>(p);
  HIP_CHECK(hipGetLastError());
}
```

```cpp
#include <hip/hip_runtime.h>
#include <rccl/rccl.h>
#include <cstdio>

#define HIP_CHECK(expr) do { hipError_t _e = (expr); if (_e != hipSuccess) { \
    fprintf(stderr, "HIP error %s at %s:%d\n", hipGetErrorString(_e), __FILE__, __LINE__); \
    return; } } while (0)

#define LAS __attribute__((address_space(3)))
typedef short bf16x8 __attribute__((ext_vector_type(8)));
typedef short bf16x4 __attribute__((ext_vector_type(4)));
typedef float f32x4 __attribute__((ext_vector_type(4)));
typedef float f32x2 __attribute__((ext_vector_type(2)));
typedef unsigned u32x4 __attribute__((ext_vector_type(4)));
typedef __bf16 bfx2 __attribute__((ext_vector_type(2)));

#define SX 1
#define TAU_O 16
#define TAU_X2 4
#define XB 1
#define OB 2
#define LMAP 1
#define SEG128 1
#define ODD (SEG128 ? 8 * 4096 : 64)
#define REV1 0
#define REV2 0
#define XTOUCH 0
#define XT0 32
#define XT1 12
#define GSEL(n) (n)
#define AUXA 17
#define NS 5
#define NSA NS
#define NSB NS
constexpr int NR = 8;
constexpr int KDIM = 4096, NDIM = 16384, MLOC = 512, NSH = 2048;
constexpr int NKT = 64, SLOT = 16384, NJ = 2 * NKT, NWG = 256;
constexpr int RING_BYTES = (NSA + NSB) * SLOT;
constexpr int LDS_BYTES = RING_BYTES;
constexpr int RR = 4 - SX;
constexpr unsigned OUT_PER_DEST = RR * 16;
constexpr int Q4 = 80;
constexpr size_t WS_DONE = 0, WS_TMO = 128, WS_OUTCNT = 4096, WS_XCNT = 8192, WS_XLOC = 16384, WS_STAMP = 65536, WS_XBUF = 1u << 20;
constexpr size_t XTILE = 128 * KDIM * 2;
constexpr size_t XPAR = (size_t)NR * 2 * XTILE;
constexpr unsigned NSLOT_O = 37, OL = 4 / OB, FRAME_O = NSLOT_O * TAU_O * OL, XI = 4 / XB;
constexpr unsigned long long DEADLINE = 40000000ull;

struct Params { const char* x; const char* w; char* out[NR]; char* ws[NR]; int rank; unsigned desc[NWG]; unsigned char slot[NWG]; };
struct Tile { int src; int rt; int dest; int ct; };
__device__ __forceinline__ Tile tile_of(unsigned d, int rank) {
  Tile r; r.src = (rank + (int)((d >> 13) & 7)) & 7; r.rt = d & 3; r.dest = (rank + (int)((d >> 2) & 7)) & 7; r.ct = (d >> 5) & 15; return r;
}
static inline unsigned tile_code(int so, int rt, int dof, int ct, int G, int rot) { return (unsigned)(rt | dof << 2 | ct << 5 | (G - 1) << 9 | (rot % G) << 11 | so << 13); }
static void build_desc(unsigned* desc, unsigned char* slot) {
  unsigned T[2][8][32]; int n[2][8] = {};
  auto put = [&](int w, int x, unsigned c) { T[w][x][n[w][x]++] = c; };
  auto panel = [&](int w, int x, int q, int rt) { put(w, x, tile_code(0, rt, 1 + q % 7, q / 7, q == Q4 ? 4 : 3, rt + q)); };
  for (int q = 0; q < 80; ++q) for (int rt = 0; rt < 3; ++rt) panel(0, q / 10, q, rt);
  for (int i = 0; i < 16; ++i) panel(0, i / 2, i < 12 ? 81 + i / 3 : Q4, i < 12 ? i % 3 : i - 12);
  for (int q = 85; q < 109; ++q) for (int rt = 0; rt < 3; ++rt) panel(1, (q - 85) / 6, q, rt);
  for (int k = 0; k < 3; ++k) { panel(1, k, 109 + k, 0); panel(1, k, 109 + k, 1); }
  panel(1, 3, 109, 2); panel(1, 3, 110, 2);
  for (int x = 0; x < 4; ++x) for (int c = 0; c < 4; ++c) for (int r = 0; r < 3; ++r) put(1, x, tile_code(0, r, 0, 4 * x + c, 3, r + c));
  for (int g = 0; g < 2; ++g) for (int h = 0; h < 2; ++h) for (int c = 0; c < 8; ++c) for (int r = 0; r < 4; ++r) {
    if (4 * g + r == 8 - (1 + Q4 % 7) && 8 * h + c == Q4 / 7) panel(1, 4 + 2 * g + h, 111, 2);
    else put(1, 4 + 2 * g + h, tile_code(4 * g + r, 3, 0, 8 * h + c, 4, r + c + 2 * h + g));
  }
  int seen[8] = {0}, cnt[8] = {0};
  for (int b = 0; b < NWG; ++b) { desc[b] = T[0][b & 7][b >> 3] | T[1][b & 7][b >> 3] << 16; cnt[(desc[b] >> 2) & 7]++; }
  for (int b = 0; b < NWG; ++b) { const int d = (desc[b] >> 2) & 7; slot[b] = (unsigned char)((seen[d]++ * 256 + d * 36) / cnt[d]); }
}

__device__ __forceinline__ unsigned cvtpk(float lo, float hi) { f32x2 v = {lo, hi}; bfx2 b = __builtin_convertvector(v, bfx2); return __builtin_bit_cast(unsigned, b); }
__device__ __forceinline__ float gelu1(float y) {
  const float k0 = -2.0f * 0.7978845608028654f * 1.4426950408889634f, k1 = k0 * 0.044715f;
  const float z = y * __builtin_fmaf(y * y, k1, k0);
  return y * __builtin_amdgcn_rcpf(1.0f + __builtin_amdgcn_exp2f(z));
}
__device__ __forceinline__ unsigned rfl(unsigned v) { return (unsigned)__builtin_amdgcn_readfirstlane((int)v); }
__device__ __forceinline__ unsigned now32() { return (unsigned)__builtin_amdgcn_s_memrealtime(); }
__device__ __forceinline__ unsigned next_slot(unsigned t, unsigned slot, unsigned frame) { return t - (t - slot) % frame + frame; }
template <int N> __device__ __forceinline__ void wait_vm() { asm volatile("s_waitcnt vmcnt(%0)" :: "i"(N) : "memory"); }
__device__ __forceinline__ void st16_sys(char* p, u32x4 v) { asm volatile("global_store_dwordx4 %0, %1, off sc0 sc1\n\ts_nop 1" :: "v"(p), "v"(v) : "memory"); }
#define BAR() do { asm volatile("" ::: "memory"); __builtin_amdgcn_s_barrier(); asm volatile("" ::: "memory"); } while (0)
#define SCHED() __builtin_amdgcn_sched_barrier(0)
#define TS(k) do {} while (0)

__device__ __forceinline__ bool wait_ge(const unsigned* flag, unsigned want, unsigned* tmo, unsigned call, bool active) {
  const unsigned long long t0 = __builtin_amdgcn_s_memrealtime();
  for (;;) {
    const bool ok = !active || __hip_atomic_load(flag, __ATOMIC_RELAXED, __HIP_MEMORY_SCOPE_SYSTEM) >= want;
    if (__all(ok)) return true;
    if (__hip_atomic_load(tmo, __ATOMIC_RELAXED, __HIP_MEMORY_SCOPE_AGENT) == call || __builtin_amdgcn_s_memrealtime() - t0 > DEADLINE) {
      __hip_atomic_store(tmo, call, __ATOMIC_RELAXED, __HIP_MEMORY_SCOPE_AGENT);
      return false;
    }
    __builtin_amdgcn_s_sleep(2);
  }
}

__global__ void __launch_bounds__(512, 2) gemm_a2a_kernel(Params p) {
  extern __shared__ __attribute__((aligned(16))) char smem[];
  LAS char* lds = (LAS char*)smem;
  const int tid = threadIdx.x, wid = (int)rfl(tid >> 6), lane = tid & 63;
  const int b = blockIdx.x, rank = p.rank;
  const unsigned desc = p.desc[b];
  const Tile T0 = tile_of(desc, rank), T1 = tile_of(desc >> 16, rank);
  const unsigned oslot = (p.slot[b] * FRAME_O) >> 8;
  constexpr int J0 = NKT - NSA;

  char* myws = p.ws[rank];
  unsigned* tmo = (unsigned*)(myws + WS_TMO);
  const unsigned peer = (unsigned)(rank + 1 + lane) & 7;
  if (wid == 7) {
    if (T1.src == rank) return;
    const unsigned call = rfl(__hip_atomic_load((unsigned*)(myws + WS_DONE), __ATOMIC_RELAXED, __HIP_MEMORY_SCOPE_AGENT)) / NWG + 1;
    const unsigned* fl = (const unsigned*)(myws + WS_XCNT + T1.src * 512);
    BAR();
    int it = 0;
#pragma unroll
    for (int c = 0; c < 4; ++c) {
      for (; it < J0 + 16 * c - 5; ++it) BAR();
      const unsigned seen = __hip_atomic_load(fl + (REV2 ? 3 - c : c) * 32, __ATOMIC_RELAXED, __HIP_MEMORY_SCOPE_SYSTEM);
      for (; it < J0 + 16 * c; ++it) BAR();
      if (rfl(seen) < call) { wait_ge(fl + (REV2 ? 3 - c : c) * 32, call, tmo, call, true); wait_vm<0>(); }
      if (c == 0) TS(6);
    }
  } else if (wid == 6) {
    u32x4 xv[4];
#pragma unroll
    for (int m = 0; m < XB; ++m)
#pragma unroll
      for (int i = 0; i < (int)XI; ++i) {
        const int P = (m * NWG + b) * XI + i;
        xv[m * XI + i] = *(const u32x4*)(p.x + RR * XTILE + (size_t)((P & 15) * 8 + (lane >> 3)) * 8192 + (P >> 4) * 128 + ((lane & 7) ^ ((4 * (P & 1) + (lane >> 4)) & 7)) * 16);
      }
    const unsigned call = rfl(__hip_atomic_load((unsigned*)(myws + WS_DONE), __ATOMIC_RELAXED, __HIP_MEMORY_SCOPE_AGENT)) / NWG + 1;
    char* pws = p.ws[0];
#pragma unroll
    for (int q = 1; q < NR; ++q) pws = peer == (unsigned)q ? p.ws[q] : pws;
    const size_t so = WS_XBUF + (call & 1) * XPAR + (size_t)rank * 2 * XTILE + (size_t)b * XI * 1024 + lane * 16;
    __builtin_amdgcn_s_waitcnt(0x0F70);
    BAR();
    int it = 0;
    unsigned tn = now32() + (unsigned)b * XI * TAU_X2 / 2;
#pragma unroll
    for (int m = 0; m < XB; ++m) {
      const int ch = ((m * NWG + b) * XI) >> 8;
      while ((int)(now32() - tn) < 0 && it < 40 + 16 * ch) { BAR(); ++it; }
      tn += NWG * XI * TAU_X2 / 2;
      const int i0 = it;
#pragma unroll
      for (int i = 0; i < (int)XI; ++i)
#pragma unroll
        for (int q = 0; q < 7; ++q) st16_sys(p.ws[(rank + 1 + q) & 7] + so + (m * NWG * XI + i) * 1024, xv[m * XI + i]);
      if (m == 0) TS(9);
      for (; it < i0 + 5; ++it) BAR();
      wait_vm<0>();
      unsigned tick; unsigned long long sv;
      asm volatile("s_mov_b64 %1, exec\n\ts_mov_b64 exec, 1\n\tglobal_atomic_add %0, %2, %3, off sc0 sc1\n\ts_mov_b64 exec, %1"
                   : "=&v"(tick), "=&s"(sv) : "v"(myws + WS_XLOC + ch * 128), "v"(1u) : "memory");
      for (; it < i0 + 8; ++it) BAR();
      asm volatile("s_waitcnt vmcnt(0)" : "+v"(tick) :: "memory");
      if (rfl(tick) == call * (64 * XB) - 1) {
        if (lane < 7) __hip_atomic_store((unsigned*)(pws + WS_XCNT + rank * 512 + ch * 128), call, __ATOMIC_RELAXED, __HIP_MEMORY_SCOPE_SYSTEM);
      }
    }
    TS(5);
  } else if (wid >= 4) {
    const int pw = wid - 4;
    unsigned done0;
    asm volatile("global_load_dword %0, %1, off sc1" : "=v"(done0) : "v"(p.ws[rank] + WS_DONE) : "memory");
    auto srdA = __builtin_amdgcn_make_buffer_rsrc((void*)(p.x + (size_t)T0.rt * XTILE), 0, (int)XTILE, 0x00020000);
    const auto srdB = __builtin_amdgcn_make_buffer_rsrc((void*)p.w, 0, KDIM * NDIM * 2, 0x00020000);
    unsigned voffA[2], voffB[2];
#pragma unroll
    for (int i = 0; i < 2; ++i) {
      voffA[i] = (unsigned)(lane >> 3) * 8192u + (unsigned)(((lane & 7) ^ ((4 * i + (lane >> 4)) & 7)) * 16);
      voffB[i] = (unsigned)(lane >> 4) * 32768u + (unsigned)(((lane & 15) ^ (((lane >> 4) << 2) | (i << 1))) * 16);
    }
    unsigned sA = (unsigned)(64 * pw) * 8192u, stepK = 128u, stepI = 65536u;
    unsigned sB = (unsigned)(32 * pw) * 32768u + (unsigned)(T0.dest * 16 + T0.ct) * 256u;
    unsigned G = ((desc >> 9) & 3) + 1, kv = (desc >> 11) & 3, kp = 0, kb = 0, kx = REV1 ? 63u : 0u;
    unsigned soA, soB, ldsb = (unsigned)pw * 8192u;
    auto next = [&]() {
      const unsigned kt = min(kb + kv, 63u) ^ kx;
      soA = sA + kt * stepK; soB = sB + kt * (64u * 32768u);
      kv = kv + 1 == G ? 0 : kv + 1; ++kp;
      if (kp == G) { kp = 0; kb += G; }
    };
    auto issue = [&]() {
      LAS char* d = lds + ldsb;
#pragma unroll
      for (int i = 0; i < 8; ++i) __builtin_amdgcn_raw_ptr_buffer_load_lds(srdA, (LAS void*)(d + i * 1024), 16, voffA[i & 1], soA + i * stepI, 0, AUXA);
#pragma unroll
      for (int i = 0; i < 8; ++i) __builtin_amdgcn_raw_ptr_buffer_load_lds(srdB, (LAS void*)(d + NS * SLOT + i * 1024), 16, voffB[(i >> 1) & 1], soB + i * (4u * 32768u), 0, 0);
      ldsb = ldsb + SLOT >= NS * SLOT ? ldsb + SLOT - NS * SLOT : ldsb + SLOT;
    };
#pragma unroll
    for (int i = 0; i < NS; ++i) { next(); issue(); }
    constexpr int NW = 16 * (NS - 2);
    static_assert(NW <= 63, "vmcnt range");
    next();
    asm volatile("s_waitcnt vmcnt(%1)" : "+v"(done0) : "i"(NW + 16 > 63 ? 63 : NW + 16) : "memory");
    BAR();
    for (int j = 0; j < NJ - NS; ++j) {
      wait_vm<NW>(); BAR();
      issue();
      SCHED();
      if (j == J0 - 1) {
        const char* a1 = p.x + (size_t)T1.rt * XTILE;
        if (T1.src != rank) {
          a1 = p.ws[rank] + WS_XBUF + ((rfl(done0) / NWG + 1) & 1) * XPAR + (size_t)(T1.src * 2 + T1.rt - RR) * XTILE;
          sA = (unsigned)pw * 8192u; stepK = 16384u; stepI = 1024u; voffA[0] = voffA[1] = (unsigned)lane * 16u;
        }
        srdA = __builtin_amdgcn_make_buffer_rsrc((void*)a1, 0, (int)XTILE, 0x00020000);
        sB = (unsigned)(32 * pw) * 32768u + (unsigned)(T1.dest * 16 + T1.ct) * 256u;
        G = ((desc >> 25) & 3) + 1; kv = (desc >> 27) & 3; kp = 0; kb = 0; kx = REV2 ? 63u : 0u;
      }
      next();
    }
    wait_vm<NW>(); BAR();
    if (NS > 4) { wait_vm<32>(); BAR(); }
    if (NS > 3) { wait_vm<16>(); BAR(); }
    wait_vm<0>(); BAR();
    BAR();
  } else {
    const int wr = wid >> 1, wc = wid & 1, fr = lane & 15, fq = lane >> 4;
    const unsigned baseA0 = (unsigned)(wr * 64 + fr) * 128u + (unsigned)((fq ^ ((fr >> 1) & 7)) * 16);
    unsigned baseB[4];
    { const int q = (lane & 15) >> 2, pp = lane & 3; const int s = (q << 2) | ((fq & 1) << 1);
#pragma unroll
      for (int n = 0; n < 4; ++n) baseB[n] = (unsigned)(NSA * SLOT) + (unsigned)(8 * fq + q) * 256u + (unsigned)(((8 * wc + 2 * n) ^ s) * 16 + 8 * pp); }
    f32x4 acc[4][4];
    bf16x8 FA[2][4], FB[2][4];
    u32x4 O[8];
    auto readA = [&](bf16x8 (&F)[4], unsigned addr) {
#pragma unroll
      for (int m = 0; m < 4; ++m) F[m] = *(const LAS bf16x8*)(lds + addr + m * 2048);
    };
    auto readB = [&](bf16x8 (&F)[4], unsigned so, int ks) {
#pragma unroll
      for (int n = 0; n < 4; ++n) {
        bf16x4 lo = __builtin_amdgcn_ds_read_tr16_b64_v4i16((LAS bf16x4*)(lds + baseB[n] + so + ks * 8192));
        bf16x4 hi = __builtin_amdgcn_ds_read_tr16_b64_v4i16((LAS bf16x4*)(lds + baseB[n] + so + ks * 8192 + 1024));
        F[n] = __builtin_shufflevector(lo, hi, 0, 1, 2, 3, 4, 5, 6, 7);
      }
    };
    auto mma = [&](bf16x8 (&A)[4], bf16x8 (&B)[4]) {
#pragma unroll
      for (int i = 0; i < 16; ++i) {
        const int n = i >> 2, m = (n & 1) ? 3 - (i & 3) : (i & 3);
        acc[m][n] = __builtin_amdgcn_mfma_f32_16x16x32_bf16(B[n], A[m], acc[m][n], 0, 0, 0);
      }
    };
    auto interleave = [&]() {
#pragma unroll
      for (int i = 0; i < 12; ++i) { __builtin_amdgcn_sched_group_barrier(0x8, 1, 0); __builtin_amdgcn_sched_group_barrier(0x100, 1, 0); }
      __builtin_amdgcn_sched_group_barrier(0x8, 4, 0);
    };
    auto out_ptr = [&](const Tile& T) {
      return p.out[T.dest & 7] + (size_t)(T.src * MLOC + T.rt * 128 + wr * 64 + (fr & 7)) * (NSH * 2) + (T.ct * 128 + wc * 64 + 32 * (fr >> 3) + 8 * ((fq >> 1) + 2 * (fq & 1))) * 2;
    };
    auto pack = [&]() {
#pragma unroll
      for (int m = 0; m < 4; ++m)
#pragma unroll
        for (int q = 0; q < 2; ++q) {
          const unsigned x0 = cvtpk(gelu1(acc[m][2 * q][0]), gelu1(acc[m][2 * q][1])), x1 = cvtpk(gelu1(acc[m][2 * q][2]), gelu1(acc[m][2 * q][3]));
          const unsigned y0 = cvtpk(gelu1(acc[m][2 * q + 1][0]), gelu1(acc[m][2 * q + 1][1])), y1 = cvtpk(gelu1(acc[m][2 * q + 1][2]), gelu1(acc[m][2 * q + 1][3]));
          const auto r0 = __builtin_amdgcn_permlane16_swap(x0, y0, false, false);
          const auto r1 = __builtin_amdgcn_permlane16_swap(x1, y1, false, false);
          O[m * 2 + q] = (u32x4){r0[0], r1[0], r0[1], r1[1]};
        }
#pragma unroll
      for (int m = 0; m < 4; ++m)
#pragma unroll
        for (int e = 0; e < 4; ++e) {
          const unsigned s0 = O[2 * m][e], s1 = O[2 * m + 1][e];
          O[2 * m][e] = (unsigned)__builtin_amdgcn_update_dpp((int)s0, (int)s1, 0x128, 0xf, 0xc, false);
          O[2 * m + 1][e] = (unsigned)__builtin_amdgcn_update_dpp((int)s1, (int)s0, 0x128, 0xf, 0x3, false);
        }
    };
    auto zero = [&]() {
#pragma unroll
      for (int m = 0; m < 4; ++m)
#pragma unroll
        for (int n = 0; n < 4; ++n) acc[m][n] = (f32x4){0.f, 0.f, 0.f, 0.f};
    };
    unsigned soA = 0, soB = 0;
#define HALF1() do { SCHED(); readA(FA[1], (baseA0 ^ 64u) + soA); readB(FB[1], soB, 1); mma(FA[0], FB[0]); interleave(); SCHED(); } while (0)
#define HALF2() do { __builtin_amdgcn_s_waitcnt(0xC07F); __builtin_amdgcn_s_barrier(); SCHED(); \
      soA = soA + SLOT == NSA * SLOT ? 0 : soA + SLOT; soB = soB + SLOT == NSB * SLOT ? 0 : soB + SLOT; \
      readA(FA[0], baseA0 + soA); readB(FB[0], soB, 0); mma(FA[1], FB[1]); interleave(); SCHED(); } while (0)
    unsigned done0, tmov;
    asm volatile("global_load_dword %0, %1, off sc1" : "=v"(done0) : "v"(p.ws[rank] + WS_DONE) : "memory");
    if (wid == 0) TS(0);
    BAR();
    if (wid == 0) TS(1);
    readA(FA[0], baseA0); readB(FB[0], 0, 0);
    zero();
    for (int kt = 0; kt < NKT; ++kt) {
      HALF1();
      HALF2();
    }
    if (wid == 0) TS(2);
    pack();
    zero();
    char* o0 = out_ptr(T0);
    auto burst = [&](int m) {
      if (m == 0) { st16_sys(o0, O[0]); st16_sys(o0 + ODD, O[1]); }
      else if (m == 1) { st16_sys(o0 + 16 * 4096, O[2]); st16_sys(o0 + 16 * 4096 + ODD, O[3]); }
      else if (m == 2) { st16_sys(o0 + 32 * 4096, O[4]); st16_sys(o0 + 32 * 4096 + ODD, O[5]); }
      else { st16_sys(o0 + 48 * 4096, O[6]); st16_sys(o0 + 48 * 4096 + ODD, O[7]); }
    };
    int nb = 0;
    unsigned tn = next_slot(now32(), oslot, FRAME_O);
    for (int kt = 0; kt < NKT; ++kt) {
      unsigned long long t64;
      __builtin_amdgcn_s_waitcnt(0xC07F);
      asm volatile("s_memrealtime %0" : "=s"(t64));
      HALF1();
      asm volatile("s_waitcnt lgkmcnt(0)" : "+s"(t64) :: "memory");
      const unsigned tnow = (unsigned)t64;
      if ((int)(tnow - tn) >= 0 && nb < 4) { burst(nb); ++nb; if (nb % OL == 0) tn += FRAME_O; }
      HALF2();
    }
    if (wid == 0) TS(3);
    for (; nb < 4; ++nb) burst(nb);
    asm volatile("global_load_dword %0, %1, off sc1" : "=v"(tmov) : "v"(p.ws[rank] + WS_TMO) : "memory");
    pack();
    SCHED();
    asm volatile("s_waitcnt vmcnt(0)" : "+v"(done0), "+v"(tmov) :: "memory");
    const unsigned call = rfl(done0) / NWG + 1;
    if (rfl(tmov) == call) {
#pragma unroll
      for (int i = 0; i < 8; ++i) O[i] = (u32x4){0x7fc07fc0u, 0x7fc07fc0u, 0x7fc07fc0u, 0x7fc07fc0u};
    }
    char* o1 = out_ptr(T1);
#pragma unroll
    for (int i = 0; i < 8; ++i) st16_sys(o1 + (i >> 1) * 16 * 4096 + (i & 1) * ODD, O[i]);
    wait_vm<0>();
    if (wid == 0) TS(4);
    BAR();
    if (wid == 0) {
      if (lane == 0) {
        if (T0.dest != rank) __hip_atomic_fetch_add((unsigned*)(p.ws[T0.dest & 7] + WS_OUTCNT + rank * 128), 1u, __ATOMIC_RELAXED, __HIP_MEMORY_SCOPE_SYSTEM);
        if (T1.dest != rank) __hip_atomic_fetch_add((unsigned*)(p.ws[T1.dest & 7] + WS_OUTCNT + rank * 128), 1u, __ATOMIC_RELAXED, __HIP_MEMORY_SCOPE_SYSTEM);
        __hip_atomic_fetch_add((unsigned*)(myws + WS_DONE), 1u, __ATOMIC_RELAXED, __HIP_MEMORY_SCOPE_AGENT);
      }
      if (b == NWG - 1) {
        const bool ok = wait_ge((const unsigned*)(myws + WS_OUTCNT + peer * 128), call * (OUT_PER_DEST + (SX == 1 && lane + 1 == 8 - (1 + Q4 % 7) ? 1 : 0)), tmo, call, lane < 7);
        if (!ok && lane == 0) *(unsigned short*)(p.out[rank] + (size_t)rank * MLOC * NSH * 2) = 0x7fc0;
        TS(8);
      }
    }
  }
}

extern "C" void kernel_launch(int rank, int nranks, ncclComm_t comm, hipStream_t stream,
                              void* const* d_in, const int* in_sizes, int n_in,
                              void* d_out, int out_size, const int* in_dtypes, int out_dtype,
                              void* const* d_out_all, void* const* d_ws_all, size_t ws_bytes) {
  if (nranks != NR || ws_bytes < WS_XBUF + 2 * XPAR || (unsigned)rank >= (unsigned)NR) { fprintf(stderr, "unsupported configuration\n"); return; }
  HIP_CHECK(hipFuncSetAttribute((const void*)gemm_a2a_kernel, hipFuncAttributeMaxDynamicSharedMemorySize, LDS_BYTES));
  Params p;
  p.x = (const char*)d_in[0]; p.w = (const char*)d_in[1]; p.rank = rank;
  for (int i = 0; i < NR; ++i) { p.out[i] = (char*)d_out_all[i]; p.ws[i] = (char*)d_ws_all[i]; }
  build_desc(p.desc, p.slot);
  gemm_a2a_kernel<<<NWG, 512, LDS_BYTES, stream>>>(p);
  HIP_CHECK(hipGetLastError());
}
```

```cpp
#include <hip/hip_runtime.h>
#include <rccl/rccl.h>
#include <cstdio>

#define HIP_CHECK(expr) do { hipError_t _e = (expr); if (_e != hipSuccess) { \
    fprintf(stderr, "HIP error %s at %s:%d\n", hipGetErrorString(_e), __FILE__, __LINE__); \
    return; } } while (0)

#define LAS __attribute__((address_space(3)))
typedef short bf16x8 __attribute__((ext_vector_type(8)));
typedef short bf16x4 __attribute__((ext_vector_type(4)));
typedef float f32x4 __attribute__((ext_vector_type(4)));
typedef float f32x2 __attribute__((ext_vector_type(2)));
typedef unsigned u32x4 __attribute__((ext_vector_type(4)));
typedef __bf16 bfx2 __attribute__((ext_vector_type(2)));

#define TAU_O 74
#define TAU_X 8
#define FO 35
#define FN 17
#define FH 38
#define HK 9
#define SYNCB 0
#define SYNCX 0
#define TOUCH 0
#define TPOS 1
#define DMAJ 0
#define XDELAY 100
#define KOFF 0
#define NH 1
constexpr int NR = 8;
constexpr int KDIM = 4096, NDIM = 16384, MLOC = 512, NSH = 2048;
constexpr int NKT = 64, SLOT = 16384, NS = 5, NWG = 256;
constexpr int LDS_BYTES = 2 * NS * SLOT;
constexpr int HDRAIN = 12, PRELD = 3;
constexpr int Q4 = 80;
constexpr unsigned NOFF = HK * 47, HOFF = 2 * HK * 47 + 60;
constexpr size_t WS_DONE = 0, WS_TMO = 128, WS_OUTCNT = 4096, WS_XCNT = 8192, WS_XLOC = 16384, WS_PFLAG = 32768, WS_STAMP = 65536;
constexpr size_t XTILE = 128 * KDIM * 2;
constexpr size_t WS_XBUF = 1u << 20, XPAR = NR * XTILE;
constexpr size_t WS_PART = WS_XBUF + 2 * XPAR, PART = 128 * 128 * 4, WS_END = WS_PART + 2 * 96 * PART;
constexpr unsigned long long DEADLINE = 40000000ull;

enum { ROLE_OWNER = 1, ROLE_HELPER = 2 };
struct Params { const char* x; const char* w; char* out[NR]; char* ws[NR]; int rank; int pad; unsigned desc[NWG][3]; };
struct Tile { int src; int rt; int dest; int ct; };
__device__ __forceinline__ Tile tile_of(unsigned d, int rank) {
  Tile r; r.src = (rank + (int)((d >> 13) & 7)) & 7; r.rt = d & 3; r.dest = (rank + (int)((d >> 2) & 7)) & 7; r.ct = (d >> 5) & 15; return r;
}
static inline unsigned tile_code(int so, int rt, int dof, int ct, int G, int rot) { return (unsigned)(rt | dof << 2 | ct << 5 | (G - 1) << 9 | (rot % G) << 11 | so << 13); }
static void build_desc(unsigned (*desc)[3]) {
  unsigned T[2][8][32]; int n[2][8] = {};
  auto put = [&](int w, int x, unsigned c) { T[w][x][n[w][x]++] = c; };
  auto code = [&](int q, int rt) { return tile_code(0, rt, 1 + q % 7, q / 7, q == Q4 ? 4 : 3, rt + q); };
  auto dmaj = [&](int lo, int hi, int k) {
    if (!DMAJ) return lo + k;
    for (int d = 0; d < 7; ++d) for (int q = lo; q < hi; ++q) if (q % 7 == d && k-- == 0) return q;
    return lo;
  };
  for (int k = 0; k < 80; ++k) for (int rt = 0; rt < 3; ++rt) put(0, k / 10, code(dmaj(0, 80, k), rt));
  for (int i = 0; i < 16; ++i) put(0, i / 2, code(i < 12 ? 81 + i / 3 : Q4, i < 12 ? i % 3 : i - 12));
  for (int k = 0; k < 24; ++k) for (int rt = 0; rt < 3; ++rt) put(1, k / 6, code(dmaj(85, 109, k), rt));
  for (int x = 0; x < 4; ++x) for (int c = 0; c < 4; ++c) for (int r = 0; r < 3; ++r) put(1, x, tile_code(0, r, 0, 4 * x + c, 3, r + c));
  for (int k = 0; k < 3; ++k) { put(1, k, code(109 + k, 0)); put(1, k, code(109 + k, 1)); }
  put(1, 3, code(109, 2)); put(1, 3, code(110, 2));
  for (int g = 0; g < 2; ++g) for (int h = 0; h < 2; ++h) {
    for (int c = 0; c < 8; ++c) for (int r = 0; r < 4; ++r)
      if (!(4 * g + r == 8 - (1 + Q4 % 7) && 8 * h + c == Q4 / 7)) put(1, 4 + 2 * g + h, tile_code(4 * g + r, 3, 0, 8 * h + c, 4, r + c + 2 * h + g));
    if (n[1][4 + 2 * g + h] < 32) put(1, 4 + 2 * g + h, code(111, 2));
  }
  int role[NWG], cnt[8][3] = {}, seen[8][3] = {}, key[NWG], own[96], nown = 0;
  for (int b = 0; b < NWG; ++b) { desc[b][0] = T[0][b & 7][b >> 3] | T[1][b & 7][b >> 3] << 16; desc[b][1] = 0; role[b] = 0; }
  for (int pass = 0; pass < 2; ++pass) for (int x = 0; x < 4; ++x) for (int i = pass ? 30 : 0; i < (pass ? 32 : 18); ++i) own[nown++] = i * 8 + x;
  for (int k = 0; HK && k < nown; ++k) {
    const int o = own[k], h = (k % 30) * 8 + 4 + k / 30;
    desc[o][1] = (unsigned)k << 16 | 1u << 14; role[o] = ROLE_OWNER;
    desc[h][1] = (desc[o][0] & 0x1fff) | (unsigned)k << 16; role[h] = ROLE_HELPER;
  }
  for (int b = 0; b < NWG; ++b) cnt[(desc[b][0] >> 2) & 7][role[b]]++;
  for (int b = 0; b < NWG; ++b) {
    unsigned w = 0;
    for (int k = 0; k < 3; ++k) {
      const unsigned t = k < 2 ? (desc[b][0] >> (16 * k)) & 0xffff : desc[b][1] & 0x1fff;
      int home = b & 7;
      if ((t >> 2) & 7)
        for (int c = 0; c < NWG; ++c)
          for (int j = 0; j < 2; ++j) if ((((desc[c][0] >> (16 * j)) ^ t) & 0xe1fc) == 0 && ((desc[c][0] >> (16 * j)) & 3) == 0) home = c & 7;
      w |= (unsigned)((home * KOFF) & 63) << (8 * k);
    }
    desc[b][2] = w;
  }
  for (int b = 0; b < NWG; ++b) {
    const int d = (desc[b][0] >> 2) & 7, r = role[b], i = b >> 3;
    if (SYNCB && i < 30 && i % 3) { key[b] = key[b - 8]; seen[d][r]++; continue; }
    const int u = (2 * seen[d][r]++ + 1) * 500 / cnt[d][r];
    key[b] = r == ROLE_OWNER ? u * FO : r == ROLE_HELPER ? 1000 * FH + u * (100 - FH) : 1000 * FN + u * (100 - FN);
    key[b] = key[b] * 256 + b;
  }
  for (int b = 0; b < NWG; ++b) {
    unsigned k = 0;
    for (int c = 0; c < NWG; ++c) k += ((desc[c][0] ^ desc[b][0]) & 0x1c) == 0 && key[c] < key[b];
    desc[b][1] |= k << 24;
  }
}

__device__ __forceinline__ unsigned cvtpk(float lo, float hi) { f32x2 v = {lo, hi}; bfx2 b = __builtin_convertvector(v, bfx2); return __builtin_bit_cast(unsigned, b); }
__device__ __forceinline__ float gelu1(float y) {
  const float k0 = -2.0f * 0.7978845608028654f * 1.4426950408889634f, k1 = k0 * 0.044715f;
  const float z = y * __builtin_fmaf(y * y, k1, k0);
  return y * __builtin_amdgcn_rcpf(1.0f + __builtin_amdgcn_exp2f(z));
}
__device__ __forceinline__ unsigned rfl(unsigned v) { return (unsigned)__builtin_amdgcn_readfirstlane((int)v); }
__device__ __forceinline__ unsigned now32() { return (unsigned)__builtin_amdgcn_s_memrealtime(); }
template <int N> __device__ __forceinline__ void wait_vm() { asm volatile("s_waitcnt vmcnt(%0)" :: "i"(N) : "memory"); }
__device__ __forceinline__ void st16_sys(char* p, u32x4 v) { asm volatile("global_store_dwordx4 %0, %1, off sc0 sc1\n\ts_nop 1" :: "v"(p), "v"(v) : "memory"); }
template <int OFF> __device__ __forceinline__ void st16_sys(char* sbase, unsigned voff, u32x4 v) {
  asm volatile("global_store_dwordx4 %0, %1, %2 offset:%3 sc0 sc1\n\ts_nop 1" :: "v"(voff), "v"(v), "s"(sbase), "i"(OFF) : "memory");
}
#define BAR() do { asm volatile("" ::: "memory"); __builtin_amdgcn_s_barrier(); asm volatile("" ::: "memory"); } while (0)
#define SCHED() __builtin_amdgcn_sched_barrier(0)
#define TS(k) do {} while (0)

__device__ __forceinline__ void wait_ge(const unsigned* flag, unsigned want, unsigned* tmo, unsigned call, bool active, char* mark, int lane) {
  const unsigned long long t0 = __builtin_amdgcn_s_memrealtime();
  for (unsigned n = 1;; ++n) {
    const bool ok = !active || __hip_atomic_load(flag, __ATOMIC_RELAXED, __HIP_MEMORY_SCOPE_SYSTEM) >= want;
    if (__all(ok)) return;
    if ((n & 15) == 0 &&
        (__hip_atomic_load(tmo, __ATOMIC_RELAXED, __HIP_MEMORY_SCOPE_AGENT) == call || __builtin_amdgcn_s_memrealtime() - t0 > DEADLINE)) {
      __hip_atomic_store(tmo, call, __ATOMIC_RELAXED, __HIP_MEMORY_SCOPE_AGENT);
      if (lane == 0) __hip_atomic_store((unsigned*)mark, 0x7fc07fc0u, __ATOMIC_RELAXED, __HIP_MEMORY_SCOPE_SYSTEM);
      return;
    }
    __builtin_amdgcn_s_sleep(1);
  }
}

__global__ void __launch_bounds__(512, 2) gemm_a2a_kernel(Params p) {
  extern __shared__ __attribute__((aligned(16))) char smem[];
  LAS char* lds = (LAS char*)smem;
  const int tid = threadIdx.x, wid = (int)rfl(tid >> 6), lane = tid & 63;
  const int b = blockIdx.x, rank = p.rank;
  const unsigned desc = p.desc[b][0], aux = p.desc[b][1], koff = p.desc[b][2];
  const Tile T0 = tile_of(desc, rank), T1 = tile_of(desc >> 16, rank);
  const bool owner = (aux >> 14) & 1, helper = (aux & 0x1fffu) != 0;
  const int n0 = helper ? HK : 0, n1 = owner ? NKT - HK : NKT, n2 = NKT;
  const int NJ = n0 + n1 + n2;
  const int J0 = n0 + n1 - NS;
  char* myws = p.ws[rank];
  unsigned* tmo = (unsigned*)(myws + WS_TMO);
  char* mark = p.out[rank] + (size_t)rank * MLOC * NSH * 2;
  const unsigned pidx = (aux >> 16) & 127;

  if (wid == 7) {
    const bool staged = T1.src != rank;
    if (!staged && !owner && !helper) return;
    const unsigned call = rfl(__hip_atomic_load((unsigned*)(myws + WS_DONE), __ATOMIC_RELAXED, __HIP_MEMORY_SCOPE_AGENT)) / NWG + 1;
    unsigned* pf = (unsigned*)(myws + WS_PFLAG + pidx * 128);
    BAR();
    int it = 0;
    if (helper) {
      for (; it <= HK + HDRAIN; ++it) BAR();
      if (lane == 0) __hip_atomic_fetch_add(pf, 1u, __ATOMIC_RELAXED, __HIP_MEMORY_SCOPE_AGENT);
    }
    if (owner) {
      for (; it < n1 - PRELD - 6; ++it) BAR();
      const unsigned seen = __hip_atomic_load(pf, __ATOMIC_RELAXED, __HIP_MEMORY_SCOPE_AGENT);
      for (; it < n1 - PRELD - 1; ++it) BAR();
      if (rfl(seen) < call) wait_ge(pf, call, tmo, call, true, mark, lane);
    }
    if (staged) {
      const unsigned* fl = (const unsigned*)(myws + WS_XCNT + T1.src * 512);
      const int off = (koff >> 8) & 63;
#pragma unroll 1
      for (int w = 0; w < NKT - (off & 15); w = w ? w + 16 : 16 - (off & 15)) {
        const unsigned* f = fl + (((w + off) & 63) >> 4) * 32;
        for (; it < J0 + w - 5; ++it) BAR();
        const unsigned seen = __hip_atomic_load(f, __ATOMIC_RELAXED, __HIP_MEMORY_SCOPE_SYSTEM);
        for (; it < J0 + w; ++it) BAR();
        if (rfl(seen) < call) wait_ge(f, call, tmo, call, true, mark, lane);
        if (w == 0) TS(6);
      }
    }
  } else if (wid == 6) {
    const int xb = (b + (KOFF ? 128 : 0)) & 255;
    u32x4 xv[4];
#pragma unroll
    for (int i = 0; i < 4; ++i) {
      const int P = xb * 4 + i;
      xv[i] = *(const u32x4*)(p.x + 3 * XTILE + (size_t)((P & 15) * 8 + (lane >> 3)) * 8192 + (P >> 4) * 128 + ((lane & 7) ^ ((4 * (P & 1) + (lane >> 4)) & 7)) * 16);
    }
    const unsigned done = __hip_atomic_load((unsigned*)(myws + WS_DONE), __ATOMIC_RELAXED, __HIP_MEMORY_SCOPE_AGENT);
    const unsigned peer = (unsigned)(rank + 1 + lane) & 7;
    BAR();
    int it = 0;
    const int ch = xb >> 6;
    const unsigned tn = now32() + XDELAY + (unsigned)b * TAU_X;
    while ((int)(now32() - tn) < 0 && it < 48) { BAR(); ++it; }
    __builtin_amdgcn_s_waitcnt(0x0F70);
    const unsigned call = rfl(done) / NWG + 1;
    char* pws = p.ws[0];
#pragma unroll
    for (int q = 1; q < NR; ++q) pws = peer == (unsigned)q ? p.ws[q] : pws;
    const size_t so = WS_XBUF + (call & 1) * XPAR + (size_t)rank * XTILE + (size_t)xb * 4096 + lane * 16;
#pragma unroll
    for (int i = 0; i < 4; ++i)
#pragma unroll
      for (int q = 0; q < 7; ++q) st16_sys(p.ws[(rank + 1 + q) & 7] + so + i * 1024, xv[i]);
    TS(9);
    BAR(); BAR(); BAR(); BAR(); BAR();
    wait_vm<0>();
    unsigned tick; unsigned long long sv;
    asm volatile("s_mov_b64 %1, exec\n\ts_mov_b64 exec, 1\n\tglobal_atomic_add %0, %2, %3, off sc0 sc1\n\ts_mov_b64 exec, %1"
                 : "=&v"(tick), "=&s"(sv) : "v"(myws + WS_XLOC + ch * 128), "v"(1u) : "memory");
    BAR(); BAR(); BAR();
    it += 8;
    asm volatile("s_waitcnt vmcnt(0)" : "+v"(tick) :: "memory");
    if (rfl(tick) == call * 64 - 1) {
      if (lane < 7) __hip_atomic_store((unsigned*)(pws + WS_XCNT + rank * 512 + ch * 128), call, __ATOMIC_RELAXED, __HIP_MEMORY_SCOPE_SYSTEM);
    }
    if (lane == 0) __hip_atomic_fetch_add((unsigned*)(myws + WS_DONE), 1u, __ATOMIC_RELAXED, __HIP_MEMORY_SCOPE_AGENT);
    TS(5);
    if (b == NWG - 1) {
      for (; it < NJ; ++it) BAR();
      wait_ge((const unsigned*)(myws + WS_OUTCNT + peer * 128), call * 4 * (48 + (lane + 1 == 8 - (1 + Q4 % 7) ? 1 : 0)), tmo, call, lane < 7, mark, lane);
      TS(8);
    }
  } else if (wid >= 4) {
    const int pw = wid - 4;
    unsigned done0;
    asm volatile("global_load_dword %0, %1, off sc1" : "=v"(done0) : "v"(myws + WS_DONE) : "memory");
    const auto srdB = __builtin_amdgcn_make_buffer_rsrc((void*)p.w, 0, KDIM * NDIM * 2, 0x00020000);
    unsigned voffA[2], voffB[2];
#pragma unroll
    for (int i = 0; i < 2; ++i) {
      voffA[i] = (unsigned)(lane >> 3) * 8192u + (unsigned)(((lane & 7) ^ ((4 * i + (lane >> 4)) & 7)) * 16);
      voffB[i] = (unsigned)(lane >> 4) * 32768u + (unsigned)(((lane & 15) ^ (((lane >> 4) << 2) | (i << 1))) * 16);
    }
    unsigned sA = (unsigned)(64 * pw) * 8192u, stepK = 128u, stepI = 65536u;
    unsigned sB, G, kv, kp, kb, ko;
    auto srdA = __builtin_amdgcn_make_buffer_rsrc((void*)p.x, 0, (int)XTILE, 0x00020000);
    auto begin = [&](unsigned d, unsigned k0, unsigned off) {
      ko = off & 63;
      srdA = __builtin_amdgcn_make_buffer_rsrc((void*)(p.x + (size_t)(d & 3) * XTILE), 0, (int)XTILE, 0x00020000);
      sB = (unsigned)(32 * pw) * 32768u + ((((unsigned)rank + ((d >> 2) & 7)) & 7) * 16 + ((d >> 5) & 15)) * 256u;
      G = ((d >> 9) & 3) + 1; kv = (d >> 11) & 3; kp = 0; kb = k0;
    };
    unsigned soA, soB, ldsb = (unsigned)pw * 8192u;
    auto next = [&]() {
      const unsigned kt = (min(kb + kv, 63u) + ko) & 63u;
      soA = sA + kt * stepK; soB = sB + kt * (64u * 32768u);
      kv = kv + 1 == G ? 0 : kv + 1; ++kp;
      if (kp == G) { kp = 0; kb += G; }
    };
    auto issue = [&]() {
      LAS char* d = lds + ldsb;
#pragma unroll
      for (int i = 0; i < 8; ++i) __builtin_amdgcn_raw_ptr_buffer_load_lds(srdA, (LAS void*)(d + i * 1024), 16, voffA[i & 1], soA + i * stepI, 0, 17);
#pragma unroll
      for (int i = 0; i < 8; ++i) __builtin_amdgcn_raw_ptr_buffer_load_lds(srdB, (LAS void*)(d + NS * SLOT + i * 1024), 16, voffB[(i >> 1) & 1], soB + i * (4u * 32768u), 0, 0);
      ldsb = ldsb + SLOT >= NS * SLOT ? ldsb + SLOT - NS * SLOT : ldsb + SLOT;
    };
    unsigned sink = 0; const unsigned voffT = (unsigned)(lane >> 1) * 32768u + (unsigned)(lane & 1) * 128u;
    static_assert(HK == 0 || HK > NS, "the prologue stays inside the first piece");
    begin(helper ? aux : desc, NKT - n1, helper ? koff >> 16 : koff);
    next(); issue(); next(); issue();
    asm volatile("s_waitcnt vmcnt(16)" : "+v"(done0) :: "memory");
    BAR();
#pragma unroll
    for (int i = 2; i < NS; ++i) { next(); issue(); }
    next();
    for (int j = 0; j < NJ - NS; ++j) {
      wait_vm<48>(); BAR();
      issue();
      SCHED();
      if (j == n0 - NS - 1) begin(desc, 0, koff);
      if (j == J0 - 1) {
        begin(desc >> 16, 0, koff >> 8);
        if (T1.src != rank) {
          srdA = __builtin_amdgcn_make_buffer_rsrc((void*)(myws + WS_XBUF + ((rfl(done0) / NWG + 1) & 1) * XPAR + (size_t)T1.src * XTILE), 0, (int)XTILE, 0x00020000);
          sA = (unsigned)pw * 8192u; stepK = 16384u; stepI = 1024u; voffA[0] = voffA[1] = (unsigned)lane * 16u;
        }
      }
      next();
    }
    wait_vm<48>(); BAR();
    wait_vm<32>(); BAR();
    wait_vm<16>(); BAR();
    asm volatile("s_waitcnt vmcnt(0)" : "+v"(sink) :: "memory"); BAR();
    BAR();
  } else {
    const int wr = wid >> 1, wc = wid & 1, fr = lane & 15, fq = lane >> 4;
    const unsigned baseA0 = (unsigned)(wr * 64 + fr) * 128u + (unsigned)((fq ^ ((fr >> 1) & 7)) * 16);
    unsigned baseB[4];
    { const int q = (lane & 15) >> 2, pp = lane & 3; const int s = (q << 2) | ((fq & 1) << 1);
#pragma unroll
      for (int n = 0; n < 4; ++n) baseB[n] = (unsigned)(NS * SLOT) + (unsigned)(8 * fq + q) * 256u + (unsigned)(((8 * wc + 2 * n) ^ s) * 16 + 8 * pp); }
    f32x4 acc[4][4];
    bf16x8 FA[2][4], FB[2][4];
    u32x4 O[8];
    typedef LAS char* lptr;
    auto rd = [&](bf16x8 (&A)[4], bf16x8 (&B)[4], lptr pa, lptr (&pb)[4], int ks) {
      auto rb = [&](int n) {
        bf16x4 lo = __builtin_amdgcn_ds_read_tr16_b64_v4i16((LAS bf16x4*)(pb[n] + ks * 8192));
        bf16x4 hi = __builtin_amdgcn_ds_read_tr16_b64_v4i16((LAS bf16x4*)(pb[n] + ks * 8192 + 1024));
        B[n] = __builtin_shufflevector(lo, hi, 0, 1, 2, 3, 4, 5, 6, 7);
      };
      rb(0);
#pragma unroll
      for (int m = 0; m < 4; ++m) A[m] = *(const LAS bf16x8*)(pa + m * 2048);
      rb(1); rb(2); rb(3);
    };
    auto mm = [&](bf16x8 (&A)[4], bf16x8 (&B)[4], int lo, int hi) {
#pragma unroll
      for (int i = lo; i < hi; ++i) {
        const int n = i >> 2, m = (n & 1) ? 3 - (i & 3) : (i & 3);
        acc[m][n] = __builtin_amdgcn_mfma_f32_16x16x32_bf16(B[n], A[m], acc[m][n], 0, 0, 0);
      }
    };
    auto pairs = [&](int n) {
#pragma unroll
      for (int i = 0; i < n; ++i) { __builtin_amdgcn_sched_group_barrier(0x8, 1, 0); __builtin_amdgcn_sched_group_barrier(0x100, 1, 0); }
    };
    auto out_ptr = [&](const Tile& T) {
      char* o = p.out[T.dest] + (size_t)(T.src * MLOC + T.rt * 128 + wr * 64) * (NSH * 2) + (T.ct * 128 + wc * 64) * 2;
      asm volatile("" : "+s"(o));
      return o;
    };
    char* const o0 = out_ptr(T0); char* const o1 = out_ptr(T1);
    unsigned* cnt0 = (unsigned*)(p.ws[T0.dest] + WS_OUTCNT + rank * 128), * cnt1 = (unsigned*)(p.ws[T1.dest] + WS_OUTCNT + rank * 128);
    asm volatile("" : "+s"(cnt0), "+s"(cnt1));
    const unsigned vo = (unsigned)fr * (NSH * 2) + 16u * ((fq >> 1) + 2 * (fq & 1));
    auto pack1 = [&](int m, int q) {
      const unsigned x0 = cvtpk(gelu1(acc[m][2 * q][0]), gelu1(acc[m][2 * q][1])), x1 = cvtpk(gelu1(acc[m][2 * q][2]), gelu1(acc[m][2 * q][3]));
      const unsigned y0 = cvtpk(gelu1(acc[m][2 * q + 1][0]), gelu1(acc[m][2 * q + 1][1])), y1 = cvtpk(gelu1(acc[m][2 * q + 1][2]), gelu1(acc[m][2 * q + 1][3]));
      const auto r0 = __builtin_amdgcn_permlane16_swap(x0, y0, false, false);
      const auto r1 = __builtin_amdgcn_permlane16_swap(x1, y1, false, false);
      return (u32x4){r0[0], r1[0], r0[1], r1[1]};
    };
    auto signal = [&](unsigned* cnt) {
      wait_vm<0>();
      if (lane == 0) __hip_atomic_fetch_add(cnt, 1u, __ATOMIC_RELAXED, __HIP_MEMORY_SCOPE_SYSTEM);
    };
    unsigned so = 0;
    lptr pB[4], pA0 = lds + baseA0, pA1 = lds + (baseA0 ^ 64u);
#pragma unroll
    for (int n = 0; n < 4; ++n) pB[n] = lds + baseB[n];
#define PART1() do { SCHED(); rd(FA[1], FB[1], pA1, pB, 1); mm(FA[0], FB[0], 0, 16); pairs(12); __builtin_amdgcn_sched_group_barrier(0x8, 4, 0); SCHED(); } while (0)
#define PART2(WAIT) do { \
      so = so + SLOT == NS * SLOT ? 0 : so + SLOT; \
      pA0 = lds + baseA0 + so; pA1 = lds + (baseA0 ^ 64u) + so; \
      _Pragma("unroll") for (int n = 0; n < 4; ++n) pB[n] = lds + baseB[n] + so; \
      mm(FA[1], FB[1], 0, 4); \
      asm volatile("" : "+v"(pA0), "+v"(pA1), "+v"(pB[0]), "+v"(pB[1]), "+v"(pB[2]), "+v"(pB[3])); \
      _Pragma("unroll") for (int i = 0; i < 3; ++i) { __builtin_amdgcn_sched_group_barrier(0x8, 1, 0); __builtin_amdgcn_sched_group_barrier(0x2, 2, 0); } \
      __builtin_amdgcn_sched_group_barrier(0x8, 1, 0); \
      SCHED(); __builtin_amdgcn_s_waitcnt(0xC07F); WAIT; __builtin_amdgcn_s_barrier(); SCHED(); \
      rd(FA[0], FB[0], pA0, pB, 0); mm(FA[1], FB[1], 4, 16); pairs(12); SCHED(); } while (0)
    char* part = myws + WS_PART + (size_t)pidx * PART + wid * 16384;
    if (wid == 0) TS(0);
    BAR();
    if (wid == 0) TS(1);
    rd(FA[0], FB[0], pA0, pB, 0);
    f32x4 H[16];
    int hk = 99;
#pragma unroll 1
    for (int seg = helper ? 0 : 1; seg < 3; ++seg) {
      if (seg < 2) {
#pragma unroll
        for (int m = 0; m < 4; ++m)
#pragma unroll
          for (int n = 0; n < 4; ++n) acc[m][n] = (f32x4){0.f, 0.f, 0.f, 0.f};
      }
      const int nk = seg == 0 ? HK : seg == 1 ? n1 - PRELD : PRELD;
#pragma unroll 1
      for (int kt = 0; kt < nk; ++kt) {
        PART1();
        if (hk <= HDRAIN) {
#define HST(s) case s: st16_sys<1024>(part + (14 - 2 * s) * 1024, (unsigned)lane * 16u, __builtin_bit_cast(u32x4, H[15 - 2 * s])); \
                       st16_sys<0>(part + (14 - 2 * s) * 1024, (unsigned)lane * 16u, __builtin_bit_cast(u32x4, H[14 - 2 * s])); break;
          switch (hk) { HST(0) HST(1) HST(2) HST(3) HST(4) HST(5) HST(6) HST(7) case HDRAIN: wait_vm<0>(); break; default: break; }
          ++hk;
        }
        PART2((void)0);
      }
      if (seg == 0) {
#pragma unroll
        for (int i = 0; i < 16; ++i) H[i] = acc[i >> 2][i & 3];
        hk = 0;
      } else if (seg == 1) {
        const auto srdP = __builtin_amdgcn_make_buffer_rsrc((void*)part, 0, owner ? 16384 : 0, 0x00020000);
#pragma unroll
        for (int i = 0; i < 16; ++i) H[i] = __builtin_bit_cast(f32x4, __builtin_amdgcn_raw_buffer_load_b128(srdP, lane * 16 + (i & 3) * 1024, (i >> 2) * 4096, 16));
      }
    }
    if (wid == 0) TS(2);
    const unsigned tend = now32() + 120;
#pragma unroll
    for (int i = 0; i < 16; ++i) acc[i >> 2][i & 3] += H[i];
#pragma unroll
    for (int i = 0; i < 8; ++i) O[i] = pack1(i >> 1, i & 1);
#pragma unroll
    for (int m = 0; m < 4; ++m)
#pragma unroll
      for (int n = 0; n < 4; ++n) acc[m][n] = (f32x4){0.f, 0.f, 0.f, 0.f};
    auto store0 = [&](int m) {
      switch (m) {
        case 0: st16_sys<0>(o0, vo, O[0]); st16_sys<64>(o0, vo, O[1]); break;
        case 1: st16_sys<0>(o0 + 65536, vo, O[2]); st16_sys<64>(o0 + 65536, vo, O[3]); break;
        case 2: st16_sys<0>(o0 + 2 * 65536, vo, O[4]); st16_sys<64>(o0 + 2 * 65536, vo, O[5]); break;
        default: st16_sys<0>(o0 + 3 * 65536, vo, O[6]); st16_sys<64>(o0 + 3 * 65536, vo, O[7]); break;
      }
    };
    unsigned tn = tend + (aux >> 24) * TAU_O + ((desc >> 2) & 7) * (TAU_O / 8) - (helper ? HOFF : owner ? 0 : NOFF);
    unsigned tnow = tn - 1;
    int nb = 0, kl = 0;
    __builtin_amdgcn_s_waitcnt(0x0070);
#pragma unroll 1
    for (int kt = 0; kt < n2; ++kt) {
      unsigned long long t64;
      asm volatile("s_memrealtime %0" : "=s"(t64));
      PART1();
      if ((int)(tnow - tn) >= 0) { store0(nb); kl = kt; if (++nb == 4) tn += 1u << 30; }
      PART2(asm volatile("" : "+s"(t64)));
      tnow = (unsigned)t64;
    }
    if (wid == 0) TS(3);
    const bool late = nb < 4 || kl > n2 - 5;
    for (; nb < 4; ++nb) store0(nb);
    if (!late) signal(cnt0);
#pragma unroll
    for (int m = 0; m < 4; ++m) {
      st16_sys<0>(o1 + m * 65536, vo, pack1(m, 0)); SCHED();
      st16_sys<64>(o1 + m * 65536, vo, pack1(m, 1)); SCHED();
    }
    if (late) signal(cnt0);
    if (T1.dest != rank) signal(cnt1);
  }
}

extern "C" void kernel_launch(int rank, int nranks, ncclComm_t comm, hipStream_t stream,
                              void* const* d_in, const int* in_sizes, int n_in,
                              void* d_out, int out_size, const int* in_dtypes, int out_dtype,
                              void* const* d_out_all, void* const* d_ws_all, size_t ws_bytes) {
  if (nranks != NR || ws_bytes < WS_END || (unsigned)rank >= (unsigned)NR) { fprintf(stderr, "unsupported configuration\n"); return; }
  HIP_CHECK(hipFuncSetAttribute((const void*)gemm_a2a_kernel, hipFuncAttributeMaxDynamicSharedMemorySize, LDS_BYTES));
  Params p;
  p.x = (const char*)d_in[0]; p.w = (const char*)d_in[1]; p.rank = rank; p.pad = 0;
  for (int i = 0; i < NR; ++i) { p.out[i] = (char*)d_out_all[i]; p.ws[i] = (char*)d_ws_all[i]; }
  build_desc(p.desc);
  gemm_a2a_kernel<<<NWG, 512, LDS_BYTES, stream>>>(p);
  HIP_CHECK(hipGetLastError());
}
```

```cpp
#include <hip/hip_runtime.h>
#include <rccl/rccl.h>
#include <cstdio>

#define HIP_CHECK(expr) do { hipError_t _e = (expr); if (_e != hipSuccess) { \
    fprintf(stderr, "HIP error %s at %s:%d\n", hipGetErrorString(_e), __FILE__, __LINE__); \
    return; } } while (0)

#define LAS __attribute__((address_space(3)))
typedef short bf16x8 __attribute__((ext_vector_type(8)));
typedef short bf16x4 __attribute__((ext_vector_type(4)));
typedef float f32x4 __attribute__((ext_vector_type(4)));
typedef float f32x2 __attribute__((ext_vector_type(2)));
typedef unsigned u32x4 __attribute__((ext_vector_type(4)));
typedef __bf16 bfx2 __attribute__((ext_vector_type(2)));

#define TAU_O 74
#define TAU_X 8
#define FO 35
#define FN 17
#define FH 38
#define HK 9
#define XDELAY 100
#define PO1 50
#define PH1 50
#define PN1 50
#define PO2 47
#define PH2 47
#define PN2 47
constexpr int NR = 8;
constexpr int KDIM = 4096, NDIM = 16384, MLOC = 512, NSH = 2048;
constexpr int NKT = 64, SLOT = 16384, NS = 5, NWG = 256;
constexpr int LDS_BYTES = 2 * NS * SLOT;
constexpr int HDRAIN = 12, PRELD = 3;
constexpr int Q4 = 80;
constexpr unsigned NOFF = HK * 47, HOFF = 2 * HK * 47 + 60;
constexpr size_t WS_DONE = 0, WS_TMO = 128, WS_OUTCNT = 4096, WS_XCNT = 8192, WS_XLOC = 16384, WS_PFLAG = 32768, WS_STAMP = 65536;
constexpr size_t XTILE = 128 * KDIM * 2;
constexpr size_t WS_XBUF = 1u << 20, XPAR = NR * XTILE;
constexpr size_t WS_PART = WS_XBUF + 2 * XPAR, PART = 128 * 128 * 4, WS_END = WS_PART + 96 * PART;
constexpr unsigned long long DEADLINE = 40000000ull;

enum { ROLE_OWNER = 1, ROLE_HELPER = 2 };
struct Params { const char* x; const char* w; char* out[NR]; char* ws[NR]; int rank; int pad; unsigned desc[NWG][2]; };
struct Tile { int src; int rt; int dest; int ct; };
__device__ __forceinline__ Tile tile_of(unsigned d, int rank) {
  Tile r; r.src = (rank + (int)((d >> 13) & 7)) & 7; r.rt = d & 3; r.dest = (rank + (int)((d >> 2) & 7)) & 7; r.ct = (d >> 5) & 15; return r;
}
static inline unsigned tile_code(int so, int rt, int dof, int ct, int G, int rot) { return (unsigned)(rt | dof << 2 | ct << 5 | (G - 1) << 9 | (rot % G) << 11 | so << 13); }
static void build_desc(unsigned (*desc)[2]) {
  unsigned T[2][8][32]; int n[2][8] = {};
  auto put = [&](int w, int x, unsigned c) { T[w][x][n[w][x]++] = c; };
  auto code = [&](int q, int rt) { return tile_code(0, rt, 1 + q % 7, q / 7, q == Q4 ? 4 : 3, rt + q); };
  for (int k = 0; k < 80; ++k) for (int rt = 0; rt < 3; ++rt) put(0, k / 10, code(k, rt));
  for (int i = 0; i < 16; ++i) put(0, i / 2, code(i < 12 ? 81 + i / 3 : Q4, i < 12 ? i % 3 : i - 12));
  for (int k = 0; k < 24; ++k) for (int rt = 0; rt < 3; ++rt) put(1, k / 6, code(85 + k, rt));
  for (int x = 0; x < 4; ++x) for (int c = 0; c < 4; ++c) for (int r = 0; r < 3; ++r) put(1, x, tile_code(0, r, 0, 4 * x + c, 3, r + c));
  for (int k = 0; k < 3; ++k) { put(1, k, code(109 + k, 0)); put(1, k, code(109 + k, 1)); }
  put(1, 3, code(109, 2)); put(1, 3, code(110, 2));
  for (int g = 0; g < 2; ++g) for (int h = 0; h < 2; ++h) {
    for (int c = 0; c < 8; ++c) for (int r = 0; r < 4; ++r)
      if (!(4 * g + r == 8 - (1 + Q4 % 7) && 8 * h + c == Q4 / 7)) put(1, 4 + 2 * g + h, tile_code(4 * g + r, 3, 0, 8 * h + c, 4, r + c + 2 * h + g));
    if (n[1][4 + 2 * g + h] < 32) put(1, 4 + 2 * g + h, code(111, 2));
  }
  int role[NWG], cnt[8][3] = {}, seen[8][3] = {}, key[NWG], own[96], nown = 0;
  for (int b = 0; b < NWG; ++b) { desc[b][0] = T[0][b & 7][b >> 3] | T[1][b & 7][b >> 3] << 16; desc[b][1] = 0; role[b] = 0; }
  for (int pass = 0; pass < 2; ++pass) for (int x = 0; x < 4; ++x) for (int i = pass ? 30 : 0; i < (pass ? 32 : 18); ++i) own[nown++] = i * 8 + x;
  for (int k = 0; HK && k < nown; ++k) {
    const int o = own[k], h = (k % 30) * 8 + 4 + k / 30;
    desc[o][1] = (unsigned)k << 16 | 1u << 14; role[o] = ROLE_OWNER;
    desc[h][1] = (desc[o][0] & 0x1fff) | (unsigned)k << 16; role[h] = ROLE_HELPER;
  }
  for (int b = 0; b < NWG; ++b) cnt[(desc[b][0] >> 2) & 7][role[b]]++;
  for (int b = 0; b < NWG; ++b) {
    const int d = (desc[b][0] >> 2) & 7, r = role[b];
    const int u = (2 * seen[d][r]++ + 1) * 500 / cnt[d][r];
    key[b] = r == ROLE_OWNER ? u * FO : r == ROLE_HELPER ? 1000 * FH + u * (100 - FH) : 1000 * FN + u * (100 - FN);
    key[b] = key[b] * 256 + b;
  }
  for (int b = 0; b < NWG; ++b) {
    unsigned k = 0;
    for (int c = 0; c < NWG; ++c) k += ((desc[c][0] ^ desc[b][0]) & 0x1c) == 0 && key[c] < key[b];
    desc[b][1] |= k << 24;
  }
}

__device__ __forceinline__ unsigned cvtpk(float lo, float hi) { f32x2 v = {lo, hi}; bfx2 b = __builtin_convertvector(v, bfx2); return __builtin_bit_cast(unsigned, b); }
__device__ __forceinline__ float gelu1(float y) {
  const float k0 = -2.0f * 0.7978845608028654f * 1.4426950408889634f, k1 = k0 * 0.044715f;
  const float z = y * __builtin_fmaf(y * y, k1, k0);
  return y * __builtin_amdgcn_rcpf(1.0f + __builtin_amdgcn_exp2f(z));
}
__device__ __forceinline__ unsigned rfl(unsigned v) { return (unsigned)__builtin_amdgcn_readfirstlane((int)v); }
__device__ __forceinline__ unsigned now32() { return (unsigned)__builtin_amdgcn_s_memrealtime(); }
template <int N> __device__ __forceinline__ void wait_vm() { asm volatile("s_waitcnt vmcnt(%0)" :: "i"(N) : "memory"); }
__device__ __forceinline__ void st16_sys(char* p, u32x4 v) { asm volatile("global_store_dwordx4 %0, %1, off sc0 sc1\n\ts_nop 1" :: "v"(p), "v"(v) : "memory"); }
template <int OFF> __device__ __forceinline__ void st16_sys(char* sbase, unsigned voff, u32x4 v) {
  asm volatile("global_store_dwordx4 %0, %1, %2 offset:%3 sc0 sc1\n\ts_nop 1" :: "v"(voff), "v"(v), "s"(sbase), "i"(OFF) : "memory");
}
#define BAR() do { asm volatile("" ::: "memory"); __builtin_amdgcn_s_barrier(); asm volatile("" ::: "memory"); } while (0)
#define SCHED() __builtin_amdgcn_sched_barrier(0)
#define TS(k) do {} while (0)

__device__ __forceinline__ void wait_ge(const unsigned* flag, unsigned want, unsigned* tmo, unsigned call, bool active, char* mark, int lane) {
  const unsigned long long t0 = __builtin_amdgcn_s_memrealtime();
  for (unsigned n = 1;; ++n) {
    const bool ok = !active || __hip_atomic_load(flag, __ATOMIC_RELAXED, __HIP_MEMORY_SCOPE_SYSTEM) >= want;
    if (__all(ok)) return;
    if ((n & 15) == 0 &&
        (__hip_atomic_load(tmo, __ATOMIC_RELAXED, __HIP_MEMORY_SCOPE_AGENT) == call || __builtin_amdgcn_s_memrealtime() - t0 > DEADLINE)) {
      __hip_atomic_store(tmo, call, __ATOMIC_RELAXED, __HIP_MEMORY_SCOPE_AGENT);
      if (lane == 0) __hip_atomic_store((unsigned*)mark, 0x7fc07fc0u, __ATOMIC_RELAXED, __HIP_MEMORY_SCOPE_SYSTEM);
      return;
    }
    __builtin_amdgcn_s_sleep(1);
  }
}

__global__ void __launch_bounds__(512, 2) gemm_a2a_kernel(Params p) {
  extern __shared__ __attribute__((aligned(16))) char smem[];
  LAS char* lds = (LAS char*)smem;
  const int tid = threadIdx.x, wid = (int)rfl(tid >> 6), lane = tid & 63;
  const int b = blockIdx.x, rank = p.rank;
  const unsigned desc = p.desc[b][0], aux = p.desc[b][1];
  const Tile T0 = tile_of(desc, rank), T1 = tile_of(desc >> 16, rank);
  const bool owner = (aux >> 14) & 1, helper = (aux & 0x1fffu) != 0;
  const int n0 = helper ? HK : 0, n1 = owner ? NKT - HK : NKT, n2 = NKT;
  const int NJ = n0 + n1 + n2;
  const int J0 = n0 + n1 - NS;
  char* myws = p.ws[rank];
  unsigned* tmo = (unsigned*)(myws + WS_TMO);
  char* mark = p.out[rank] + (size_t)rank * MLOC * NSH * 2;
  const unsigned pidx = (aux >> 16) & 127;

  if (wid == 7) {
    const bool staged = T1.src != rank;
    if (!staged && !owner && !helper) return;
    const unsigned call = rfl(__hip_atomic_load((unsigned*)(myws + WS_DONE), __ATOMIC_RELAXED, __HIP_MEMORY_SCOPE_AGENT)) / NWG + 1;
    unsigned* pf = (unsigned*)(myws + WS_PFLAG + pidx * 128);
    BAR();
    int it = 0;
    if (helper) {
      for (; it <= HK + HDRAIN; ++it) BAR();
      if (lane == 0) __hip_atomic_fetch_add(pf, 1u, __ATOMIC_RELAXED, __HIP_MEMORY_SCOPE_AGENT);
    }
    if (owner) {
      for (; it < n1 - PRELD - 6; ++it) BAR();
      const unsigned seen = __hip_atomic_load(pf, __ATOMIC_RELAXED, __HIP_MEMORY_SCOPE_AGENT);
      for (; it < n1 - PRELD - 1; ++it) BAR();
      if (rfl(seen) < call) wait_ge(pf, call, tmo, call, true, mark, lane);
    }
    if (staged) {
      const unsigned* fl = (const unsigned*)(myws + WS_XCNT + T1.src * 512);
#pragma unroll 1
      for (int c = 0; c < 4; ++c) {
        for (; it < J0 + 16 * c - 5; ++it) BAR();
        const unsigned seen = __hip_atomic_load(fl + c * 32, __ATOMIC_RELAXED, __HIP_MEMORY_SCOPE_SYSTEM);
        for (; it < J0 + 16 * c; ++it) BAR();
        if (rfl(seen) < call) wait_ge(fl + c * 32, call, tmo, call, true, mark, lane);
        if (c == 0) TS(6);
      }
    }
  } else if (wid == 6) {
    u32x4 xv[4];
#pragma unroll
    for (int i = 0; i < 4; ++i) {
      const int P = b * 4 + i;
      xv[i] = *(const u32x4*)(p.x + 3 * XTILE + (size_t)((P & 15) * 8 + (lane >> 3)) * 8192 + (P >> 4) * 128 + ((lane & 7) ^ ((4 * (P & 1) + (lane >> 4)) & 7)) * 16);
    }
    const unsigned done = __hip_atomic_load((unsigned*)(myws + WS_DONE), __ATOMIC_RELAXED, __HIP_MEMORY_SCOPE_AGENT);
    const unsigned peer = (unsigned)(rank + 1 + lane) & 7;
    BAR();
    int it = 0;
    const int ch = b >> 6;
    const unsigned tn = now32() + XDELAY + (unsigned)b * TAU_X;
    while ((int)(now32() - tn) < 0 && it < 36 + 16 * ch) { BAR(); ++it; }
    __builtin_amdgcn_s_waitcnt(0x0F70);
    const unsigned call = rfl(done) / NWG + 1;
    char* pws = p.ws[0];
#pragma unroll
    for (int q = 1; q < NR; ++q) pws = peer == (unsigned)q ? p.ws[q] : pws;
    const size_t so = WS_XBUF + (call & 1) * XPAR + (size_t)rank * XTILE + (size_t)b * 4096 + lane * 16;
#pragma unroll
    for (int i = 0; i < 4; ++i)
#pragma unroll
      for (int q = 0; q < 7; ++q) st16_sys(p.ws[(rank + 1 + q) & 7] + so + i * 1024, xv[i]);
    TS(9);
    BAR(); BAR(); BAR(); BAR(); BAR();
    wait_vm<0>();
    unsigned tick; unsigned long long sv;
    asm volatile("s_mov_b64 %1, exec\n\ts_mov_b64 exec, 1\n\tglobal_atomic_add %0, %2, %3, off sc0 sc1\n\ts_mov_b64 exec, %1"
                 : "=&v"(tick), "=&s"(sv) : "v"(myws + WS_XLOC + ch * 128), "v"(1u) : "memory");
    BAR(); BAR(); BAR();
    it += 8;
    asm volatile("s_waitcnt vmcnt(0)" : "+v"(tick) :: "memory");
    if (rfl(tick) == call * 64 - 1) {
      if (lane < 7) __hip_atomic_store((unsigned*)(pws + WS_XCNT + rank * 512 + ch * 128), call, __ATOMIC_RELAXED, __HIP_MEMORY_SCOPE_SYSTEM);
    }
    if (lane == 0) __hip_atomic_fetch_add((unsigned*)(myws + WS_DONE), 1u, __ATOMIC_RELAXED, __HIP_MEMORY_SCOPE_AGENT);
    TS(5);
    if (b == NWG - 1) {
      for (; it < NJ; ++it) BAR();
      wait_ge((const unsigned*)(myws + WS_OUTCNT + peer * 128), call * 4 * (48 + (lane + 1 == 8 - (1 + Q4 % 7) ? 1 : 0)), tmo, call, lane < 7, mark, lane);
      TS(8);
    }
  } else if (wid >= 4) {
    const int pw = wid - 4;
    unsigned done0;
    asm volatile("global_load_dword %0, %1, off sc1" : "=v"(done0) : "v"(myws + WS_DONE) : "memory");
    const auto srdB = __builtin_amdgcn_make_buffer_rsrc((void*)p.w, 0, KDIM * NDIM * 2, 0x00020000);
    unsigned voffA[2], voffB[2];
#pragma unroll
    for (int i = 0; i < 2; ++i) {
      voffA[i] = (unsigned)(lane >> 3) * 8192u + (unsigned)(((lane & 7) ^ ((4 * i + (lane >> 4)) & 7)) * 16);
      voffB[i] = (unsigned)(lane >> 4) * 32768u + (unsigned)(((lane & 15) ^ (((lane >> 4) << 2) | (i << 1))) * 16);
    }
    unsigned sA = (unsigned)(64 * pw) * 8192u, stepK = 128u, stepI = 65536u;
    unsigned sB, G, kv, kp, kb;
    auto srdA = __builtin_amdgcn_make_buffer_rsrc((void*)p.x, 0, (int)XTILE, 0x00020000);
    auto begin = [&](unsigned d, unsigned k0) {
      srdA = __builtin_amdgcn_make_buffer_rsrc((void*)(p.x + (size_t)(d & 3) * XTILE), 0, (int)XTILE, 0x00020000);
      sB = (unsigned)(32 * pw) * 32768u + ((((unsigned)rank + ((d >> 2) & 7)) & 7) * 16 + ((d >> 5) & 15)) * 256u;
      G = ((d >> 9) & 3) + 1; kv = (d >> 11) & 3; kp = 0; kb = k0;
    };
    unsigned soA, soB, ldsb = (unsigned)pw * 8192u;
    auto next = [&]() {
      const unsigned kt = min(kb + kv, 63u);
      soA = sA + kt * stepK; soB = sB + kt * (64u * 32768u);
      kv = kv + 1 == G ? 0 : kv + 1; ++kp;
      if (kp == G) { kp = 0; kb += G; }
    };
    auto issue = [&]() {
      LAS char* d = lds + ldsb;
#pragma unroll
      for (int i = 0; i < 8; ++i) __builtin_amdgcn_raw_ptr_buffer_load_lds(srdA, (LAS void*)(d + i * 1024), 16, voffA[i & 1], soA + i * stepI, 0, 17);
#pragma unroll
      for (int i = 0; i < 8; ++i) __builtin_amdgcn_raw_ptr_buffer_load_lds(srdB, (LAS void*)(d + NS * SLOT + i * 1024), 16, voffB[(i >> 1) & 1], soB + i * (4u * 32768u), 0, 0);
      ldsb = ldsb + SLOT >= NS * SLOT ? ldsb + SLOT - NS * SLOT : ldsb + SLOT;
    };
    static_assert(HK == 0 || HK > NS, "the prologue stays inside the first piece");
    begin(helper ? aux : desc, NKT - n1);
    next(); issue(); next(); issue();
    asm volatile("s_waitcnt vmcnt(16)" : "+v"(done0) :: "memory");
    BAR();
#pragma unroll
    for (int i = 2; i < NS; ++i) { next(); issue(); }
    next();
    unsigned tpace = now32();
    const bool crit = owner || T1.dest != rank;
    const unsigned pace1 = helper ? PH1 : crit ? PO1 : PN1, pace2 = helper ? PH2 : crit ? PO2 : PN2;
    for (int j = 0; j < NJ - NS; ++j) {
      tpace += j < NKT ? pace1 : pace2;
      while ((int)(now32() - tpace) < 0) __builtin_amdgcn_s_sleep(1);
      wait_vm<48>(); BAR();
      issue();
      SCHED();
      if (j == n0 - NS - 1) begin(desc, 0);
      if (j == J0 - 1) {
        begin(desc >> 16, 0);
        if (T1.src != rank) {
          srdA = __builtin_amdgcn_make_buffer_rsrc((void*)(myws + WS_XBUF + ((rfl(done0) / NWG + 1) & 1) * XPAR + (size_t)T1.src * XTILE), 0, (int)XTILE, 0x00020000);
          sA = (unsigned)pw * 8192u; stepK = 16384u; stepI = 1024u; voffA[0] = voffA[1] = (unsigned)lane * 16u;
        }
      }
      next();
    }
    wait_vm<48>(); BAR();
    wait_vm<32>(); BAR();
    wait_vm<16>(); BAR();
    wait_vm<0>(); BAR();
    BAR();
  } else {
    const int wr = wid >> 1, wc = wid & 1, fr = lane & 15, fq = lane >> 4;
    const unsigned baseA0 = (unsigned)(wr * 64 + fr) * 128u + (unsigned)((fq ^ ((fr >> 1) & 7)) * 16);
    unsigned baseB[4];
    { const int q = (lane & 15) >> 2, pp = lane & 3; const int s = (q << 2) | ((fq & 1) << 1);
#pragma unroll
      for (int n = 0; n < 4; ++n) baseB[n] = (unsigned)(NS * SLOT) + (unsigned)(8 * fq + q) * 256u + (unsigned)(((8 * wc + 2 * n) ^ s) * 16 + 8 * pp); }
    f32x4 acc[4][4];
    bf16x8 FA[2][4], FB[2][4];
    u32x4 O[8];
    typedef LAS char* lptr;
    auto rd = [&](bf16x8 (&A)[4], bf16x8 (&B)[4], lptr pa, lptr (&pb)[4], int ks) {
      auto rb = [&](int n) {
        bf16x4 lo = __builtin_amdgcn_ds_read_tr16_b64_v4i16((LAS bf16x4*)(pb[n] + ks * 8192));
        bf16x4 hi = __builtin_amdgcn_ds_read_tr16_b64_v4i16((LAS bf16x4*)(pb[n] + ks * 8192 + 1024));
        B[n] = __builtin_shufflevector(lo, hi, 0, 1, 2, 3, 4, 5, 6, 7);
      };
      rb(0);
#pragma unroll
      for (int m = 0; m < 4; ++m) A[m] = *(const LAS bf16x8*)(pa + m * 2048);
      rb(1); rb(2); rb(3);
    };
    auto mm = [&](bf16x8 (&A)[4], bf16x8 (&B)[4], int lo, int hi) {
#pragma unroll
      for (int i = lo; i < hi; ++i) {
        const int n = i >> 2, m = (n & 1) ? 3 - (i & 3) : (i & 3);
        acc[m][n] = __builtin_amdgcn_mfma_f32_16x16x32_bf16(B[n], A[m], acc[m][n], 0, 0, 0);
      }
    };
    auto pairs = [&](int n) {
#pragma unroll
      for (int i = 0; i < n; ++i) { __builtin_amdgcn_sched_group_barrier(0x8, 1, 0); __builtin_amdgcn_sched_group_barrier(0x100, 1, 0); }
    };
    auto out_ptr = [&](const Tile& T) {
      char* o = p.out[T.dest] + (size_t)(T.src * MLOC + T.rt * 128 + wr * 64) * (NSH * 2) + (T.ct * 128 + wc * 64) * 2;
      asm volatile("" : "+s"(o));
      return o;
    };
    char* const o0 = out_ptr(T0); char* const o1 = out_ptr(T1);
    unsigned* cnt0 = (unsigned*)(p.ws[T0.dest] + WS_OUTCNT + rank * 128), * cnt1 = (unsigned*)(p.ws[T1.dest] + WS_OUTCNT + rank * 128);
    asm volatile("" : "+s"(cnt0), "+s"(cnt1));
    const unsigned vo = (unsigned)fr * (NSH * 2) + 16u * ((fq >> 1) + 2 * (fq & 1));
    auto pack1 = [&](int m, int q) {
      const unsigned x0 = cvtpk(gelu1(acc[m][2 * q][0]), gelu1(acc[m][2 * q][1])), x1 = cvtpk(gelu1(acc[m][2 * q][2]), gelu1(acc[m][2 * q][3]));
      const unsigned y0 = cvtpk(gelu1(acc[m][2 * q + 1][0]), gelu1(acc[m][2 * q + 1][1])), y1 = cvtpk(gelu1(acc[m][2 * q + 1][2]), gelu1(acc[m][2 * q + 1][3]));
      const auto r0 = __builtin_amdgcn_permlane16_swap(x0, y0, false, false);
      const auto r1 = __builtin_amdgcn_permlane16_swap(x1, y1, false, false);
      return (u32x4){r0[0], r1[0], r0[1], r1[1]};
    };
    auto signal = [&](unsigned* cnt) {
      wait_vm<0>();
      if (lane == 0) __hip_atomic_fetch_add(cnt, 1u, __ATOMIC_RELAXED, __HIP_MEMORY_SCOPE_SYSTEM);
    };
    unsigned so = 0;
    lptr pB[4], pA0 = lds + baseA0, pA1 = lds + (baseA0 ^ 64u);
#pragma unroll
    for (int n = 0; n < 4; ++n) pB[n] = lds + baseB[n];
#define PART1() do { SCHED(); rd(FA[1], FB[1], pA1, pB, 1); mm(FA[0], FB[0], 0, 16); pairs(12); __builtin_amdgcn_sched_group_barrier(0x8, 4, 0); SCHED(); } while (0)
#define PART2(WAIT) do { \
      so = so + SLOT == NS * SLOT ? 0 : so + SLOT; \
      pA0 = lds + baseA0 + so; pA1 = lds + (baseA0 ^ 64u) + so; \
      _Pragma("unroll") for (int n = 0; n < 4; ++n) pB[n] = lds + baseB[n] + so; \
      mm(FA[1], FB[1], 0, 4); \
      asm volatile("" : "+v"(pA0), "+v"(pA1), "+v"(pB[0]), "+v"(pB[1]), "+v"(pB[2]), "+v"(pB[3])); \
      _Pragma("unroll") for (int i = 0; i < 3; ++i) { __builtin_amdgcn_sched_group_barrier(0x8, 1, 0); __builtin_amdgcn_sched_group_barrier(0x2, 2, 0); } \
      __builtin_amdgcn_sched_group_barrier(0x8, 1, 0); \
      SCHED(); __builtin_amdgcn_s_waitcnt(0xC07F); WAIT; __builtin_amdgcn_s_barrier(); SCHED(); \
      rd(FA[0], FB[0], pA0, pB, 0); mm(FA[1], FB[1], 4, 16); pairs(12); SCHED(); } while (0)
    char* part = myws + WS_PART + (size_t)pidx * PART + wid * 16384;
    if (wid == 0) TS(0);
    BAR();
    if (wid == 0) TS(1);
    rd(FA[0], FB[0], pA0, pB, 0);
    f32x4 H[16];
    int hk = 99;
#pragma unroll 1
    for (int seg = helper ? 0 : 1; seg < 3; ++seg) {
      if (seg < 2) {
#pragma unroll
        for (int m = 0; m < 4; ++m)
#pragma unroll
          for (int n = 0; n < 4; ++n) acc[m][n] = (f32x4){0.f, 0.f, 0.f, 0.f};
      }
      const int nk = seg == 0 ? HK : seg == 1 ? n1 - PRELD : PRELD;
#pragma unroll 1
      for (int kt = 0; kt < nk; ++kt) {
        PART1();
        if (hk <= HDRAIN) {
#define HST(s) case s: st16_sys<1024>(part + (14 - 2 * s) * 1024, (unsigned)lane * 16u, __builtin_bit_cast(u32x4, H[15 - 2 * s])); \
                       st16_sys<0>(part + (14 - 2 * s) * 1024, (unsigned)lane * 16u, __builtin_bit_cast(u32x4, H[14 - 2 * s])); break;
          switch (hk) { HST(0) HST(1) HST(2) HST(3) HST(4) HST(5) HST(6) HST(7) case HDRAIN: wait_vm<0>(); break; default: break; }
          ++hk;
        }
        PART2((void)0);
      }
      if (seg == 0) {
#pragma unroll
        for (int i = 0; i < 16; ++i) H[i] = acc[i >> 2][i & 3];
        hk = 0;
      } else if (seg == 1) {
        const auto srdP = __builtin_amdgcn_make_buffer_rsrc((void*)part, 0, owner ? 16384 : 0, 0x00020000);
#pragma unroll
        for (int i = 0; i < 16; ++i) H[i] = __builtin_bit_cast(f32x4, __builtin_amdgcn_raw_buffer_load_b128(srdP, lane * 16 + (i & 3) * 1024, (i >> 2) * 4096, 16));
      }
    }
    if (wid == 0) TS(2);
    const unsigned tend = now32() + 120;
#pragma unroll
    for (int i = 0; i < 16; ++i) acc[i >> 2][i & 3] += H[i];
#pragma unroll
    for (int i = 0; i < 8; ++i) O[i] = pack1(i >> 1, i & 1);
#pragma unroll
    for (int m = 0; m < 4; ++m)
#pragma unroll
      for (int n = 0; n < 4; ++n) acc[m][n] = (f32x4){0.f, 0.f, 0.f, 0.f};
    auto store0 = [&](int m) {
      switch (m) {
        case 0: st16_sys<0>(o0, vo, O[0]); st16_sys<64>(o0, vo, O[1]); break;
        case 1: st16_sys<0>(o0 + 65536, vo, O[2]); st16_sys<64>(o0 + 65536, vo, O[3]); break;
        case 2: st16_sys<0>(o0 + 2 * 65536, vo, O[4]); st16_sys<64>(o0 + 2 * 65536, vo, O[5]); break;
        default: st16_sys<0>(o0 + 3 * 65536, vo, O[6]); st16_sys<64>(o0 + 3 * 65536, vo, O[7]); break;
      }
    };
    unsigned tn = tend + (aux >> 24) * TAU_O + ((desc >> 2) & 7) * (TAU_O / 8) - (helper ? HOFF : owner ? 0 : NOFF);
    unsigned tnow = tn - 1;
    int nb = 0, kl = 0;
    __builtin_amdgcn_s_waitcnt(0x0070);
#pragma unroll 1
    for (int kt = 0; kt < n2; ++kt) {
      unsigned long long t64;
      asm volatile("s_memrealtime %0" : "=s"(t64));
      PART1();
      if ((int)(tnow - tn) >= 0) { store0(nb); kl = kt; if (++nb == 4) tn += 1u << 30; }
      PART2(asm volatile("" : "+s"(t64)));
      tnow = (unsigned)t64;
    }
    if (wid == 0) TS(3);
    const bool late = nb < 4 || kl > n2 - 5;
    for (; nb < 4; ++nb) store0(nb);
    if (!late) signal(cnt0);
#pragma unroll
    for (int m = 0; m < 4; ++m) {
      st16_sys<0>(o1 + m * 65536, vo, pack1(m, 0)); SCHED();
      st16_sys<64>(o1 + m * 65536, vo, pack1(m, 1)); SCHED();
    }
    if (late) signal(cnt0);
    if (T1.dest != rank) signal(cnt1);
  }
}

extern "C" void kernel_launch(int rank, int nranks, ncclComm_t comm, hipStream_t stream,
                              void* const* d_in, const int* in_sizes, int n_in,
                              void* d_out, int out_size, const int* in_dtypes, int out_dtype,
                              void* const* d_out_all, void* const* d_ws_all, size_t ws_bytes) {
  if (nranks != NR || ws_bytes < WS_END || (unsigned)rank >= (unsigned)NR) { fprintf(stderr, "unsupported configuration\n"); return; }
  HIP_CHECK(hipFuncSetAttribute((const void*)gemm_a2a_kernel, hipFuncAttributeMaxDynamicSharedMemorySize, LDS_BYTES));
  Params p;
  p.x = (const char*)d_in[0]; p.w = (const char*)d_in[1]; p.rank = rank; p.pad = 0;
  for (int i = 0; i < NR; ++i) { p.out[i] = (char*)d_out_all[i]; p.ws[i] = (char*)d_ws_all[i]; }
  build_desc(p.desc);
  gemm_a2a_kernel<<<NWG, 512, LDS_BYTES, stream>>>(p);
  HIP_CHECK(hipGetLastError());
}
```

```cpp
#include <hip/hip_runtime.h>
#include <rccl/rccl.h>
#include <cstdio>

#define HIP_CHECK(expr) do { hipError_t _e = (expr); if (_e != hipSuccess) { \
    fprintf(stderr, "HIP error %s at %s:%d\n", hipGetErrorString(_e), __FILE__, __LINE__); \
    return; } } while (0)

#define LAS __attribute__((address_space(3)))
typedef short bf16x8 __attribute__((ext_vector_type(8)));
typedef short bf16x4 __attribute__((ext_vector_type(4)));
typedef float f32x4 __attribute__((ext_vector_type(4)));
typedef float f32x2 __attribute__((ext_vector_type(2)));
typedef unsigned u32x4 __attribute__((ext_vector_type(4)));
typedef __bf16 bfx2 __attribute__((ext_vector_type(2)));

#define TAU_O 74
#define TAU_X 8
#define FO 35
#define FN 17
#define FH 38
#define HK 9
#define XDELAY 100
#define PO1 49
#define PH1 49
#define PN1 49
#define PO2 46
#define PH2 46
#define PN2 46
constexpr int NR = 8;
constexpr int KDIM = 4096, NDIM = 16384, MLOC = 512, NSH = 2048;
constexpr int NKT = 64, SLOT = 16384, NS = 5, NWG = 256;
constexpr int LDS_BYTES = 2 * NS * SLOT;
constexpr int HDRAIN = 12, PRELD = 3;
constexpr int Q4 = 80;
constexpr unsigned NOFF = HK * 47, HOFF = 2 * HK * 47 + 60;
constexpr size_t WS_DONE = 0, WS_TMO = 128, WS_OUTCNT = 4096, WS_XCNT = 8192, WS_XLOC = 16384, WS_PFLAG = 32768, WS_STAMP = 65536;
constexpr size_t XTILE = 128 * KDIM * 2;
constexpr size_t WS_XBUF = 1u << 20, XPAR = NR * XTILE;
constexpr size_t WS_PART = WS_XBUF + 2 * XPAR, PART = 128 * 128 * 4, WS_END = WS_PART + 96 * PART;
constexpr unsigned long long DEADLINE = 40000000ull;

enum { ROLE_OWNER = 1, ROLE_HELPER = 2 };
struct Params { const char* x; const char* w; char* out[NR]; char* ws[NR]; int rank; int pad; unsigned desc[NWG][2]; };
struct Tile { int src; int rt; int dest; int ct; };
__device__ __forceinline__ Tile tile_of(unsigned d, int rank) {
  Tile r; r.src = (rank + (int)((d >> 13) & 7)) & 7; r.rt = d & 3; r.dest = (rank + (int)((d >> 2) & 7)) & 7; r.ct = (d >> 5) & 15; return r;
}
static inline unsigned tile_code(int so, int rt, int dof, int ct, int G, int rot) { return (unsigned)(rt | dof << 2 | ct << 5 | (G - 1) << 9 | (rot % G) << 11 | so << 13); }
static void build_desc(unsigned (*desc)[2]) {
  unsigned T[2][8][32]; int n[2][8] = {};
  auto put = [&](int w, int x, unsigned c) { T[w][x][n[w][x]++] = c; };
  auto code = [&](int q, int rt) { return tile_code(0, rt, 1 + q % 7, q / 7, q == Q4 ? 4 : 3, rt + q); };
  for (int k = 0; k < 80; ++k) for (int rt = 0; rt < 3; ++rt) put(0, k / 10, code(k, rt));
  for (int i = 0; i < 16; ++i) put(0, i / 2, code(i < 12 ? 81 + i / 3 : Q4, i < 12 ? i % 3 : i - 12));
  for (int k = 0; k < 24; ++k) for (int rt = 0; rt < 3; ++rt) put(1, k / 6, code(85 + k, rt));
  for (int x = 0; x < 4; ++x) for (int c = 0; c < 4; ++c) for (int r = 0; r < 3; ++r) put(1, x, tile_code(0, r, 0, 4 * x + c, 3, r + c));
  for (int k = 0; k < 3; ++k) { put(1, k, code(109 + k, 0)); put(1, k, code(109 + k, 1)); }
  put(1, 3, code(109, 2)); put(1, 3, code(110, 2));
  for (int g = 0; g < 2; ++g) for (int h = 0; h < 2; ++h) {
    for (int c = 0; c < 8; ++c) for (int r = 0; r < 4; ++r)
      if (!(4 * g + r == 8 - (1 + Q4 % 7) && 8 * h + c == Q4 / 7)) put(1, 4 + 2 * g + h, tile_code(4 * g + r, 3, 0, 8 * h + c, 4, r + c + 2 * h + g));
    if (n[1][4 + 2 * g + h] < 32) put(1, 4 + 2 * g + h, code(111, 2));
  }
  int role[NWG], cnt[8][3] = {}, seen[8][3] = {}, key[NWG], own[96], nown = 0;
  for (int b = 0; b < NWG; ++b) { desc[b][0] = T[0][b & 7][b >> 3] | T[1][b & 7][b >> 3] << 16; desc[b][1] = 0; role[b] = 0; }
  for (int pass = 0; pass < 2; ++pass) for (int x = 0; x < 4; ++x) for (int i = pass ? 30 : 0; i < (pass ? 32 : 18); ++i) own[nown++] = i * 8 + x;
  for (int k = 0; HK && k < nown; ++k) {
    const int o = own[k], h = (k % 30) * 8 + 4 + k / 30;
    desc[o][1] = (unsigned)k << 16 | 1u << 14; role[o] = ROLE_OWNER;
    desc[h][1] = (desc[o][0] & 0x1fff) | (unsigned)k << 16; role[h] = ROLE_HELPER;
  }
  for (int b = 0; b < NWG; ++b) cnt[(desc[b][0] >> 2) & 7][role[b]]++;
  for (int b = 0; b < NWG; ++b) {
    const int d = (desc[b][0] >> 2) & 7, r = role[b];
    const int u = (2 * seen[d][r]++ + 1) * 500 / cnt[d][r];
    key[b] = r == ROLE_OWNER ? u * FO : r == ROLE_HELPER ? 1000 * FH + u * (100 - FH) : 1000 * FN + u * (100 - FN);
    key[b] = key[b] * 256 + b;
  }
  for (int b = 0; b < NWG; ++b) {
    unsigned k = 0;
    for (int c = 0; c < NWG; ++c) k += ((desc[c][0] ^ desc[b][0]) & 0x1c) == 0 && key[c] < key[b];
    desc[b][1] |= k << 24;
  }
}

__device__ __forceinline__ unsigned cvtpk(float lo, float hi) { f32x2 v = {lo, hi}; bfx2 b = __builtin_convertvector(v, bfx2); return __builtin_bit_cast(unsigned, b); }
__device__ __forceinline__ float gelu1(float y) {
  const float k0 = -2.0f * 0.7978845608028654f * 1.4426950408889634f, k1 = k0 * 0.044715f;
  const float z = y * __builtin_fmaf(y * y, k1, k0);
  return y * __builtin_amdgcn_rcpf(1.0f + __builtin_amdgcn_exp2f(z));
}
__device__ __forceinline__ unsigned rfl(unsigned v) { return (unsigned)__builtin_amdgcn_readfirstlane((int)v); }
__device__ __forceinline__ unsigned now32() { return (unsigned)__builtin_amdgcn_s_memrealtime(); }
template <int N> __device__ __forceinline__ void wait_vm() { asm volatile("s_waitcnt vmcnt(%0)" :: "i"(N) : "memory"); }
__device__ __forceinline__ void st16_sys(char* p, u32x4 v) { asm volatile("global_store_dwordx4 %0, %1, off sc0 sc1\n\ts_nop 1" :: "v"(p), "v"(v) : "memory"); }
template <int OFF> __device__ __forceinline__ void st16_sys(char* sbase, unsigned voff, u32x4 v) {
  asm volatile("global_store_dwordx4 %0, %1, %2 offset:%3 sc0 sc1\n\ts_nop 1" :: "v"(voff), "v"(v), "s"(sbase), "i"(OFF) : "memory");
}
#define BAR() do { asm volatile("" ::: "memory"); __builtin_amdgcn_s_barrier(); asm volatile("" ::: "memory"); } while (0)
#define SCHED() __builtin_amdgcn_sched_barrier(0)
#define TS(k) do {} while (0)

__device__ __forceinline__ void wait_ge(const unsigned* flag, unsigned want, unsigned* tmo, unsigned call, bool active, char* mark, int lane) {
  const unsigned long long t0 = __builtin_amdgcn_s_memrealtime();
  for (unsigned n = 1;; ++n) {
    const bool ok = !active || __hip_atomic_load(flag, __ATOMIC_RELAXED, __HIP_MEMORY_SCOPE_SYSTEM) >= want;
    if (__all(ok)) return;
    if ((n & 15) == 0 &&
        (__hip_atomic_load(tmo, __ATOMIC_RELAXED, __HIP_MEMORY_SCOPE_AGENT) == call || __builtin_amdgcn_s_memrealtime() - t0 > DEADLINE)) {
      __hip_atomic_store(tmo, call, __ATOMIC_RELAXED, __HIP_MEMORY_SCOPE_AGENT);
      if (lane == 0) __hip_atomic_store((unsigned*)mark, 0x7fc07fc0u, __ATOMIC_RELAXED, __HIP_MEMORY_SCOPE_SYSTEM);
      return;
    }
    __builtin_amdgcn_s_sleep(1);
  }
}

__global__ void __launch_bounds__(512, 2) gemm_a2a_kernel(Params p) {
  extern __shared__ __attribute__((aligned(16))) char smem[];
  LAS char* lds = (LAS char*)smem;
  const int tid = threadIdx.x, wid = (int)rfl(tid >> 6), lane = tid & 63;
  const int b = blockIdx.x, rank = p.rank;
  const unsigned desc = p.desc[b][0], aux = p.desc[b][1];
  const Tile T0 = tile_of(desc, rank), T1 = tile_of(desc >> 16, rank);
  const bool owner = (aux >> 14) & 1, helper = (aux & 0x1fffu) != 0;
  const int n0 = helper ? HK : 0, n1 = owner ? NKT - HK : NKT, n2 = NKT;
  const int NJ = n0 + n1 + n2;
  const int J0 = n0 + n1 - NS;
  char* myws = p.ws[rank];
  unsigned* tmo = (unsigned*)(myws + WS_TMO);
  char* mark = p.out[rank] + (size_t)rank * MLOC * NSH * 2;
  const unsigned pidx = (aux >> 16) & 127;

  if (wid == 7) {
    const bool staged = T1.src != rank;
    if (!staged && !owner && !helper) return;
    const unsigned call = rfl(__hip_atomic_load((unsigned*)(myws + WS_DONE), __ATOMIC_RELAXED, __HIP_MEMORY_SCOPE_AGENT)) / NWG + 1;
    unsigned* pf = (unsigned*)(myws + WS_PFLAG + pidx * 128);
    BAR();
    int it = 0;
    if (helper) {
      for (; it <= HK + HDRAIN; ++it) BAR();
      if (lane == 0) __hip_atomic_fetch_add(pf, 1u, __ATOMIC_RELAXED, __HIP_MEMORY_SCOPE_AGENT);
    }
    if (owner) {
      for (; it < n1 - PRELD - 6; ++it) BAR();
      const unsigned seen = __hip_atomic_load(pf, __ATOMIC_RELAXED, __HIP_MEMORY_SCOPE_AGENT);
      for (; it < n1 - PRELD - 1; ++it) BAR();
      if (rfl(seen) < call) wait_ge(pf, call, tmo, call, true, mark, lane);
    }
    if (staged) {
      const unsigned* fl = (const unsigned*)(myws + WS_XCNT + T1.src * 512);
#pragma unroll 1
      for (int c = 0; c < 4; ++c) {
        for (; it < J0 + 16 * c - 5; ++it) BAR();
        const unsigned seen = __hip_atomic_load(fl + c * 32, __ATOMIC_RELAXED, __HIP_MEMORY_SCOPE_SYSTEM);
        for (; it < J0 + 16 * c; ++it) BAR();
        if (rfl(seen) < call) wait_ge(fl + c * 32, call, tmo, call, true, mark, lane);
        if (c == 0) TS(6);
      }
    }
  } else if (wid == 6) {
    u32x4 xv[4];
#pragma unroll
    for (int i = 0; i < 4; ++i) {
      const int P = b * 4 + i;
      xv[i] = *(const u32x4*)(p.x + 3 * XTILE + (size_t)((P & 15) * 8 + (lane >> 3)) * 8192 + (P >> 4) * 128 + ((lane & 7) ^ ((4 * (P & 1) + (lane >> 4)) & 7)) * 16);
    }
    const unsigned done = __hip_atomic_load((unsigned*)(myws + WS_DONE), __ATOMIC_RELAXED, __HIP_MEMORY_SCOPE_AGENT);
    const unsigned peer = (unsigned)(rank + 1 + lane) & 7;
    BAR();
    int it = 0;
    const int ch = b >> 6;
    const unsigned tn = now32() + XDELAY + (unsigned)b * TAU_X;
    while ((int)(now32() - tn) < 0 && it < 36 + 16 * ch) { BAR(); ++it; }
    __builtin_amdgcn_s_waitcnt(0x0F70);
    const unsigned call = rfl(done) / NWG + 1;
    char* pws = p.ws[0];
#pragma unroll
    for (int q = 1; q < NR; ++q) pws = peer == (unsigned)q ? p.ws[q] : pws;
    const size_t so = WS_XBUF + (call & 1) * XPAR + (size_t)rank * XTILE + (size_t)b * 4096 + lane * 16;
#pragma unroll
    for (int i = 0; i < 4; ++i)
#pragma unroll
      for (int q = 0; q < 7; ++q) st16_sys(p.ws[(rank + 1 + q) & 7] + so + i * 1024, xv[i]);
    TS(9);
    BAR(); BAR(); BAR(); BAR(); BAR();
    wait_vm<0>();
    unsigned tick; unsigned long long sv;
    asm volatile("s_mov_b64 %1, exec\n\ts_mov_b64 exec, 1\n\tglobal_atomic_add %0, %2, %3, off sc0 sc1\n\ts_mov_b64 exec, %1"
                 : "=&v"(tick), "=&s"(sv) : "v"(myws + WS_XLOC + ch * 128), "v"(1u) : "memory");
    BAR(); BAR(); BAR();
    it += 8;
    asm volatile("s_waitcnt vmcnt(0)" : "+v"(tick) :: "memory");
    if (rfl(tick) == call * 64 - 1) {
      if (lane < 7) __hip_atomic_store((unsigned*)(pws + WS_XCNT + rank * 512 + ch * 128), call, __ATOMIC_RELAXED, __HIP_MEMORY_SCOPE_SYSTEM);
    }
    if (lane == 0) __hip_atomic_fetch_add((unsigned*)(myws + WS_DONE), 1u, __ATOMIC_RELAXED, __HIP_MEMORY_SCOPE_AGENT);
    TS(5);
    if (b == NWG - 1) {
      for (; it < NJ; ++it) BAR();
      wait_ge((const unsigned*)(myws + WS_OUTCNT + peer * 128), call * 4 * (48 + (lane + 1 == 8 - (1 + Q4 % 7) ? 1 : 0)), tmo, call, lane < 7, mark, lane);
      TS(8);
    }
  } else if (wid >= 4) {
    const int pw = wid - 4;
    unsigned done0;
    asm volatile("global_load_dword %0, %1, off sc1" : "=v"(done0) : "v"(myws + WS_DONE) : "memory");
    const auto srdB = __builtin_amdgcn_make_buffer_rsrc((void*)p.w, 0, KDIM * NDIM * 2, 0x00020000);
    unsigned voffA[2], voffB[2];
#pragma unroll
    for (int i = 0; i < 2; ++i) {
      voffA[i] = (unsigned)(lane >> 3) * 8192u + (unsigned)(((lane & 7) ^ ((4 * i + (lane >> 4)) & 7)) * 16);
      voffB[i] = (unsigned)(lane >> 4) * 32768u + (unsigned)(((lane & 15) ^ (((lane >> 4) << 2) | (i << 1))) * 16);
    }
    unsigned sA = (unsigned)(64 * pw) * 8192u, stepK = 128u, stepI = 65536u;
    unsigned sB, G, kv, kp, kb;
    auto srdA = __builtin_amdgcn_make_buffer_rsrc((void*)p.x, 0, (int)XTILE, 0x00020000);
    auto begin = [&](unsigned d, unsigned k0) {
      srdA = __builtin_amdgcn_make_buffer_rsrc((void*)(p.x + (size_t)(d & 3) * XTILE), 0, (int)XTILE, 0x00020000);
      sB = (unsigned)(32 * pw) * 32768u + ((((unsigned)rank + ((d >> 2) & 7)) & 7) * 16 + ((d >> 5) & 15)) * 256u;
      G = ((d >> 9) & 3) + 1; kv = (d >> 11) & 3; kp = 0; kb = k0;
    };
    unsigned soA, soB, ldsb = (unsigned)pw * 8192u;
    auto next = [&]() {
      const unsigned kt = min(kb + kv, 63u);
      soA = sA + kt * stepK; soB = sB + kt * (64u * 32768u);
      kv = kv + 1 == G ? 0 : kv + 1; ++kp;
      if (kp == G) { kp = 0; kb += G; }
    };
    auto issue = [&]() {
      LAS char* d = lds + ldsb;
#pragma unroll
      for (int i = 0; i < 8; ++i) __builtin_amdgcn_raw_ptr_buffer_load_lds(srdA, (LAS void*)(d + i * 1024), 16, voffA[i & 1], soA + i * stepI, 0, 17);
#pragma unroll
      for (int i = 0; i < 8; ++i) __builtin_amdgcn_raw_ptr_buffer_load_lds(srdB, (LAS void*)(d + NS * SLOT + i * 1024), 16, voffB[(i >> 1) & 1], soB + i * (4u * 32768u), 0, 0);
      ldsb = ldsb + SLOT >= NS * SLOT ? ldsb + SLOT - NS * SLOT : ldsb + SLOT;
    };
    static_assert(HK == 0 || HK > NS, "the prologue stays inside the first piece");
    begin(helper ? aux : desc, NKT - n1);
    next(); issue(); next(); issue();
    asm volatile("s_waitcnt vmcnt(16)" : "+v"(done0) :: "memory");
    BAR();
#pragma unroll
    for (int i = 2; i < NS; ++i) { next(); issue(); }
    next();
    unsigned tpace = now32();
    const bool crit = owner || T1.dest != rank;
    const unsigned pace1 = helper ? PH1 : crit ? PO1 : PN1, pace2 = helper ? PH2 : crit ? PO2 : PN2;
    for (int j = 0; j < NJ - NS; ++j) {
      tpace += j < NKT ? pace1 : pace2;
      while ((int)(now32() - tpace) < 0) __builtin_amdgcn_s_sleep(1);
      wait_vm<48>(); BAR();
      issue();
      SCHED();
      if (j == n0 - NS - 1) begin(desc, 0);
      if (j == J0 - 1) {
        begin(desc >> 16, 0);
        if (T1.src != rank) {
          srdA = __builtin_amdgcn_make_buffer_rsrc((void*)(myws + WS_XBUF + ((rfl(done0) / NWG + 1) & 1) * XPAR + (size_t)T1.src * XTILE), 0, (int)XTILE, 0x00020000);
          sA = (unsigned)pw * 8192u; stepK = 16384u; stepI = 1024u; voffA[0] = voffA[1] = (unsigned)lane * 16u;
        }
      }
      next();
    }
    wait_vm<48>(); BAR();
    wait_vm<32>(); BAR();
    wait_vm<16>(); BAR();
    wait_vm<0>(); BAR();
    BAR();
  } else {
    const int wr = wid >> 1, wc = wid & 1, fr = lane & 15, fq = lane >> 4;
    const unsigned baseA0 = (unsigned)(wr * 64 + fr) * 128u + (unsigned)((fq ^ ((fr >> 1) & 7)) * 16);
    unsigned baseB[4];
    { const int q = (lane & 15) >> 2, pp = lane & 3; const int s = (q << 2) | ((fq & 1) << 1);
#pragma unroll
      for (int n = 0; n < 4; ++n) baseB[n] = (unsigned)(NS * SLOT) + (unsigned)(8 * fq + q) * 256u + (unsigned)(((8 * wc + 2 * n) ^ s) * 16 + 8 * pp); }
    f32x4 acc[4][4];
    bf16x8 FA[2][4], FB[2][4];
    u32x4 O[8];
    typedef LAS char* lptr;
    auto rd = [&](bf16x8 (&A)[4], bf16x8 (&B)[4], lptr pa, lptr (&pb)[4], int ks) {
      auto rb = [&](int n) {
        bf16x4 lo = __builtin_amdgcn_ds_read_tr16_b64_v4i16((LAS bf16x4*)(pb[n] + ks * 8192));
        bf16x4 hi = __builtin_amdgcn_ds_read_tr16_b64_v4i16((LAS bf16x4*)(pb[n] + ks * 8192 + 1024));
        B[n] = __builtin_shufflevector(lo, hi, 0, 1, 2, 3, 4, 5, 6, 7);
      };
      rb(0);
#pragma unroll
      for (int m = 0; m < 4; ++m) A[m] = *(const LAS bf16x8*)(pa + m * 2048);
      rb(1); rb(2); rb(3);
    };
    auto mm = [&](bf16x8 (&A)[4], bf16x8 (&B)[4], int lo, int hi) {
#pragma unroll
      for (int i = lo; i < hi; ++i) {
        const int n = i >> 2, m = (n & 1) ? 3 - (i & 3) : (i & 3);
        acc[m][n] = __builtin_amdgcn_mfma_f32_16x16x32_bf16(B[n], A[m], acc[m][n], 0, 0, 0);
      }
    };
    auto pairs = [&](int n) {
#pragma unroll
      for (int i = 0; i < n; ++i) { __builtin_amdgcn_sched_group_barrier(0x8, 1, 0); __builtin_amdgcn_sched_group_barrier(0x100, 1, 0); }
    };
    auto out_ptr = [&](const Tile& T) {
      char* o = p.out[T.dest] + (size_t)(T.src * MLOC + T.rt * 128 + wr * 64) * (NSH * 2) + (T.ct * 128 + wc * 64) * 2;
      asm volatile("" : "+s"(o));
      return o;
    };
    char* const o0 = out_ptr(T0); char* const o1 = out_ptr(T1);
    unsigned* cnt0 = (unsigned*)(p.ws[T0.dest] + WS_OUTCNT + rank * 128), * cnt1 = (unsigned*)(p.ws[T1.dest] + WS_OUTCNT + rank * 128);
    asm volatile("" : "+s"(cnt0), "+s"(cnt1));
    const unsigned vo = (unsigned)fr * (NSH * 2) + 16u * ((fq >> 1) + 2 * (fq & 1));
    auto pack1 = [&](int m, int q) {
      const unsigned x0 = cvtpk(gelu1(acc[m][2 * q][0]), gelu1(acc[m][2 * q][1])), x1 = cvtpk(gelu1(acc[m][2 * q][2]), gelu1(acc[m][2 * q][3]));
      const unsigned y0 = cvtpk(gelu1(acc[m][2 * q + 1][0]), gelu1(acc[m][2 * q + 1][1])), y1 = cvtpk(gelu1(acc[m][2 * q + 1][2]), gelu1(acc[m][2 * q + 1][3]));
      const auto r0 = __builtin_amdgcn_permlane16_swap(x0, y0, false, false);
      const auto r1 = __builtin_amdgcn_permlane16_swap(x1, y1, false, false);
      return (u32x4){r0[0], r1[0], r0[1], r1[1]};
    };
    auto signal = [&](unsigned* cnt) {
      wait_vm<0>();
      if (lane == 0) __hip_atomic_fetch_add(cnt, 1u, __ATOMIC_RELAXED, __HIP_MEMORY_SCOPE_SYSTEM);
    };
    unsigned so = 0;
    lptr pB[4], pA0 = lds + baseA0, pA1 = lds + (baseA0 ^ 64u);
#pragma unroll
    for (int n = 0; n < 4; ++n) pB[n] = lds + baseB[n];
#define PART1() do { SCHED(); rd(FA[1], FB[1], pA1, pB, 1); mm(FA[0], FB[0], 0, 16); pairs(12); __builtin_amdgcn_sched_group_barrier(0x8, 4, 0); SCHED(); } while (0)
#define PART2(WAIT) do { \
      so = so + SLOT == NS * SLOT ? 0 : so + SLOT; \
      pA0 = lds + baseA0 + so; pA1 = lds + (baseA0 ^ 64u) + so; \
      _Pragma("unroll") for (int n = 0; n < 4; ++n) pB[n] = lds + baseB[n] + so; \
      mm(FA[1], FB[1], 0, 4); \
      asm volatile("" : "+v"(pA0), "+v"(pA1), "+v"(pB[0]), "+v"(pB[1]), "+v"(pB[2]), "+v"(pB[3])); \
      _Pragma("unroll") for (int i = 0; i < 3; ++i) { __builtin_amdgcn_sched_group_barrier(0x8, 1, 0); __builtin_amdgcn_sched_group_barrier(0x2, 2, 0); } \
      __builtin_amdgcn_sched_group_barrier(0x8, 1, 0); \
      SCHED(); __builtin_amdgcn_s_waitcnt(0xC07F); WAIT; __builtin_amdgcn_s_barrier(); SCHED(); \
      rd(FA[0], FB[0], pA0, pB, 0); mm(FA[1], FB[1], 4, 16); pairs(12); SCHED(); } while (0)
    char* part = myws + WS_PART + (size_t)pidx * PART + wid * 16384;
    if (wid == 0) TS(0);
    BAR();
    if (wid == 0) TS(1);
    rd(FA[0], FB[0], pA0, pB, 0);
    f32x4 H[16];
    int hk = 99;
#pragma unroll 1
    for (int seg = helper ? 0 : 1; seg < 3; ++seg) {
      if (seg < 2) {
#pragma unroll
        for (int m = 0; m < 4; ++m)
#pragma unroll
          for (int n = 0; n < 4; ++n) acc[m][n] = (f32x4){0.f, 0.f, 0.f, 0.f};
      }
      const int nk = seg == 0 ? HK : seg == 1 ? n1 - PRELD : PRELD;
#pragma unroll 1
      for (int kt = 0; kt < nk; ++kt) {
        PART1();
        if (hk <= HDRAIN) {
#define HST(s) case s: st16_sys<1024>(part + (14 - 2 * s) * 1024, (unsigned)lane * 16u, __builtin_bit_cast(u32x4, H[15 - 2 * s])); \
                       st16_sys<0>(part + (14 - 2 * s) * 1024, (unsigned)lane * 16u, __builtin_bit_cast(u32x4, H[14 - 2 * s])); break;
          switch (hk) { HST(0) HST(1) HST(2) HST(3) HST(4) HST(5) HST(6) HST(7) case HDRAIN: wait_vm<0>(); break; default: break; }
          ++hk;
        }
        PART2((void)0);
      }
      if (seg == 0) {
#pragma unroll
        for (int i = 0; i < 16; ++i) H[i] = acc[i >> 2][i & 3];
        hk = 0;
      } else if (seg == 1) {
        const auto srdP = __builtin_amdgcn_make_buffer_rsrc((void*)part, 0, owner ? 16384 : 0, 0x00020000);
#pragma unroll
        for (int i = 0; i < 16; ++i) H[i] = __builtin_bit_cast(f32x4, __builtin_amdgcn_raw_buffer_load_b128(srdP, lane * 16 + (i & 3) * 1024, (i >> 2) * 4096, 16));
      }
    }
    if (wid == 0) TS(2);
    const unsigned tend = now32() + 120;
#pragma unroll
    for (int i = 0; i < 16; ++i) acc[i >> 2][i & 3] += H[i];
#pragma unroll
    for (int i = 0; i < 8; ++i) O[i] = pack1(i >> 1, i & 1);
#pragma unroll
    for (int m = 0; m < 4; ++m)
#pragma unroll
      for (int n = 0; n < 4; ++n) acc[m][n] = (f32x4){0.f, 0.f, 0.f, 0.f};
    auto store0 = [&](int m) {
      switch (m) {
        case 0: st16_sys<0>(o0, vo, O[0]); st16_sys<64>(o0, vo, O[1]); break;
        case 1: st16_sys<0>(o0 + 65536, vo, O[2]); st16_sys<64>(o0 + 65536, vo, O[3]); break;
        case 2: st16_sys<0>(o0 + 2 * 65536, vo, O[4]); st16_sys<64>(o0 + 2 * 65536, vo, O[5]); break;
        default: st16_sys<0>(o0 + 3 * 65536, vo, O[6]); st16_sys<64>(o0 + 3 * 65536, vo, O[7]); break;
      }
    };
    unsigned tn = tend + (aux >> 24) * TAU_O + ((desc >> 2) & 7) * (TAU_O / 8) - (helper ? HOFF : owner ? 0 : NOFF);
    unsigned tnow = tn - 1;
    int nb = 0, kl = 0;
    __builtin_amdgcn_s_waitcnt(0x0070);
#pragma unroll 1
    for (int kt = 0; kt < n2; ++kt) {
      unsigned long long t64;
      asm volatile("s_memrealtime %0" : "=s"(t64));
      PART1();
      if ((int)(tnow - tn) >= 0) { store0(nb); kl = kt; if (++nb == 4) tn += 1u << 30; }
      PART2(asm volatile("" : "+s"(t64)));
      tnow = (unsigned)t64;
    }
    if (wid == 0) TS(3);
    const bool late = nb < 4 || kl > n2 - 5;
    for (; nb < 4; ++nb) store0(nb);
    if (!late) signal(cnt0);
#pragma unroll
    for (int m = 0; m < 4; ++m) {
      st16_sys<0>(o1 + m * 65536, vo, pack1(m, 0)); SCHED();
      st16_sys<64>(o1 + m * 65536, vo, pack1(m, 1)); SCHED();
    }
    if (late) signal(cnt0);
    if (T1.dest != rank) signal(cnt1);
  }
}

extern "C" void kernel_launch(int rank, int nranks, ncclComm_t comm, hipStream_t stream,
                              void* const* d_in, const int* in_sizes, int n_in,
                              void* d_out, int out_size, const int* in_dtypes, int out_dtype,
                              void* const* d_out_all, void* const* d_ws_all, size_t ws_bytes) {
  if (nranks != NR || ws_bytes < WS_END || (unsigned)rank >= (unsigned)NR) { fprintf(stderr, "unsupported configuration\n"); return; }
  HIP_CHECK(hipFuncSetAttribute((const void*)gemm_a2a_kernel, hipFuncAttributeMaxDynamicSharedMemorySize, LDS_BYTES));
  Params p;
  p.x = (const char*)d_in[0]; p.w = (const char*)d_in[1]; p.rank = rank; p.pad = 0;
  for (int i = 0; i < NR; ++i) { p.out[i] = (char*)d_out_all[i]; p.ws[i] = (char*)d_ws_all[i]; }
  build_desc(p.desc);
  gemm_a2a_kernel<<<NWG, 512, LDS_BYTES, stream>>>(p);
  HIP_CHECK(hipGetLastError());
}
```

```cpp
#include <hip/hip_runtime.h>
#include <rccl/rccl.h>
#include <cstdio>

#define HIP_CHECK(expr) do { hipError_t _e = (expr); if (_e != hipSuccess) { \
    fprintf(stderr, "HIP error %s at %s:%d\n", hipGetErrorString(_e), __FILE__, __LINE__); \
    return; } } while (0)

#define LAS __attribute__((address_space(3)))
typedef short bf16x8 __attribute__((ext_vector_type(8)));
typedef short bf16x4 __attribute__((ext_vector_type(4)));
typedef float f32x4 __attribute__((ext_vector_type(4)));
typedef float f32x2 __attribute__((ext_vector_type(2)));
typedef unsigned u32x4 __attribute__((ext_vector_type(4)));
typedef __bf16 bfx2 __attribute__((ext_vector_type(2)));

#define TAU_O 74
#define TAU_X 8
#define FO 35
#define FN 17
#define FH 38
#define FHE 70
#define HK 9
#define XDELAY 100
#define PO1 49
#define PH1 49
#define PN1 49
#define PO2 46
#define PH2 46
#define PN2 46
constexpr int NR = 8;
constexpr int KDIM = 4096, NDIM = 16384, MLOC = 512, NSH = 2048;
constexpr int NKT = 64, SLOT = 16384, NS = 5, NWG = 256;
constexpr int LDS_BYTES = 2 * NS * SLOT;
constexpr int HDRAIN = 12, PRELD = 3;
constexpr int Q4 = 80;
constexpr unsigned NOFF = HK * 47, HOFF = 2 * HK * 47 + 60;
constexpr size_t WS_DONE = 0, WS_TMO = 128, WS_OUTCNT = 4096, WS_XCNT = 8192, WS_XLOC = 16384, WS_PFLAG = 32768, WS_STAMP = 65536;
constexpr size_t XTILE = 128 * KDIM * 2;
constexpr size_t WS_XBUF = 1u << 20, XPAR = NR * XTILE;
constexpr size_t WS_PART = WS_XBUF + 2 * XPAR, PART = 128 * 128 * 4, WS_END = WS_PART + 96 * PART;
constexpr unsigned long long DEADLINE = 40000000ull;

enum { ROLE_OWNER = 1, ROLE_HELPER = 2 };
struct Params { const char* x; const char* w; char* out[NR]; char* ws[NR]; int rank; int pad; unsigned desc[NWG][2]; };
struct Tile { int src; int rt; int dest; int ct; };
__device__ __forceinline__ Tile tile_of(unsigned d, int rank) {
  Tile r; r.src = (rank + (int)((d >> 13) & 7)) & 7; r.rt = d & 3; r.dest = (rank + (int)((d >> 2) & 7)) & 7; r.ct = (d >> 5) & 15; return r;
}
static inline unsigned tile_code(int so, int rt, int dof, int ct, int G, int rot) { return (unsigned)(rt | dof << 2 | ct << 5 | (G - 1) << 9 | (rot % G) << 11 | so << 13); }
static void build_desc(unsigned (*desc)[2]) {
  unsigned T[2][8][32]; int n[2][8] = {};
  auto put = [&](int w, int x, unsigned c) { T[w][x][n[w][x]++] = c; };
  auto code = [&](int q, int rt) { return tile_code(0, rt, 1 + q % 7, q / 7, q == Q4 ? 4 : 3, rt + q); };
  for (int k = 0; k < 80; ++k) for (int rt = 0; rt < 3; ++rt) put(0, k / 10, code(k, rt));
  for (int i = 0; i < 16; ++i) put(0, i / 2, code(i < 12 ? 81 + i / 3 : Q4, i < 12 ? i % 3 : i - 12));
  for (int k = 0; k < 24; ++k) for (int rt = 0; rt < 3; ++rt) put(1, k / 6, code(85 + k, rt));
  for (int x = 0; x < 4; ++x) for (int c = 0; c < 4; ++c) for (int r = 0; r < 3; ++r) put(1, x, tile_code(0, r, 0, 4 * x + c, 3, r + c));
  for (int k = 0; k < 3; ++k) { put(1, k, code(109 + k, 0)); put(1, k, code(109 + k, 1)); }
  put(1, 3, code(109, 2)); put(1, 3, code(110, 2));
  for (int g = 0; g < 2; ++g) for (int h = 0; h < 2; ++h) {
    for (int c = 0; c < 8; ++c) for (int r = 0; r < 4; ++r)
      if (!(4 * g + r == 8 - (1 + Q4 % 7) && 8 * h + c == Q4 / 7)) put(1, 4 + 2 * g + h, tile_code(4 * g + r, 3, 0, 8 * h + c, 4, r + c + 2 * h + g));
    if (n[1][4 + 2 * g + h] < 32) put(1, 4 + 2 * g + h, code(111, 2));
  }
  int role[NWG], cnt[8][3] = {}, seen[8][3] = {}, key[NWG], own[96], nown = 0;
  for (int b = 0; b < NWG; ++b) { desc[b][0] = T[0][b & 7][b >> 3] | T[1][b & 7][b >> 3] << 16; desc[b][1] = 0; role[b] = 0; }
  for (int pass = 0; pass < 2; ++pass) for (int x = 0; x < 4; ++x) for (int i = pass ? 30 : 0; i < (pass ? 32 : 18); ++i) own[nown++] = i * 8 + x;
  for (int k = 0; HK && k < nown; ++k) {
    const int o = own[k], h = (k % 30) * 8 + 4 + k / 30;
    desc[o][1] = (unsigned)k << 16 | 1u << 14; role[o] = ROLE_OWNER;
    desc[h][1] = (desc[o][0] & 0x1fff) | (unsigned)k << 16; role[h] = ROLE_HELPER;
  }
  for (int b = 0; b < NWG; ++b) cnt[(desc[b][0] >> 2) & 7][role[b]]++;
  for (int b = 0; b < NWG; ++b) {
    const int d = (desc[b][0] >> 2) & 7, r = role[b];
    const int u = (2 * seen[d][r]++ + 1) * 500 / cnt[d][r];
    key[b] = r == ROLE_OWNER ? u * FO : r == ROLE_HELPER ? 1000 * FH + u * (FHE - FH) : 1000 * FN + u * (100 - FN);
    key[b] = key[b] * 256 + b;
  }
  for (int b = 0; b < NWG; ++b) {
    unsigned k = 0;
    for (int c = 0; c < NWG; ++c) k += ((desc[c][0] ^ desc[b][0]) & 0x1c) == 0 && key[c] < key[b];
    desc[b][1] |= k << 24;
  }
}

__device__ __forceinline__ unsigned gelu2(float y0, float y1) {
  const float k0 = -2.0f * 0.7978845608028654f * 1.4426950408889634f, k1 = k0 * 0.044715f;
  const f32x2 y = {y0, y1};
  const f32x2 z = y * __builtin_elementwise_fma(y * y, (f32x2){k1, k1}, (f32x2){k0, k0});
  const f32x2 d = (f32x2){__builtin_amdgcn_exp2f(z.x), __builtin_amdgcn_exp2f(z.y)} + 1.0f;
  const f32x2 g = y * (f32x2){__builtin_amdgcn_rcpf(d.x), __builtin_amdgcn_rcpf(d.y)};
  return __builtin_bit_cast(unsigned, __builtin_convertvector(g, bfx2));
}
__device__ __forceinline__ unsigned rfl(unsigned v) { return (unsigned)__builtin_amdgcn_readfirstlane((int)v); }
__device__ __forceinline__ unsigned now32() { return (unsigned)__builtin_amdgcn_s_memrealtime(); }
template <int N> __device__ __forceinline__ void wait_vm() { asm volatile("s_waitcnt vmcnt(%0)" :: "i"(N) : "memory"); }
__device__ __forceinline__ void st16_sys(char* p, u32x4 v) { asm volatile("global_store_dwordx4 %0, %1, off sc0 sc1\n\ts_nop 1" :: "v"(p), "v"(v) : "memory"); }
template <int OFF> __device__ __forceinline__ void st16_sys(char* sbase, unsigned voff, u32x4 v) {
  asm volatile("global_store_dwordx4 %0, %1, %2 offset:%3 sc0 sc1\n\ts_nop 1" :: "v"(voff), "v"(v), "s"(sbase), "i"(OFF) : "memory");
}
#define BAR() do { asm volatile("" ::: "memory"); __builtin_amdgcn_s_barrier(); asm volatile("" ::: "memory"); } while (0)
#define SCHED() __builtin_amdgcn_sched_barrier(0)
#define TS(k) do {} while (0)

__device__ __forceinline__ void wait_ge(const unsigned* flag, unsigned want, unsigned* tmo, unsigned call, bool active, char* mark, int lane) {
  const unsigned long long t0 = __builtin_amdgcn_s_memrealtime();
  for (unsigned n = 1;; ++n) {
    const bool ok = !active || __hip_atomic_load(flag, __ATOMIC_RELAXED, __HIP_MEMORY_SCOPE_SYSTEM) >= want;
    if (__all(ok)) return;
    if ((n & 15) == 0 &&
        (__hip_atomic_load(tmo, __ATOMIC_RELAXED, __HIP_MEMORY_SCOPE_AGENT) == call || __builtin_amdgcn_s_memrealtime() - t0 > DEADLINE)) {
      __hip_atomic_store(tmo, call, __ATOMIC_RELAXED, __HIP_MEMORY_SCOPE_AGENT);
      if (lane == 0) __hip_atomic_store((unsigned*)mark, 0x7fc07fc0u, __ATOMIC_RELAXED, __HIP_MEMORY_SCOPE_SYSTEM);
      return;
    }
    __builtin_amdgcn_s_sleep(1);
  }
}

__global__ void __launch_bounds__(512, 2) gemm_a2a_kernel(Params p) {
  extern __shared__ __attribute__((aligned(16))) char smem[];
  LAS char* lds = (LAS char*)smem;
  const int tid = threadIdx.x, wid = (int)rfl(tid >> 6), lane = tid & 63;
  const int b = blockIdx.x, rank = p.rank;
  const unsigned desc = p.desc[b][0], aux = p.desc[b][1];
  const Tile T0 = tile_of(desc, rank), T1 = tile_of(desc >> 16, rank);
  const bool owner = (aux >> 14) & 1, helper = (aux & 0x1fffu) != 0;
  const int n0 = helper ? HK : 0, n1 = owner ? NKT - HK : NKT, n2 = NKT;
  const int NJ = n0 + n1 + n2;
  const int J0 = n0 + n1 - NS;
  char* myws = p.ws[rank];
  unsigned* tmo = (unsigned*)(myws + WS_TMO);
  char* mark = p.out[rank] + (size_t)rank * MLOC * NSH * 2;
  const unsigned pidx = (aux >> 16) & 127;

  if (wid == 7) {
    const bool staged = T1.src != rank;
    if (!staged && !owner && !helper) return;
    const unsigned call = rfl(__hip_atomic_load((unsigned*)(myws + WS_DONE), __ATOMIC_RELAXED, __HIP_MEMORY_SCOPE_AGENT)) / NWG + 1;
    unsigned* pf = (unsigned*)(myws + WS_PFLAG + pidx * 128);
    BAR();
    int it = 0;
    if (helper) {
      for (; it <= HK + HDRAIN; ++it) BAR();
      if (lane == 0) __hip_atomic_fetch_add(pf, 1u, __ATOMIC_RELAXED, __HIP_MEMORY_SCOPE_AGENT);
    }
    if (owner) {
      for (; it < n1 - PRELD - 6; ++it) BAR();
      const unsigned seen = __hip_atomic_load(pf, __ATOMIC_RELAXED, __HIP_MEMORY_SCOPE_AGENT);
      for (; it < n1 - PRELD - 1; ++it) BAR();
      if (rfl(seen) < call) wait_ge(pf, call, tmo, call, true, mark, lane);
    }
    if (staged) {
      const unsigned* fl = (const unsigned*)(myws + WS_XCNT + T1.src * 512);
#pragma unroll 1
      for (int c = 0; c < 4; ++c) {
        for (; it < J0 + 16 * c - 5; ++it) BAR();
        const unsigned seen = __hip_atomic_load(fl + c * 32, __ATOMIC_RELAXED, __HIP_MEMORY_SCOPE_SYSTEM);
        for (; it < J0 + 16 * c; ++it) BAR();
        if (rfl(seen) < call) wait_ge(fl + c * 32, call, tmo, call, true, mark, lane);
        if (c == 0) TS(6);
      }
    }
  } else if (wid == 6) {
    u32x4 xv[4];
#pragma unroll
    for (int i = 0; i < 4; ++i) {
      const int P = b * 4 + i;
      xv[i] = *(const u32x4*)(p.x + 3 * XTILE + (size_t)((P & 15) * 8 + (lane >> 3)) * 8192 + (P >> 4) * 128 + ((lane & 7) ^ ((4 * (P & 1) + (lane >> 4)) & 7)) * 16);
    }
    const unsigned done = __hip_atomic_load((unsigned*)(myws + WS_DONE), __ATOMIC_RELAXED, __HIP_MEMORY_SCOPE_AGENT);
    const unsigned peer = (unsigned)(rank + 1 + lane) & 7;
    BAR();
    int it = 0;
    const int ch = b >> 6;
    const unsigned tn = now32() + XDELAY + (unsigned)b * TAU_X;
    while ((int)(now32() - tn) < 0 && it < 36 + 16 * ch) { BAR(); ++it; }
    __builtin_amdgcn_s_waitcnt(0x0F70);
    const unsigned call = rfl(done) / NWG + 1;
    char* pws = p.ws[0];
#pragma unroll
    for (int q = 1; q < NR; ++q) pws = peer == (unsigned)q ? p.ws[q] : pws;
    const size_t so = WS_XBUF + (call & 1) * XPAR + (size_t)rank * XTILE + (size_t)b * 4096 + lane * 16;
#pragma unroll
    for (int i = 0; i < 4; ++i)
#pragma unroll
      for (int q = 0; q < 7; ++q) st16_sys(p.ws[(rank + 1 + q) & 7] + so + i * 1024, xv[i]);
    TS(9);
    BAR(); BAR(); BAR(); BAR(); BAR();
    wait_vm<0>();
    unsigned tick; unsigned long long sv;
    asm volatile("s_mov_b64 %1, exec\n\ts_mov_b64 exec, 1\n\tglobal_atomic_add %0, %2, %3, off sc0 sc1\n\ts_mov_b64 exec, %1"
                 : "=&v"(tick), "=&s"(sv) : "v"(myws + WS_XLOC + ch * 128), "v"(1u) : "memory");
    BAR(); BAR(); BAR();
    it += 8;
    asm volatile("s_waitcnt vmcnt(0)" : "+v"(tick) :: "memory");
    if (rfl(tick) == call * 64 - 1) {
      if (lane < 7) __hip_atomic_store((unsigned*)(pws + WS_XCNT + rank * 512 + ch * 128), call, __ATOMIC_RELAXED, __HIP_MEMORY_SCOPE_SYSTEM);
    }
    if (lane == 0) __hip_atomic_fetch_add((unsigned*)(myws + WS_DONE), 1u, __ATOMIC_RELAXED, __HIP_MEMORY_SCOPE_AGENT);
    TS(5);
    if (b == NWG - 1) {
      for (; it < NJ; ++it) BAR();
      wait_ge((const unsigned*)(myws + WS_OUTCNT + peer * 128), call * 4 * (48 + (lane + 1 == 8 - (1 + Q4 % 7) ? 1 : 0)), tmo, call, lane < 7, mark, lane);
      TS(8);
    }
  } else if (wid >= 4) {
    const int pw = wid - 4;
    unsigned done0;
    asm volatile("global_load_dword %0, %1, off sc1" : "=v"(done0) : "v"(myws + WS_DONE) : "memory");
    const auto srdB = __builtin_amdgcn_make_buffer_rsrc((void*)p.w, 0, KDIM * NDIM * 2, 0x00020000);
    unsigned voffA[2], voffB[2];
#pragma unroll
    for (int i = 0; i < 2; ++i) {
      voffA[i] = (unsigned)(lane >> 3) * 8192u + (unsigned)(((lane & 7) ^ ((4 * i + (lane >> 4)) & 7)) * 16);
      voffB[i] = (unsigned)(lane >> 4) * 32768u + (unsigned)(((lane & 15) ^ (((lane >> 4) << 2) | (i << 1))) * 16);
    }
    unsigned sA = (unsigned)(64 * pw) * 8192u, stepK = 128u, stepI = 65536u;
    unsigned sB, G, kv, kp, kb;
    auto srdA = __builtin_amdgcn_make_buffer_rsrc((void*)p.x, 0, (int)XTILE, 0x00020000);
    auto begin = [&](unsigned d, unsigned k0) {
      srdA = __builtin_amdgcn_make_buffer_rsrc((void*)(p.x + (size_t)(d & 3) * XTILE), 0, (int)XTILE, 0x00020000);
      sB = (unsigned)(32 * pw) * 32768u + ((((unsigned)rank + ((d >> 2) & 7)) & 7) * 16 + ((d >> 5) & 15)) * 256u;
      G = ((d >> 9) & 3) + 1; kv = (d >> 11) & 3; kp = 0; kb = k0;
    };
    unsigned soA, soB, ldsb = (unsigned)pw * 8192u;
    auto next = [&]() {
      const unsigned kt = min(kb + kv, 63u);
      soA = sA + kt * stepK; soB = sB + kt * (64u * 32768u);
      kv = kv + 1 == G ? 0 : kv + 1; ++kp;
      if (kp == G) { kp = 0; kb += G; }
    };
    auto issue = [&]() {
      LAS char* d = lds + ldsb;
#pragma unroll
      for (int i = 0; i < 8; ++i) __builtin_amdgcn_raw_ptr_buffer_load_lds(srdA, (LAS void*)(d + i * 1024), 16, voffA[i & 1], soA + i * stepI, 0, 17);
#pragma unroll
      for (int i = 0; i < 8; ++i) __builtin_amdgcn_raw_ptr_buffer_load_lds(srdB, (LAS void*)(d + NS * SLOT + i * 1024), 16, voffB[(i >> 1) & 1], soB + i * (4u * 32768u), 0, 0);
      ldsb = ldsb + SLOT >= NS * SLOT ? ldsb + SLOT - NS * SLOT : ldsb + SLOT;
    };
    static_assert(HK == 0 || HK > NS, "the prologue stays inside the first piece");
    begin(helper ? aux : desc, NKT - n1);
    next(); issue(); next(); issue();
    asm volatile("s_waitcnt vmcnt(16)" : "+v"(done0) :: "memory");
    BAR();
#pragma unroll
    for (int i = 2; i < NS; ++i) { next(); issue(); }
    next();
    unsigned tpace = now32();
    const bool crit = owner || T1.dest != rank;
    const unsigned pace1 = helper ? PH1 : crit ? PO1 : PN1, pace2 = helper ? PH2 : crit ? PO2 : PN2;
    for (int j = 0; j < NJ - NS; ++j) {
      tpace += j < NKT ? pace1 : pace2;
      while ((int)(now32() - tpace) < 0) __builtin_amdgcn_s_sleep(1);
      wait_vm<48>(); BAR();
      issue();
      SCHED();
      if (j == n0 - NS - 1) begin(desc, 0);
      if (j == J0 - 1) {
        begin(desc >> 16, 0);
        if (T1.src != rank) {
          srdA = __builtin_amdgcn_make_buffer_rsrc((void*)(myws + WS_XBUF + ((rfl(done0) / NWG + 1) & 1) * XPAR + (size_t)T1.src * XTILE), 0, (int)XTILE, 0x00020000);
          sA = (unsigned)pw * 8192u; stepK = 16384u; stepI = 1024u; voffA[0] = voffA[1] = (unsigned)lane * 16u;
        }
      }
      next();
    }
    wait_vm<48>(); BAR();
    wait_vm<32>(); BAR();
    wait_vm<16>(); BAR();
    wait_vm<0>(); BAR();
    BAR();
  } else {
    const int wr = wid >> 1, wc = wid & 1, fr = lane & 15, fq = lane >> 4;
    const unsigned baseA0 = (unsigned)(wr * 64 + fr) * 128u + (unsigned)((fq ^ ((fr >> 1) & 7)) * 16);
    unsigned baseB[4];
    { const int q = (lane & 15) >> 2, pp = lane & 3; const int s = (q << 2) | ((fq & 1) << 1);
#pragma unroll
      for (int n = 0; n < 4; ++n) baseB[n] = (unsigned)(NS * SLOT) + (unsigned)(8 * fq + q) * 256u + (unsigned)(((8 * wc + 2 * n) ^ s) * 16 + 8 * pp); }
    f32x4 acc[4][4];
    bf16x8 FA[2][4], FB[2][4];
    u32x4 O[8];
    typedef LAS char* lptr;
    auto rd = [&](bf16x8 (&A)[4], bf16x8 (&B)[4], lptr pa, lptr (&pb)[4], int ks) {
      auto rb = [&](int n) {
        bf16x4 lo = __builtin_amdgcn_ds_read_tr16_b64_v4i16((LAS bf16x4*)(pb[n] + ks * 8192));
        bf16x4 hi = __builtin_amdgcn_ds_read_tr16_b64_v4i16((LAS bf16x4*)(pb[n] + ks * 8192 + 1024));
        B[n] = __builtin_shufflevector(lo, hi, 0, 1, 2, 3, 4, 5, 6, 7);
      };
      rb(0);
#pragma unroll
      for (int m = 0; m < 4; ++m) A[m] = *(const LAS bf16x8*)(pa + m * 2048);
      rb(1); rb(2); rb(3);
    };
    auto mm = [&](bf16x8 (&A)[4], bf16x8 (&B)[4], int lo, int hi) {
#pragma unroll
      for (int i = lo; i < hi; ++i) {
        const int n = i >> 2, m = (n & 1) ? 3 - (i & 3) : (i & 3);
        acc[m][n] = __builtin_amdgcn_mfma_f32_16x16x32_bf16(B[n], A[m], acc[m][n], 0, 0, 0);
      }
    };
    auto pairs = [&](int n) {
#pragma unroll
      for (int i = 0; i < n; ++i) { __builtin_amdgcn_sched_group_barrier(0x8, 1, 0); __builtin_amdgcn_sched_group_barrier(0x100, 1, 0); }
    };
    auto out_ptr = [&](const Tile& T) {
      char* o = p.out[T.dest] + (size_t)(T.src * MLOC + T.rt * 128 + wr * 64) * (NSH * 2) + (T.ct * 128 + wc * 64) * 2;
      asm volatile("" : "+s"(o));
      return o;
    };
    char* const o0 = out_ptr(T0); char* const o1 = out_ptr(T1);
    unsigned* cnt0 = (unsigned*)(p.ws[T0.dest] + WS_OUTCNT + rank * 128), * cnt1 = (unsigned*)(p.ws[T1.dest] + WS_OUTCNT + rank * 128);
    asm volatile("" : "+s"(cnt0), "+s"(cnt1));
    const unsigned vo = (unsigned)fr * (NSH * 2) + 16u * ((fq >> 1) + 2 * (fq & 1));
    auto pack1 = [&](int m, int q) {
      const unsigned x0 = gelu2(acc[m][2 * q][0], acc[m][2 * q][1]), x1 = gelu2(acc[m][2 * q][2], acc[m][2 * q][3]);
      const unsigned y0 = gelu2(acc[m][2 * q + 1][0], acc[m][2 * q + 1][1]), y1 = gelu2(acc[m][2 * q + 1][2], acc[m][2 * q + 1][3]);
      const auto r0 = __builtin_amdgcn_permlane16_swap(x0, y0, false, false);
      const auto r1 = __builtin_amdgcn_permlane16_swap(x1, y1, false, false);
      return (u32x4){r0[0], r1[0], r0[1], r1[1]};
    };
    auto signal = [&](unsigned* cnt) {
      wait_vm<0>();
      if (lane == 0) __hip_atomic_fetch_add(cnt, 1u, __ATOMIC_RELAXED, __HIP_MEMORY_SCOPE_SYSTEM);
    };
    unsigned so = 0;
    lptr pB[4], pA0 = lds + baseA0, pA1 = lds + (baseA0 ^ 64u);
#pragma unroll
    for (int n = 0; n < 4; ++n) pB[n] = lds + baseB[n];
#define PART1() do { SCHED(); rd(FA[1], FB[1], pA1, pB, 1); mm(FA[0], FB[0], 0, 16); pairs(12); __builtin_amdgcn_sched_group_barrier(0x8, 4, 0); SCHED(); } while (0)
#define PART2(WAIT) do { \
      so = so + SLOT == NS * SLOT ? 0 : so + SLOT; \
      pA0 = lds + baseA0 + so; pA1 = lds + (baseA0 ^ 64u) + so; \
      _Pragma("unroll") for (int n = 0; n < 4; ++n) pB[n] = lds + baseB[n] + so; \
      mm(FA[1], FB[1], 0, 4); \
      asm volatile("" : "+v"(pA0), "+v"(pA1), "+v"(pB[0]), "+v"(pB[1]), "+v"(pB[2]), "+v"(pB[3])); \
      _Pragma("unroll") for (int i = 0; i < 3; ++i) { __builtin_amdgcn_sched_group_barrier(0x8, 1, 0); __builtin_amdgcn_sched_group_barrier(0x2, 2, 0); } \
      __builtin_amdgcn_sched_group_barrier(0x8, 1, 0); \
      SCHED(); __builtin_amdgcn_s_waitcnt(0xC07F); WAIT; __builtin_amdgcn_s_barrier(); SCHED(); \
      rd(FA[0], FB[0], pA0, pB, 0); mm(FA[1], FB[1], 4, 16); pairs(12); SCHED(); } while (0)
    char* part = myws + WS_PART + (size_t)pidx * PART + wid * 16384;
    if (wid == 0) TS(0);
    BAR();
    if (wid == 0) TS(1);
    rd(FA[0], FB[0], pA0, pB, 0);
    f32x4 H[16];
    int hk = 99;
#pragma unroll 1
    for (int seg = helper ? 0 : 1; seg < 3; ++seg) {
      if (seg < 2) {
#pragma unroll
        for (int m = 0; m < 4; ++m)
#pragma unroll
          for (int n = 0; n < 4; ++n) acc[m][n] = (f32x4){0.f, 0.f, 0.f, 0.f};
      }
      const int nk = seg == 0 ? HK : seg == 1 ? n1 - PRELD : PRELD;
#pragma unroll 1
      for (int kt = 0; kt < nk; ++kt) {
        PART1();
        if (hk <= HDRAIN) {
#define HST(s) case s: st16_sys<1024>(part + (14 - 2 * s) * 1024, (unsigned)lane * 16u, __builtin_bit_cast(u32x4, H[15 - 2 * s])); \
                       st16_sys<0>(part + (14 - 2 * s) * 1024, (unsigned)lane * 16u, __builtin_bit_cast(u32x4, H[14 - 2 * s])); break;
          switch (hk) { HST(0) HST(1) HST(2) HST(3) HST(4) HST(5) HST(6) HST(7) case HDRAIN: wait_vm<0>(); break; default: break; }
          ++hk;
        }
        PART2((void)0);
      }
      if (seg == 0) {
#pragma unroll
        for (int i = 0; i < 16; ++i) H[i] = acc[i >> 2][i & 3];
        hk = 0;
      } else if (seg == 1) {
        const auto srdP = __builtin_amdgcn_make_buffer_rsrc((void*)part, 0, owner ? 16384 : 0, 0x00020000);
#pragma unroll
        for (int i = 0; i < 16; ++i) H[i] = __builtin_bit_cast(f32x4, __builtin_amdgcn_raw_buffer_load_b128(srdP, lane * 16 + (i & 3) * 1024, (i >> 2) * 4096, 16));
      }
    }
    if (wid == 0) TS(2);
    const unsigned tend = now32() + 120;
#pragma unroll
    for (int i = 0; i < 16; ++i) acc[i >> 2][i & 3] += H[i];
#pragma unroll
    for (int i = 0; i < 8; ++i) O[i] = pack1(i >> 1, i & 1);
#pragma unroll
    for (int m = 0; m < 4; ++m)
#pragma unroll
      for (int n = 0; n < 4; ++n) acc[m][n] = (f32x4){0.f, 0.f, 0.f, 0.f};
    auto store0 = [&](int m) {
      switch (m) {
        case 0: st16_sys<0>(o0, vo, O[0]); st16_sys<64>(o0, vo, O[1]); break;
        case 1: st16_sys<0>(o0 + 65536, vo, O[2]); st16_sys<64>(o0 + 65536, vo, O[3]); break;
        case 2: st16_sys<0>(o0 + 2 * 65536, vo, O[4]); st16_sys<64>(o0 + 2 * 65536, vo, O[5]); break;
        default: st16_sys<0>(o0 + 3 * 65536, vo, O[6]); st16_sys<64>(o0 + 3 * 65536, vo, O[7]); break;
      }
    };
    unsigned tn = tend + (aux >> 24) * TAU_O + ((desc >> 2) & 7) * (TAU_O / 8) - (helper ? HOFF : owner ? 0 : NOFF);
    unsigned tnow = tn - 1;
    int nb = 0, kl = 0;
    __builtin_amdgcn_s_waitcnt(0x0070);
#pragma unroll 1
    for (int kt = 0; kt < n2; ++kt) {
      unsigned long long t64;
      asm volatile("s_memrealtime %0" : "=s"(t64));
      PART1();
      if ((int)(tnow - tn) >= 0) { store0(nb); kl = kt; if (++nb == 4) tn += 1u << 30; }
      PART2(asm volatile("" : "+s"(t64)));
      tnow = (unsigned)t64;
    }
    if (wid == 0) TS(3);
    const bool late = nb < 4 || kl > n2 - 5;
    for (; nb < 4; ++nb) store0(nb);
    if (!late) signal(cnt0);
#pragma unroll
    for (int m = 0; m < 4; ++m) {
      st16_sys<0>(o1 + m * 65536, vo, pack1(m, 0)); SCHED();
      st16_sys<64>(o1 + m * 65536, vo, pack1(m, 1)); SCHED();
    }
    if (late) signal(cnt0);
    if (T1.dest != rank) signal(cnt1);
  }
}

extern "C" void kernel_launch(int rank, int nranks, ncclComm_t comm, hipStream_t stream,
                              void* const* d_in, const int* in_sizes, int n_in,
                              void* d_out, int out_size, const int* in_dtypes, int out_dtype,
                              void* const* d_out_all, void* const* d_ws_all, size_t ws_bytes) {
  if (nranks != NR || ws_bytes < WS_END || (unsigned)rank >= (unsigned)NR) { fprintf(stderr, "unsupported configuration\n"); return; }
  HIP_CHECK(hipFuncSetAttribute((const void*)gemm_a2a_kernel, hipFuncAttributeMaxDynamicSharedMemorySize, LDS_BYTES));
  Params p;
  p.x = (const char*)d_in[0]; p.w = (const char*)d_in[1]; p.rank = rank; p.pad = 0;
  for (int i = 0; i < NR; ++i) { p.out[i] = (char*)d_out_all[i]; p.ws[i] = (char*)d_ws_all[i]; }
  build_desc(p.desc);
  gemm_a2a_kernel<<<NWG, 512, LDS_BYTES, stream>>>(p);
  HIP_CHECK(hipGetLastError());
}
```

```cpp
#include <hip/hip_runtime.h>
#include <rccl/rccl.h>
#include <cstdio>

#define HIP_CHECK(expr) do { hipError_t _e = (expr); if (_e != hipSuccess) { \
    fprintf(stderr, "HIP error %s at %s:%d\n", hipGetErrorString(_e), __FILE__, __LINE__); \
    return; } } while (0)

#define LAS __attribute__((address_space(3)))
typedef short bf16x8 __attribute__((ext_vector_type(8)));
typedef short bf16x4 __attribute__((ext_vector_type(4)));
typedef float f32x4 __attribute__((ext_vector_type(4)));
typedef float f32x2 __attribute__((ext_vector_type(2)));
typedef unsigned u32x4 __attribute__((ext_vector_type(4)));
typedef __bf16 bfx2 __attribute__((ext_vector_type(2)));

#define TAU_O 68
#define TAU_X 8
#define PRW 12
#define HK 12
#define OPLAIN 1
#define ST0 st16_l2
#define XDELAY 100
#define PACE1 51
#define PACE2 44
constexpr int NR = 8;
constexpr int KDIM = 4096, NDIM = 16384, MLOC = 512, NSH = 2048;
constexpr int NKT = 64, SLOT = 16384, NS = 5, NWG = 256;
constexpr int LDS_BYTES = 2 * NS * SLOT;
#define WBW 4
#define TTHR 1
#define XPERM 1
#define INTL 7
#define ODRAIN 5
constexpr int HDRAIN = 12, PRELD = 3;
constexpr int Q4 = 80;
constexpr size_t WS_DONE = 0, WS_TMO = 128, WS_OUTCNT = 4096, WS_XCNT = 8192, WS_O0CNT = 12288, WS_XLOC = 14336, WS_CEN = 16384, WS_ARR = 20480, WS_ACK = 24576, WS_PFLAG = 32768, WS_STAMP = 65536;
constexpr size_t XTILE = 128 * KDIM * 2;
constexpr size_t WS_XBUF = 1u << 20, XPAR = NR * XTILE;
constexpr size_t WS_PART = WS_XBUF + 2 * XPAR, PART = 128 * 128 * 4, WS_END = WS_PART + 96 * PART;
constexpr unsigned long long DEADLINE = 40000000ull;

enum { ROLE_OWNER = 1, ROLE_HELPER = 2 };
struct Params { const char* x; const char* w; char* out[NR]; char* ws[NR]; int rank; int pad; unsigned desc[NWG][2]; };
struct Tile { int src; int rt; int dest; int ct; };
__device__ __forceinline__ Tile tile_of(unsigned d, int rank) {
  Tile r; r.src = (rank + (int)((d >> 13) & 7)) & 7; r.rt = d & 3; r.dest = (rank + (int)((d >> 2) & 7)) & 7; r.ct = (d >> 5) & 15; return r;
}
static inline unsigned tile_code(int so, int rt, int dof, int ct, int G, int rot) { return (unsigned)(rt | dof << 2 | ct << 5 | (G - 1) << 9 | (rot % G) << 11 | so << 13); }
static void build_desc(unsigned (*out)[2]) {
  unsigned desc[NWG][2];
  unsigned T[2][8][32]; int n[2][8] = {};
  auto put = [&](int w, int x, unsigned c) { T[w][x][n[w][x]++] = c; };
  auto code = [&](int q, int rt) { return tile_code(0, rt, 1 + q % 7, q / 7, q == Q4 ? 4 : 3, rt + q); };
  for (int k = 0; k < 80; ++k) for (int rt = 0; rt < 3; ++rt) put(0, k / 10, code((INTL & 1) ? k / 10 + 8 * (k % 10) : k, rt));
  for (int i = 0; i < 16; ++i) put(0, i / 2, code(i < 12 ? 81 + i / 3 : Q4, i < 12 ? i % 3 : i - 12));
  for (int k = 0; k < 24; ++k) for (int rt = 0; rt < 3; ++rt) put(1, k / 6, code(85 + ((INTL & 2) ? k / 6 + 4 * (k % 6) : k), rt));
  for (int x = 0; x < 4; ++x) for (int c = 0; c < 4; ++c) for (int r = 0; r < 3; ++r) put(1, x, tile_code(0, r, 0, (INTL & 4) ? x + 4 * c : 4 * x + c, 3, r + c));
  for (int k = 0; k < 3; ++k) { put(1, k, code(109 + k, 0)); put(1, k, code(109 + k, 1)); }
  put(1, 3, code(109, 2)); put(1, 3, code(110, 2));
  for (int g = 0; g < 2; ++g) for (int h = 0; h < 2; ++h) {
    for (int c = 0; c < 8; ++c) for (int r = 0; r < 4; ++r)
      if (!(4 * g + r == 8 - (1 + Q4 % 7) && 8 * h + c == Q4 / 7)) put(1, 4 + 2 * g + h, tile_code(4 * g + r, 3, 0, 8 * h + c, 4, r + c + 2 * h + g));
    if (n[1][4 + 2 * g + h] < 32) put(1, 4 + 2 * g + h, code(111, 2));
  }
  int role[NWG], own[96], nown = 0;
  for (int b = 0; b < NWG; ++b) { desc[b][0] = T[0][b & 7][b >> 3] | T[1][b & 7][b >> 3] << 16; desc[b][1] = 0; role[b] = 0; }
  for (int pass = 0; pass < 2; ++pass) for (int x = 0; x < 4; ++x) for (int i = pass ? 30 : 0; i < (pass ? 32 : 18); ++i) own[nown++] = i * 8 + x;
  for (int k = 0; HK && k < nown; ++k) {
    const int o = own[k], h = (k % 30) * 8 + 4 + k / 30;
    desc[o][1] = (unsigned)k << 16 | 1u << 14; role[o] = ROLE_OWNER;
    desc[h][1] = (desc[o][0] & 0x1fff) | (unsigned)k << 16; role[h] = ROLE_HELPER;
  }
  auto sched = [](int j) { return j < NKT ? j * PACE1 : NKT * PACE1 + (j - NKT) * PACE2; };
  auto turn_kt = [&](int k, int d, int r) {
    const int t = sched(NKT - HK) + k * TAU_O + d * (TAU_O / 8), j1 = r == ROLE_OWNER ? NKT - HK : r == ROLE_HELPER ? NKT + HK : NKT;
    if (t < sched(j1)) return -1;
    int j = j1; while (sched(j) < t) ++j;
    return j - j1 < NKT - 12 ? j - j1 : NKT - 12;
  };
  bool sent[NWG] = {};
  int load[64][8] = {};
  for (int k = 0, left = NWG; left; ++k)
    for (int dd = 0; dd < 7; ++dd) {
      const int d = 1 + (dd + k) % 7;
      int best = -1, bc = 1 << 30;
      for (int b = 0; b < NWG; ++b) {
        if (sent[b] || (int)((desc[b][0] >> 2) & 7) != d) continue;
        if (turn_kt(k, d, role[b]) < 0) continue;
        const int x = b & 7, c = 8 * (2 * load[k][x] + (k > 0 ? load[k - 1][x] : 0) + (k > 1 ? load[k - 2][x] : 0)) + PRW * (role[b] == ROLE_OWNER ? 0 : role[b] == ROLE_HELPER ? 1 : 2);
        if (c < bc) { bc = c; best = b; }
      }
      if (best >= 0) { sent[best] = true; --left; load[k][best & 7]++; desc[best][1] |= (unsigned)turn_kt(k, d, role[best]) << 24; }
    }
  for (int b = 0; b < NWG; ++b) { const int l = XPERM ? (b & ~7) | (b & 1) << 2 | (b & 7) >> 1 : b; out[b][0] = desc[l][0]; out[b][1] = desc[l][1]; }
}

__device__ __forceinline__ unsigned gelu2(float y0, float y1) {
  const float k0 = -2.0f * 0.7978845608028654f * 1.4426950408889634f, k1 = k0 * 0.044715f;
  const f32x2 y = {y0, y1};
  const f32x2 z = y * __builtin_elementwise_fma(y * y, (f32x2){k1, k1}, (f32x2){k0, k0});
  const f32x2 d = (f32x2){__builtin_amdgcn_exp2f(z.x), __builtin_amdgcn_exp2f(z.y)} + 1.0f;
  const f32x2 g = y * (f32x2){__builtin_amdgcn_rcpf(d.x), __builtin_amdgcn_rcpf(d.y)};
  return __builtin_bit_cast(unsigned, __builtin_convertvector(g, bfx2));
}
__device__ __forceinline__ unsigned rfl(unsigned v) { return (unsigned)__builtin_amdgcn_readfirstlane((int)v); }
__device__ __forceinline__ unsigned now32() { return (unsigned)__builtin_amdgcn_s_memrealtime(); }
template <int N> __device__ __forceinline__ void wait_vm() { asm volatile("s_waitcnt vmcnt(%0)" :: "i"(N) : "memory"); }
__device__ __forceinline__ void st16_sys(char* p, u32x4 v) { asm volatile("global_store_dwordx4 %0, %1, off sc0 sc1\n\ts_nop 1" :: "v"(p), "v"(v) : "memory"); }
template <int OFF> __device__ __forceinline__ void st16_sys(char* sbase, unsigned voff, u32x4 v) {
  asm volatile("global_store_dwordx4 %0, %1, %2 offset:%3 sc0 sc1\n\ts_nop 1" :: "v"(voff), "v"(v), "s"(sbase), "i"(OFF) : "memory");
}
__device__ __forceinline__ void st16_l2(char* sbase, unsigned voff, u32x4 v) {
  asm volatile("global_store_dwordx4 %0, %1, %2\n\ts_nop 1" :: "v"(voff), "v"(v), "s"(sbase) : "memory");
}
#define BAR() do { asm volatile("" ::: "memory"); __builtin_amdgcn_s_barrier(); asm volatile("" ::: "memory"); } while (0)
#define SCHED() __builtin_amdgcn_sched_barrier(0)
#define TS(k) do {} while (0)

__device__ __forceinline__ void wait_ge(const unsigned* flag, unsigned want, unsigned* tmo, unsigned call, bool active, char* mark, int lane) {
  const unsigned long long t0 = __builtin_amdgcn_s_memrealtime();
  for (unsigned n = 1;; ++n) {
    const bool ok = !active || __hip_atomic_load(flag, __ATOMIC_RELAXED, __HIP_MEMORY_SCOPE_SYSTEM) >= want;
    if (__all(ok)) return;
    if ((n & 15) == 0 &&
        (__hip_atomic_load(tmo, __ATOMIC_RELAXED, __HIP_MEMORY_SCOPE_AGENT) == call || __builtin_amdgcn_s_memrealtime() - t0 > DEADLINE)) {
      __hip_atomic_store(tmo, call, __ATOMIC_RELAXED, __HIP_MEMORY_SCOPE_AGENT);
      if (lane == 0) __hip_atomic_store((unsigned*)mark, 0x7fc07fc0u, __ATOMIC_RELAXED, __HIP_MEMORY_SCOPE_SYSTEM);
      return;
    }
    __builtin_amdgcn_s_sleep(1);
  }
}

__global__ void __launch_bounds__(512, 2) gemm_a2a_kernel(Params p) {
  extern __shared__ __attribute__((aligned(16))) char smem[];
  LAS char* lds = (LAS char*)smem;
  const int tid = threadIdx.x, wid = (int)rfl(tid >> 6), lane = tid & 63;
  const int b = blockIdx.x, rank = p.rank;
  const unsigned desc = p.desc[b][0], aux = p.desc[b][1];
  const Tile T0 = tile_of(desc, rank), T1 = tile_of(desc >> 16, rank);
  const bool owner = (aux >> 14) & 1, helper = (aux & 0x1fffu) != 0;
  const int n0 = helper ? HK : 0, n1 = owner ? NKT - HK : NKT, n2 = NKT;
  const int NJ = n0 + n1 + n2;
  const int J0 = n0 + n1 - NS;
  char* myws = p.ws[rank];
  unsigned* tmo = (unsigned*)(myws + WS_TMO);
  char* mark = p.out[rank] + (size_t)rank * MLOC * NSH * 2;
  const unsigned pidx = (aux >> 16) & 127;
  const int kb0 = (int)(aux >> 24);

  if (wid == 7) {
    const bool staged = T1.src != rank;
    if (!staged && !owner && !helper) return;
    const unsigned call = rfl(__hip_atomic_load((unsigned*)(myws + WS_DONE), __ATOMIC_RELAXED, __HIP_MEMORY_SCOPE_AGENT)) / NWG + 1;
    unsigned* pf = (unsigned*)(myws + WS_PFLAG + pidx * 128);
    BAR();
    int it = 0;
    if (helper) {
      for (; it <= HK + HDRAIN; ++it) BAR();
      if (lane == 0) __hip_atomic_fetch_add(pf, 1u, __ATOMIC_RELAXED, __HIP_MEMORY_SCOPE_AGENT);
    }
    if (owner) {
      for (; it < n1 - PRELD - 6; ++it) BAR();
      const unsigned seen = __hip_atomic_load(pf, __ATOMIC_RELAXED, __HIP_MEMORY_SCOPE_AGENT);
      for (; it < n1 - PRELD - 1; ++it) BAR();
      if (rfl(seen) < call) wait_ge(pf, call, tmo, call, true, mark, lane);
    }
    if (staged) {
      const unsigned* fl = (const unsigned*)(myws + WS_XCNT + T1.src * 512);
#pragma unroll 1
      for (int c = 0; c < 4; ++c) {
        for (; it < J0 + 16 * c - 5; ++it) BAR();
        const unsigned seen = __hip_atomic_load(fl + c * 32, __ATOMIC_RELAXED, __HIP_MEMORY_SCOPE_SYSTEM);
        for (; it < J0 + 16 * c; ++it) BAR();
        if (rfl(seen) < call) wait_ge(fl + c * 32, call, tmo, call, true, mark, lane);
        if (c == 0) TS(6);
      }
    }
  } else if (wid == 6) {
    u32x4 xv[4];
#pragma unroll
    for (int i = 0; i < 4; ++i) {
      const int P = b * 4 + i;
      xv[i] = *(const u32x4*)(p.x + 3 * XTILE + (size_t)((P & 15) * 8 + (lane >> 3)) * 8192 + (P >> 4) * 128 + ((lane & 7) ^ ((4 * (P & 1) + (lane >> 4)) & 7)) * 16);
    }
    const unsigned done = __hip_atomic_load((unsigned*)(myws + WS_DONE), __ATOMIC_RELAXED, __HIP_MEMORY_SCOPE_AGENT);
    const unsigned xcc = (unsigned)__builtin_amdgcn_s_getreg((3 << 11) | 20) & 15u;
    if (lane == 0) __hip_atomic_fetch_add((unsigned*)(myws + WS_CEN + xcc * 128), 1u, __ATOMIC_RELAXED, __HIP_MEMORY_SCOPE_AGENT);
    const unsigned peer = (unsigned)(rank + 1 + lane) & 7;
    BAR();
    int it = 0;
    auto bars = [&](int n) { for (int e = min(it + n, NJ); it < e; ++it) BAR(); };
    const int ch = b >> 6;
    const unsigned tn = now32() + XDELAY + (unsigned)b * TAU_X;
    while ((int)(now32() - tn) < 0 && it < 36 + 16 * ch) { BAR(); ++it; }
    __builtin_amdgcn_s_waitcnt(0x0F70);
    const unsigned call = rfl(done) / NWG + 1;
    char* pws = p.ws[0];
#pragma unroll
    for (int q = 1; q < NR; ++q) pws = peer == (unsigned)q ? p.ws[q] : pws;
    const size_t so = WS_XBUF + (call & 1) * XPAR + (size_t)rank * XTILE + (size_t)b * 4096;
#pragma unroll
    for (int i = 0; i < 4; ++i)
#pragma unroll
      for (int q = 0; q < 7; ++q) st16_sys<0>(p.ws[(rank + 1 + q) & 7] + so + i * 1024, (unsigned)lane * 16u, xv[i]);
    TS(9);
    bars(5);
    wait_vm<0>();
    unsigned tick; unsigned long long sv;
#define ATOM0(OP, ADDR, VAL) asm volatile("s_mov_b64 %1, exec\n\ts_mov_b64 exec, 1\n\t" OP " %0, %2, %3, off sc0 sc1\n\ts_mov_b64 exec, %1" \
                                          : "=&v"(tick), "=&s"(sv) : "v"(ADDR), "v"(VAL) : "memory")
    ATOM0("global_atomic_add", myws + WS_XLOC + ch * 128, 1u);
    bars(3);
    asm volatile("s_waitcnt vmcnt(0)" : "+v"(tick) :: "memory");
    if (rfl(tick) == call * 64 - 1) {
      if (lane < 7) __hip_atomic_store((unsigned*)(pws + WS_XCNT + rank * 512 + ch * 128), call, __ATOMIC_RELAXED, __HIP_MEMORY_SCOPE_SYSTEM);
    }
    if (lane == 0) __hip_atomic_fetch_add((unsigned*)(myws + WS_DONE), 1u, __ATOMIC_RELAXED, __HIP_MEMORY_SCOPE_AGENT);
    TS(5);
    bars(n0 + n1 + kb0 + ODRAIN + 1 - it);
    ATOM0("global_atomic_add", myws + WS_ARR + xcc * 128, 1u);
    const unsigned cen = __hip_atomic_load((unsigned*)(myws + WS_CEN + (lane & 15) * 128), __ATOMIC_RELAXED, __HIP_MEMORY_SCOPE_AGENT);
    bars(2);
    asm volatile("s_waitcnt vmcnt(0)" : "+v"(tick) :: "memory");
    unsigned total = 0;
#pragma unroll
    for (int k = 0; k < 16; ++k) total += (unsigned)__builtin_amdgcn_readlane((int)cen, k);
    const unsigned t = rfl(tick) + 1;
    if (total != call * NWG || t == (unsigned)__builtin_amdgcn_readlane((int)cen, (int)xcc)) {
      asm volatile("buffer_wbl2 sc0 sc1" ::: "memory");
      bars(WBW);
      wait_vm<0>();
      ATOM0("global_atomic_umax", myws + WS_ACK + xcc * 128, t);
      bars(2);
      asm volatile("s_waitcnt vmcnt(0)" : "+v"(tick) :: "memory");
      const unsigned old = rfl(tick);
      if (t > old && lane < 7) __hip_atomic_fetch_add((unsigned*)(pws + WS_O0CNT + rank * 128), t - old, __ATOMIC_RELAXED, __HIP_MEMORY_SCOPE_SYSTEM);
    }
    TS(7);
    if (b == NWG - 1) {
      bars(NJ);
      const unsigned d = 7 - (lane & 7), n2nd = 3 * ((111 - (d - 1)) / 7 - (84 - (d - 1)) / 7);
      wait_ge((const unsigned*)(myws + (lane < 8 ? WS_OUTCNT : WS_O0CNT) + peer * 128), lane < 8 ? call * 4 * n2nd : call * NWG, tmo, call, lane < 15 && lane != 7, mark, lane);
      TS(8);
    }
  } else if (wid >= 4) {
    const int pw = wid - 4;
    unsigned done0;
    asm volatile("global_load_dword %0, %1, off sc1" : "=v"(done0) : "v"(myws + WS_DONE) : "memory");
    const auto srdB = __builtin_amdgcn_make_buffer_rsrc((void*)p.w, 0, KDIM * NDIM * 2, 0x00020000);
    unsigned voffA[2], voffB[2];
#pragma unroll
    for (int i = 0; i < 2; ++i) {
      voffA[i] = (unsigned)(lane >> 3) * 8192u + (unsigned)(((lane & 7) ^ ((4 * i + (lane >> 4)) & 7)) * 16);
      voffB[i] = (unsigned)(lane >> 4) * 32768u + (unsigned)(((lane & 15) ^ (((lane >> 4) << 2) | (i << 1))) * 16);
    }
    unsigned sA = (unsigned)(64 * pw) * 8192u, stepK = 128u, stepI = 65536u;
    unsigned sB, G, kv, kp, kb;
    auto srdA = __builtin_amdgcn_make_buffer_rsrc((void*)p.x, 0, (int)XTILE, 0x00020000);
    auto begin = [&](unsigned d, unsigned k0) {
      srdA = __builtin_amdgcn_make_buffer_rsrc((void*)(p.x + (size_t)(d & 3) * XTILE), 0, (int)XTILE, 0x00020000);
      sB = (unsigned)(32 * pw) * 32768u + ((((unsigned)rank + ((d >> 2) & 7)) & 7) * 16 + ((d >> 5) & 15)) * 256u;
      G = ((d >> 9) & 3) + 1; kv = (d >> 11) & 3; kp = 0; kb = k0;
    };
    unsigned soA, soB, ldsb = (unsigned)pw * 8192u;
    auto next = [&]() {
      const unsigned kt = min(kb + kv, 63u);
      soA = sA + kt * stepK; soB = sB + kt * (64u * 32768u);
      kv = kv + 1 == G ? 0 : kv + 1; ++kp;
      if (kp == G) { kp = 0; kb += G; }
    };
    auto issue = [&]() {
      LAS char* d = lds + ldsb;
#pragma unroll
      for (int i = 0; i < 8; ++i) __builtin_amdgcn_raw_ptr_buffer_load_lds(srdA, (LAS void*)(d + i * 1024), 16, voffA[i & 1], soA + i * stepI, 0, 17);
#pragma unroll
      for (int i = 0; i < 8; ++i) __builtin_amdgcn_raw_ptr_buffer_load_lds(srdB, (LAS void*)(d + NS * SLOT + i * 1024), 16, voffB[(i >> 1) & 1], soB + i * (4u * 32768u), 0, 0);
      ldsb = ldsb + SLOT >= NS * SLOT ? ldsb + SLOT - NS * SLOT : ldsb + SLOT;
    };
    static_assert(HK == 0 || HK > NS, "the prologue stays inside the first piece");
    begin(helper ? aux : desc, NKT - n1);
    next(); issue(); next(); issue();
    asm volatile("s_waitcnt vmcnt(16)" : "+v"(done0) :: "memory");
    BAR();
#pragma unroll
    for (int i = 2; i < NS; ++i) { next(); issue(); }
    next();
    unsigned tpace = now32();
    for (int j = 0; j < NJ - NS; ++j) {
      tpace += j < NKT ? PACE1 : PACE2;
      while ((int)(now32() - tpace) < 0) __builtin_amdgcn_s_sleep(1);
      wait_vm<48>(); BAR();
      issue();
      SCHED();
      if (j == n0 - NS - 1) begin(desc, 0);
      if (j == J0 - 1) {
        begin(desc >> 16, 0);
        if (T1.src != rank) {
          srdA = __builtin_amdgcn_make_buffer_rsrc((void*)(myws + WS_XBUF + ((rfl(done0) / NWG + 1) & 1) * XPAR + (size_t)T1.src * XTILE), 0, (int)XTILE, 0x00020000);
          sA = (unsigned)pw * 8192u; stepK = 16384u; stepI = 1024u; voffA[0] = voffA[1] = (unsigned)lane * 16u;
        }
      }
      next();
    }
    wait_vm<48>(); BAR();
    wait_vm<32>(); BAR();
    wait_vm<16>(); BAR();
    wait_vm<0>(); BAR();
    BAR();
  } else {
    const int wr = wid >> 1, wc = wid & 1, fr = lane & 15, fq = lane >> 4;
    const unsigned baseA0 = (unsigned)(wr * 64 + fr) * 128u + (unsigned)((fq ^ ((fr >> 1) & 7)) * 16);
    unsigned baseB[4];
    { const int q = (lane & 15) >> 2, pp = lane & 3; const int s = (q << 2) | ((fq & 1) << 1);
#pragma unroll
      for (int n = 0; n < 4; ++n) baseB[n] = (unsigned)(NS * SLOT) + (unsigned)(8 * fq + q) * 256u + (unsigned)(((8 * wc + 2 * n) ^ s) * 16 + 8 * pp); }
    f32x4 acc[4][4];
    bf16x8 FA[2][4], FB[2][4];
    u32x4 O[8];
    typedef LAS char* lptr;
    auto rd = [&](bf16x8 (&A)[4], bf16x8 (&B)[4], lptr pa, lptr (&pb)[4], int ks) {
      auto rb = [&](int n) {
        bf16x4 lo = __builtin_amdgcn_ds_read_tr16_b64_v4i16((LAS bf16x4*)(pb[n] + ks * 8192));
        bf16x4 hi = __builtin_amdgcn_ds_read_tr16_b64_v4i16((LAS bf16x4*)(pb[n] + ks * 8192 + 1024));
        B[n] = __builtin_shufflevector(lo, hi, 0, 1, 2, 3, 4, 5, 6, 7);
      };
      rb(0);
#pragma unroll
      for (int m = 0; m < 4; ++m) A[m] = *(const LAS bf16x8*)(pa + m * 2048);
      rb(1); rb(2); rb(3);
    };
    auto mm = [&](bf16x8 (&A)[4], bf16x8 (&B)[4], int lo, int hi) {
#pragma unroll
      for (int i = lo; i < hi; ++i) {
        const int n = i >> 2, m = (n & 1) ? 3 - (i & 3) : (i & 3);
        acc[m][n] = __builtin_amdgcn_mfma_f32_16x16x32_bf16(B[n], A[m], acc[m][n], 0, 0, 0);
      }
    };
    auto pairs = [&](int n) {
#pragma unroll
      for (int i = 0; i < n; ++i) { __builtin_amdgcn_sched_group_barrier(0x8, 1, 0); __builtin_amdgcn_sched_group_barrier(0x100, 1, 0); }
    };
    auto out_ptr = [&](const Tile& T) {
      char* o = p.out[T.dest] + (size_t)(T.src * MLOC + T.rt * 128 + wr * 64) * (NSH * 2) + (T.ct * 128 + wc * 64) * 2;
      asm volatile("" : "+s"(o));
      return o;
    };
    char* const o0 = out_ptr(T0); char* const o1 = out_ptr(T1);
    unsigned* cnt1 = (unsigned*)(p.ws[T1.dest] + WS_OUTCNT + rank * 128);
    asm volatile("" : "+s"(cnt1));
    const unsigned vo = (unsigned)(fr & 7) * (NSH * 2) + 64u * (fr >> 3) + 16u * ((fq >> 1) + 2 * (fq & 1));
    auto pack1 = [&](int m, int q) {
      const unsigned x0 = gelu2(acc[m][2 * q][0], acc[m][2 * q][1]), x1 = gelu2(acc[m][2 * q][2], acc[m][2 * q][3]);
      const unsigned y0 = gelu2(acc[m][2 * q + 1][0], acc[m][2 * q + 1][1]), y1 = gelu2(acc[m][2 * q + 1][2], acc[m][2 * q + 1][3]);
      const auto r0 = __builtin_amdgcn_permlane16_swap(x0, y0, false, false);
      const auto r1 = __builtin_amdgcn_permlane16_swap(x1, y1, false, false);
      return (u32x4){r0[0], r1[0], r0[1], r1[1]};
    };
    auto pack2 = [&](int m, u32x4& lo, u32x4& hi) {
      const u32x4 a = pack1(m, 0), c = pack1(m, 1);
#pragma unroll
      for (int e = 0; e < 4; ++e) {
        lo[e] = (unsigned)__builtin_amdgcn_update_dpp((int)a[e], (int)c[e], 0x128, 0xf, 0xc, false);
        hi[e] = (unsigned)__builtin_amdgcn_update_dpp((int)c[e], (int)a[e], 0x128, 0xf, 0x3, false);
      }
    };
    auto signal = [&](unsigned* cnt) {
      wait_vm<0>();
      if (lane == 0) __hip_atomic_fetch_add(cnt, 1u, __ATOMIC_RELAXED, __HIP_MEMORY_SCOPE_SYSTEM);
    };
    unsigned so = 0;
    lptr pB[4], pA0 = lds + baseA0, pA1 = lds + (baseA0 ^ 64u);
#pragma unroll
    for (int n = 0; n < 4; ++n) pB[n] = lds + baseB[n];
#define PART1() do { SCHED(); rd(FA[1], FB[1], pA1, pB, 1); mm(FA[0], FB[0], 0, 16); pairs(12); __builtin_amdgcn_sched_group_barrier(0x8, 4, 0); SCHED(); } while (0)
#define PART2() do { \
      so = so + SLOT == NS * SLOT ? 0 : so + SLOT; \
      pA0 = lds + baseA0 + so; pA1 = lds + (baseA0 ^ 64u) + so; \
      _Pragma("unroll") for (int n = 0; n < 4; ++n) pB[n] = lds + baseB[n] + so; \
      mm(FA[1], FB[1], 0, 4); \
      asm volatile("" : "+v"(pA0), "+v"(pA1), "+v"(pB[0]), "+v"(pB[1]), "+v"(pB[2]), "+v"(pB[3])); \
      _Pragma("unroll") for (int i = 0; i < 3; ++i) { __builtin_amdgcn_sched_group_barrier(0x8, 1, 0); __builtin_amdgcn_sched_group_barrier(0x2, 2, 0); } \
      __builtin_amdgcn_sched_group_barrier(0x8, 1, 0); \
      SCHED(); __builtin_amdgcn_s_waitcnt(0xC07F); __builtin_amdgcn_s_barrier(); SCHED(); \
      rd(FA[0], FB[0], pA0, pB, 0); mm(FA[1], FB[1], 4, 16); pairs(12); SCHED(); } while (0)
    char* part = myws + WS_PART + (size_t)pidx * PART + wid * 16384;
    if (wid == 0) TS(0);
    BAR();
    if (wid == 0) TS(1);
    rd(FA[0], FB[0], pA0, pB, 0);
    f32x4 H[16];
    int hk = 99;
#pragma unroll 1
    for (int seg = helper ? 0 : 1; seg < 3; ++seg) {
      if (seg < 2) {
#pragma unroll
        for (int m = 0; m < 4; ++m)
#pragma unroll
          for (int n = 0; n < 4; ++n) acc[m][n] = (f32x4){0.f, 0.f, 0.f, 0.f};
      }
      const int nk = seg == 0 ? HK : seg == 1 ? n1 - PRELD : PRELD;
#pragma unroll 1
      for (int kt = 0; kt < nk; ++kt) {
        PART1();
        if (hk <= HDRAIN) {
#define HST(s) case s: st16_sys<1024>(part + (14 - 2 * s) * 1024, (unsigned)lane * 16u, __builtin_bit_cast(u32x4, H[15 - 2 * s])); \
                       st16_sys<0>(part + (14 - 2 * s) * 1024, (unsigned)lane * 16u, __builtin_bit_cast(u32x4, H[14 - 2 * s])); break;
          switch (hk) { HST(0) HST(1) HST(2) HST(3) HST(4) HST(5) HST(6) HST(7) case HDRAIN: wait_vm<0>(); break; default: break; }
          ++hk;
        }
        PART2();
      }
      if (seg == 0) {
#pragma unroll
        for (int i = 0; i < 16; ++i) H[i] = acc[i >> 2][i & 3];
        hk = 0;
      } else if (seg == 1) {
        const auto srdP = __builtin_amdgcn_make_buffer_rsrc((void*)part, 0, owner ? 16384 : 0, 0x00020000);
#pragma unroll
        for (int i = 0; i < 16; ++i) H[i] = __builtin_bit_cast(f32x4, __builtin_amdgcn_raw_buffer_load_b128(srdP, lane * 16 + (i & 3) * 1024, (i >> 2) * 4096, 16));
      }
    }
    if (wid == 0) TS(2);
#pragma unroll
    for (int i = 0; i < 16; ++i) acc[i >> 2][i & 3] += H[i];
#pragma unroll
    for (int m = 0; m < 4; ++m) pack2(m, O[2 * m], O[2 * m + 1]);
#pragma unroll
    for (int m = 0; m < 4; ++m)
#pragma unroll
      for (int n = 0; n < 4; ++n) acc[m][n] = (f32x4){0.f, 0.f, 0.f, 0.f};
    auto store0 = [&](int m) {
      switch (m) {
        case 0: ST0(o0, vo, O[0]); ST0(o0 + 32768, vo, O[1]); break;
        case 1: ST0(o0 + 65536, vo, O[2]); ST0(o0 + 65536 + 32768, vo, O[3]); break;
        case 2: ST0(o0 + 2 * 65536, vo, O[4]); ST0(o0 + 2 * 65536 + 32768, vo, O[5]); break;
        default: ST0(o0 + 3 * 65536, vo, O[6]); ST0(o0 + 3 * 65536 + 32768, vo, O[7]); break;
      }
    };
    __builtin_amdgcn_s_waitcnt(0x0F70);
#pragma unroll 1
    for (int kt = -kb0; kt < n2 - kb0; ++kt) {
      PART1();
      if ((unsigned)kt <= (unsigned)ODRAIN) {
        if (kt < 4) store0(kt);
        else if (kt == ODRAIN) wait_vm<0>();
      }
      PART2();
    }
    if (wid == 0) TS(3);
#pragma unroll
    for (int m = 0; m < 4; ++m) {
      u32x4 lo, hi; pack2(m, lo, hi);
      if (TTHR < 9 && T1.dest != rank) wait_vm<TTHR>();
      st16_sys<0>(o1 + m * 65536, vo, lo);
      if (TTHR < 9 && T1.dest != rank) wait_vm<TTHR>();
      st16_sys<0>(o1 + m * 65536 + 32768, vo, hi); SCHED();
    }
    if (T1.dest != rank) signal(cnt1);
  }
}

extern "C" void kernel_launch(int rank, int nranks, ncclComm_t comm, hipStream_t stream,
                              void* const* d_in, const int* in_sizes, int n_in,
                              void* d_out, int out_size, const int* in_dtypes, int out_dtype,
                              void* const* d_out_all, void* const* d_ws_all, size_t ws_bytes) {
  if (nranks != NR || ws_bytes < WS_END || (unsigned)rank >= (unsigned)NR) { fprintf(stderr, "unsupported configuration\n"); return; }
  HIP_CHECK(hipFuncSetAttribute((const void*)gemm_a2a_kernel, hipFuncAttributeMaxDynamicSharedMemorySize, LDS_BYTES));
  Params p;
  p.x = (const char*)d_in[0]; p.w = (const char*)d_in[1]; p.rank = rank; p.pad = 0;
  for (int i = 0; i < NR; ++i) { p.out[i] = (char*)d_out_all[i]; p.ws[i] = (char*)d_ws_all[i]; }
  build_desc(p.desc);
  gemm_a2a_kernel<<<NWG, 512, LDS_BYTES, stream>>>(p);
  HIP_CHECK(hipGetLastError());
}
```

```cpp
#include <hip/hip_runtime.h>
#include <rccl/rccl.h>
#include <cstdio>

#define HIP_CHECK(expr) do { hipError_t _e = (expr); if (_e != hipSuccess) { \
    fprintf(stderr, "HIP error %s at %s:%d\n", hipGetErrorString(_e), __FILE__, __LINE__); \
    return; } } while (0)

#define LAS __attribute__((address_space(3)))
typedef short bf16x8 __attribute__((ext_vector_type(8)));
typedef short bf16x4 __attribute__((ext_vector_type(4)));
typedef float f32x4 __attribute__((ext_vector_type(4)));
typedef float f32x2 __attribute__((ext_vector_type(2)));
typedef unsigned u32x4 __attribute__((ext_vector_type(4)));
typedef __bf16 bfx2 __attribute__((ext_vector_type(2)));

#define TAU_O 68
#define TAU_X 8
#define XDELAY 100
#define HK 12
#define PACE1 53
#define PACE1H 50
#define PACE1N 54
#define PACE2 44
constexpr int NR = 8;
constexpr int KDIM = 4096, NDIM = 16384, MLOC = 512, NSH = 2048;
constexpr int NKT = 64, SLOT = 16384, NS = 5, NWG = 256;
constexpr int LDS_BYTES = 2 * NS * SLOT;
constexpr int HDRAIN = 12, ODRAIN = 5, PRELD = 3;
constexpr int Q4 = 80;
constexpr size_t WS_DONE = 0, WS_TMO = 128, WS_OUTCNT = 4096, WS_XCNT = 8192, WS_O0CNT = 12288, WS_XLOC = 14336, WS_CEN = 16384, WS_ARR = 20480, WS_ACK = 24576, WS_PFLAG = 32768, WS_STAMP = 65536;
constexpr size_t XTILE = 128 * KDIM * 2;
constexpr size_t WS_XBUF = 1u << 20, XPAR = NR * XTILE;
constexpr size_t WS_PART = WS_XBUF + 2 * XPAR, PART = 128 * 128 * 4, WS_END = WS_PART + 96 * PART;
constexpr unsigned long long DEADLINE = 40000000ull;

enum { ROLE_OWNER = 1, ROLE_HELPER = 2 };
struct Params { const char* x; const char* w; char* out[NR]; char* ws[NR]; int rank; int pad; unsigned desc[NWG][2]; };
struct Tile { int src; int rt; int dest; int ct; };
__device__ __forceinline__ Tile tile_of(unsigned d, int rank) {
  Tile r; r.src = (rank + (int)((d >> 13) & 7)) & 7; r.rt = d & 3; r.dest = (rank + (int)((d >> 2) & 7)) & 7; r.ct = (d >> 5) & 15; return r;
}
static inline unsigned tile_code(int so, int rt, int dof, int ct, int G, int rot) { return (unsigned)(rt | dof << 2 | ct << 5 | (G - 1) << 9 | (rot % G) << 11 | so << 13); }
static void build_desc(unsigned (*out)[2]) {
  unsigned desc[NWG][2];
  unsigned T[2][8][32]; int n[2][8] = {};
  auto put = [&](int w, int x, unsigned c) { T[w][x][n[w][x]++] = c; };
  auto code = [&](int q, int rt) { return tile_code(0, rt, 1 + q % 7, q / 7, q == Q4 ? 4 : 3, rt + q); };
  for (int k = 0; k < 80; ++k) for (int rt = 0; rt < 3; ++rt) put(0, k / 10, code(k / 10 + 8 * (k % 10), rt));
  for (int i = 0; i < 16; ++i) put(0, i / 2, code(i < 12 ? 81 + i / 3 : Q4, i < 12 ? i % 3 : i - 12));
  for (int k = 0; k < 24; ++k) for (int rt = 0; rt < 3; ++rt) put(1, k / 6, code(85 + k / 6 + 4 * (k % 6), rt));
  for (int x = 0; x < 4; ++x) for (int c = 0; c < 4; ++c) for (int r = 0; r < 3; ++r) put(1, x, tile_code(0, r, 0, x + 4 * c, 3, r + c));
  for (int k = 0; k < 3; ++k) { put(1, k, code(109 + k, 0)); put(1, k, code(109 + k, 1)); }
  put(1, 3, code(109, 2)); put(1, 3, code(110, 2));
  for (int g = 0; g < 2; ++g) for (int h = 0; h < 2; ++h) {
    for (int c = 0; c < 8; ++c) for (int r = 0; r < 4; ++r)
      if (!(4 * g + r == 8 - (1 + Q4 % 7) && 8 * h + c == Q4 / 7)) put(1, 4 + 2 * g + h, tile_code(4 * g + r, 3, 0, 8 * h + c, 4, r + c + 2 * h + g));
    if (n[1][4 + 2 * g + h] < 32) put(1, 4 + 2 * g + h, code(111, 2));
  }
  int role[NWG], own[96], nown = 0;
  for (int b = 0; b < NWG; ++b) { desc[b][0] = T[0][b & 7][b >> 3] | T[1][b & 7][b >> 3] << 16; desc[b][1] = 0; role[b] = 0; }
  for (int pass = 0; pass < 2; ++pass) for (int x = 0; x < 4; ++x) for (int i = pass ? 30 : 0; i < (pass ? 32 : 18); ++i) own[nown++] = i * 8 + x;
  for (int k = 0; HK && k < nown; ++k) {
    const int o = own[k], h = (k % 30) * 8 + 4 + k / 30;
    desc[o][1] = (unsigned)k << 16 | 1u << 14; role[o] = ROLE_OWNER;
    desc[h][1] = (desc[o][0] & 0x1fff) | (unsigned)k << 16; role[h] = ROLE_HELPER;
  }
  auto sched = [](int r, int j) { const int p1 = r == ROLE_OWNER ? PACE1 : r == ROLE_HELPER ? PACE1H : PACE1N; return j < NKT ? j * p1 : NKT * p1 + (j - NKT) * PACE2; };
  auto turn_kt = [&](int k, int d, int r) {
    const int t = sched(ROLE_OWNER, NKT - HK) + k * TAU_O + d * (TAU_O / 8), j1 = r == ROLE_OWNER ? NKT - HK : r == ROLE_HELPER ? NKT + HK : NKT;
    if (t < sched(r, j1)) return -1;
    int j = j1; while (sched(r, j) < t) ++j;
    return j - j1 < NKT - 12 ? j - j1 : NKT - 12;
  };
  bool sent[NWG] = {};
  int load[64][8] = {};
  for (int k = 0, left = NWG; left; ++k)
    for (int dd = 0; dd < 7; ++dd) {
      const int d = 1 + (dd + k) % 7;
      int best = -1, bc = 1 << 30;
      for (int b = 0; b < NWG; ++b) {
        if (sent[b] || (int)((desc[b][0] >> 2) & 7) != d) continue;
        if (turn_kt(k, d, role[b]) < 0) continue;
        const int x = b & 7, c = 8 * (2 * load[k][x] + (k > 0 ? load[k - 1][x] : 0) + (k > 1 ? load[k - 2][x] : 0)) + 12 * (role[b] == ROLE_OWNER ? 0 : role[b] == ROLE_HELPER ? 1 : 2);
        if (c < bc) { bc = c; best = b; }
      }
      if (best >= 0) { sent[best] = true; --left; load[k][best & 7]++; desc[best][1] |= (unsigned)turn_kt(k, d, role[best]) << 24; }
    }
  for (int b = 0; b < NWG; ++b) { const int l = (b & ~7) | (b & 1) << 2 | (b & 7) >> 1; out[b][0] = desc[l][0]; out[b][1] = desc[l][1]; }
}

__device__ __forceinline__ unsigned gelu2(float y0, float y1) {
  const float k0 = -2.0f * 0.7978845608028654f * 1.4426950408889634f, k1 = k0 * 0.044715f;
  const f32x2 y = {y0, y1};
  const f32x2 z = y * __builtin_elementwise_fma(y * y, (f32x2){k1, k1}, (f32x2){k0, k0});
  const f32x2 d = (f32x2){__builtin_amdgcn_exp2f(z.x), __builtin_amdgcn_exp2f(z.y)} + 1.0f;
  const f32x2 g = y * (f32x2){__builtin_amdgcn_rcpf(d.x), __builtin_amdgcn_rcpf(d.y)};
  return __builtin_bit_cast(unsigned, __builtin_convertvector(g, bfx2));
}
__device__ __forceinline__ unsigned rfl(unsigned v) { return (unsigned)__builtin_amdgcn_readfirstlane((int)v); }
__device__ __forceinline__ unsigned now32() { return (unsigned)__builtin_amdgcn_s_memrealtime(); }
template <int N> __device__ __forceinline__ void wait_vm() { asm volatile("s_waitcnt vmcnt(%0)" :: "i"(N) : "memory"); }
template <int OFF> __device__ __forceinline__ void st16_sys(char* sbase, unsigned voff, u32x4 v) {
  asm volatile("global_store_dwordx4 %0, %1, %2 offset:%3 sc0 sc1\n\ts_nop 1" :: "v"(voff), "v"(v), "s"(sbase), "i"(OFF) : "memory");
}
__device__ __forceinline__ void st16_l2(char* sbase, unsigned voff, u32x4 v) {
  asm volatile("global_store_dwordx4 %0, %1, %2\n\ts_nop 1" :: "v"(voff), "v"(v), "s"(sbase) : "memory");
}
#define BAR() do { asm volatile("" ::: "memory"); __builtin_amdgcn_s_barrier(); asm volatile("" ::: "memory"); } while (0)
#define SCHED() __builtin_amdgcn_sched_barrier(0)
#define TS(k) do {} while (0)

__device__ __forceinline__ void wait_ge(const unsigned* flag, unsigned want, unsigned* tmo, unsigned call, bool active, char* mark, int lane) {
  const unsigned long long t0 = __builtin_amdgcn_s_memrealtime();
  for (unsigned n = 1;; ++n) {
    const bool ok = !active || __hip_atomic_load(flag, __ATOMIC_RELAXED, __HIP_MEMORY_SCOPE_SYSTEM) >= want;
    if (__all(ok)) return;
    if ((n & 15) == 0 &&
        (__hip_atomic_load(tmo, __ATOMIC_RELAXED, __HIP_MEMORY_SCOPE_AGENT) == call || __builtin_amdgcn_s_memrealtime() - t0 > DEADLINE)) {
      __hip_atomic_store(tmo, call, __ATOMIC_RELAXED, __HIP_MEMORY_SCOPE_AGENT);
      if (lane == 0) __hip_atomic_store((unsigned*)mark, 0x7fc07fc0u, __ATOMIC_RELAXED, __HIP_MEMORY_SCOPE_SYSTEM);
      return;
    }
    __builtin_amdgcn_s_sleep(1);
  }
}

__global__ void __launch_bounds__(512, 2) gemm_a2a_kernel(Params p) {
  extern __shared__ __attribute__((aligned(16))) char smem[];
  LAS char* lds = (LAS char*)smem;
  const int tid = threadIdx.x, wid = (int)rfl(tid >> 6), lane = tid & 63;
  const int b = blockIdx.x, rank = p.rank;
  const unsigned desc = p.desc[b][0], aux = p.desc[b][1];
  const Tile T0 = tile_of(desc, rank), T1 = tile_of(desc >> 16, rank);
  const bool owner = (aux >> 14) & 1, helper = (aux & 0x1fffu) != 0;
  const int n0 = helper ? HK : 0, n1 = owner ? NKT - HK : NKT, n2 = NKT;
  const int NJ = n0 + n1 + n2;
  const int J0 = n0 + n1 - NS;
  char* myws = p.ws[rank];
  unsigned* tmo = (unsigned*)(myws + WS_TMO);
  char* mark = p.out[rank] + (size_t)rank * MLOC * NSH * 2;
  const unsigned pidx = (aux >> 16) & 127;
  const int kb0 = (int)(aux >> 24);

  if (wid == 7) {
    const bool staged = T1.src != rank;
    if (!staged && !owner && !helper) return;
    const unsigned call = rfl(__hip_atomic_load((unsigned*)(myws + WS_DONE), __ATOMIC_RELAXED, __HIP_MEMORY_SCOPE_AGENT)) / NWG + 1;
    unsigned* pf = (unsigned*)(myws + WS_PFLAG + pidx * 128);
    BAR();
    int it = 0;
    if (helper) {
      for (; it <= HK + HDRAIN; ++it) BAR();
      if (lane == 0) __hip_atomic_fetch_add(pf, 1u, __ATOMIC_RELAXED, __HIP_MEMORY_SCOPE_AGENT);
    }
    if (owner) {
      for (; it < n1 - PRELD - 6; ++it) BAR();
      const unsigned seen = __hip_atomic_load(pf, __ATOMIC_RELAXED, __HIP_MEMORY_SCOPE_AGENT);
      for (; it < n1 - PRELD - 1; ++it) BAR();
      if (rfl(seen) < call) wait_ge(pf, call, tmo, call, true, mark, lane);
    }
    if (staged) {
      const unsigned* fl = (const unsigned*)(myws + WS_XCNT + T1.src * 512);
#pragma unroll 1
      for (int c = 0; c < 4; ++c) {
        for (; it < J0 + 16 * c - 5; ++it) BAR();
        const unsigned seen = __hip_atomic_load(fl + c * 32, __ATOMIC_RELAXED, __HIP_MEMORY_SCOPE_SYSTEM);
        for (; it < J0 + 16 * c; ++it) BAR();
        if (rfl(seen) < call) wait_ge(fl + c * 32, call, tmo, call, true, mark, lane);
        if (c == 0) TS(6);
      }
    }
  } else if (wid == 6) {
    u32x4 xv[4];
#pragma unroll
    for (int i = 0; i < 4; ++i) {
      const int P = b * 4 + i;
      xv[i] = *(const u32x4*)(p.x + 3 * XTILE + (size_t)((P & 15) * 8 + (lane >> 3)) * 8192 + (P >> 4) * 128 + ((lane & 7) ^ ((4 * (P & 1) + (lane >> 4)) & 7)) * 16);
    }
    const unsigned done = __hip_atomic_load((unsigned*)(myws + WS_DONE), __ATOMIC_RELAXED, __HIP_MEMORY_SCOPE_AGENT);
    const unsigned xcc = (unsigned)__builtin_amdgcn_s_getreg((3 << 11) | 20) & 15u;
    if (lane == 0) __hip_atomic_fetch_add((unsigned*)(myws + WS_CEN + xcc * 128), 1u, __ATOMIC_RELAXED, __HIP_MEMORY_SCOPE_AGENT);
    const unsigned peer = (unsigned)(rank + 1 + lane) & 7;
    BAR();
    int it = 0;
    auto bars = [&](int n) { for (int e = min(it + n, NJ); it < e; ++it) BAR(); };
    const int ch = b >> 6;
    const unsigned tn = now32() + XDELAY + (unsigned)b * TAU_X;
    while ((int)(now32() - tn) < 0 && it < 36 + 16 * ch) { BAR(); ++it; }
    __builtin_amdgcn_s_waitcnt(0x0F70);
    const unsigned call = rfl(done) / NWG + 1;
    char* pws = p.ws[0];
#pragma unroll
    for (int q = 1; q < NR; ++q) pws = peer == (unsigned)q ? p.ws[q] : pws;
    const size_t so = WS_XBUF + (call & 1) * XPAR + (size_t)rank * XTILE + (size_t)b * 4096;
#pragma unroll
    for (int i = 0; i < 4; ++i)
#pragma unroll
      for (int q = 0; q < 7; ++q) st16_sys<0>(p.ws[(rank + 1 + q) & 7] + so + i * 1024, (unsigned)lane * 16u, xv[i]);
    TS(9);
    bars(5);
    wait_vm<0>();
    unsigned tick; unsigned long long sv;
#define ATOM0(OP, ADDR, VAL) asm volatile("s_mov_b64 %1, exec\n\ts_mov_b64 exec, 1\n\t" OP " %0, %2, %3, off sc0 sc1\n\ts_mov_b64 exec, %1" \
                                          : "=&v"(tick), "=&s"(sv) : "v"(ADDR), "v"(VAL) : "memory")
    ATOM0("global_atomic_add", myws + WS_XLOC + ch * 128, 1u);
    bars(3);
    asm volatile("s_waitcnt vmcnt(0)" : "+v"(tick) :: "memory");
    if (rfl(tick) == call * 64 - 1) {
      if (lane < 7) __hip_atomic_store((unsigned*)(pws + WS_XCNT + rank * 512 + ch * 128), call, __ATOMIC_RELAXED, __HIP_MEMORY_SCOPE_SYSTEM);
    }
    if (lane == 0) __hip_atomic_fetch_add((unsigned*)(myws + WS_DONE), 1u, __ATOMIC_RELAXED, __HIP_MEMORY_SCOPE_AGENT);
    TS(5);
    bars(n0 + n1 + kb0 + ODRAIN + 1 - it);
    ATOM0("global_atomic_add", myws + WS_ARR + xcc * 128, 1u);
    const unsigned cen = __hip_atomic_load((unsigned*)(myws + WS_CEN + (lane & 15) * 128), __ATOMIC_RELAXED, __HIP_MEMORY_SCOPE_AGENT);
    bars(2);
    asm volatile("s_waitcnt vmcnt(0)" : "+v"(tick) :: "memory");
    unsigned total = 0;
#pragma unroll
    for (int k = 0; k < 16; ++k) total += (unsigned)__builtin_amdgcn_readlane((int)cen, k);
    const unsigned t = rfl(tick) + 1;
    if (total != call * NWG || t == (unsigned)__builtin_amdgcn_readlane((int)cen, (int)xcc)) {
      asm volatile("buffer_wbl2 sc0 sc1" ::: "memory");
      bars(4);
      wait_vm<0>();
      ATOM0("global_atomic_umax", myws + WS_ACK + xcc * 128, t);
      bars(2);
      asm volatile("s_waitcnt vmcnt(0)" : "+v"(tick) :: "memory");
      const unsigned old = rfl(tick);
      if (t > old && lane < 7) __hip_atomic_fetch_add((unsigned*)(pws + WS_O0CNT + rank * 128), t - old, __ATOMIC_RELAXED, __HIP_MEMORY_SCOPE_SYSTEM);
    }
    TS(7);
    if (b == NWG - 1) {
      bars(NJ);
      const unsigned d = 7 - (lane & 7), n2nd = 3 * ((111 - (d - 1)) / 7 - (84 - (d - 1)) / 7);
      wait_ge((const unsigned*)(myws + (lane < 8 ? WS_OUTCNT : WS_O0CNT) + peer * 128), lane < 8 ? call * 4 * n2nd : call * NWG, tmo, call, lane < 15 && lane != 7, mark, lane);
      TS(8);
    }
  } else if (wid >= 4) {
    const int pw = wid - 4;
    unsigned done0;
    asm volatile("global_load_dword %0, %1, off sc1" : "=v"(done0) : "v"(myws + WS_DONE) : "memory");
    const auto srdB = __builtin_amdgcn_make_buffer_rsrc((void*)p.w, 0, KDIM * NDIM * 2, 0x00020000);
    unsigned voffA[2], voffB[2];
#pragma unroll
    for (int i = 0; i < 2; ++i) {
      voffA[i] = (unsigned)(lane >> 3) * 8192u + (unsigned)(((lane & 7) ^ ((4 * i + (lane >> 4)) & 7)) * 16);
      voffB[i] = (unsigned)(lane >> 4) * 32768u + (unsigned)(((lane & 15) ^ (((lane >> 4) << 2) | (i << 1))) * 16);
    }
    unsigned sA = (unsigned)(64 * pw) * 8192u, stepK = 128u, stepI = 65536u;
    unsigned sB, G, kv, kp, kb;
    auto srdA = __builtin_amdgcn_make_buffer_rsrc((void*)p.x, 0, (int)XTILE, 0x00020000);
    auto begin = [&](unsigned d, unsigned k0) {
      srdA = __builtin_amdgcn_make_buffer_rsrc((void*)(p.x + (size_t)(d & 3) * XTILE), 0, (int)XTILE, 0x00020000);
      sB = (unsigned)(32 * pw) * 32768u + ((((unsigned)rank + ((d >> 2) & 7)) & 7) * 16 + ((d >> 5) & 15)) * 256u;
      G = ((d >> 9) & 3) + 1; kv = (d >> 11) & 3; kp = 0; kb = k0;
    };
    unsigned soA, soB, ldsb = (unsigned)pw * 8192u;
    auto next = [&]() {
      const unsigned kt = min(kb + kv, 63u);
      soA = sA + kt * stepK; soB = sB + kt * (64u * 32768u);
      kv = kv + 1 == G ? 0 : kv + 1; ++kp;
      if (kp == G) { kp = 0; kb += G; }
    };
    auto issue = [&]() {
      LAS char* d = lds + ldsb;
#pragma unroll
      for (int i = 0; i < 8; ++i) __builtin_amdgcn_raw_ptr_buffer_load_lds(srdA, (LAS void*)(d + i * 1024), 16, voffA[i & 1], soA + i * stepI, 0, 17);
#pragma unroll
      for (int i = 0; i < 8; ++i) __builtin_amdgcn_raw_ptr_buffer_load_lds(srdB, (LAS void*)(d + NS * SLOT + i * 1024), 16, voffB[(i >> 1) & 1], soB + i * (4u * 32768u), 0, 0);
      ldsb = ldsb + SLOT >= NS * SLOT ? ldsb + SLOT - NS * SLOT : ldsb + SLOT;
    };
    static_assert(HK == 0 || HK > NS, "the prologue stays inside the first piece");
    begin(helper ? aux : desc, NKT - n1);
    next(); issue(); next(); issue();
    asm volatile("s_waitcnt vmcnt(16)" : "+v"(done0) :: "memory");
    BAR();
#pragma unroll
    for (int i = 2; i < NS; ++i) { next(); issue(); }
    next();
    unsigned tpace = now32();
    const unsigned pace1 = owner ? PACE1 : helper ? PACE1H : PACE1N;
    for (int j = 0; j < NJ - NS; ++j) {
      tpace += j < NKT ? pace1 : PACE2;
      while ((int)(now32() - tpace) < 0) __builtin_amdgcn_s_sleep(1);
      wait_vm<48>(); BAR();
      issue();
      SCHED();
      if (j == n0 - NS - 1) begin(desc, 0);
      if (j == J0 - 1) {
        begin(desc >> 16, 0);
        if (T1.src != rank) {
          srdA = __builtin_amdgcn_make_buffer_rsrc((void*)(myws + WS_XBUF + ((rfl(done0) / NWG + 1) & 1) * XPAR + (size_t)T1.src * XTILE), 0, (int)XTILE, 0x00020000);
          sA = (unsigned)pw * 8192u; stepK = 16384u; stepI = 1024u; voffA[0] = voffA[1] = (unsigned)lane * 16u;
        }
      }
      next();
    }
    wait_vm<48>(); BAR();
    wait_vm<32>(); BAR();
    wait_vm<16>(); BAR();
    wait_vm<0>(); BAR();
    BAR();
  } else {
    const int wr = wid >> 1, wc = wid & 1, fr = lane & 15, fq = lane >> 4;
    const unsigned baseA0 = (unsigned)(wr * 64 + fr) * 128u + (unsigned)((fq ^ ((fr >> 1) & 7)) * 16);
    unsigned baseB[4];
    { const int q = (lane & 15) >> 2, pp = lane & 3; const int s = (q << 2) | ((fq & 1) << 1);
#pragma unroll
      for (int n = 0; n < 4; ++n) baseB[n] = (unsigned)(NS * SLOT) + (unsigned)(8 * fq + q) * 256u + (unsigned)(((8 * wc + 2 * n) ^ s) * 16 + 8 * pp); }
    f32x4 acc[4][4];
    bf16x8 FA[2][4], FB[2][4];
    u32x4 O[8];
    typedef LAS char* lptr;
    auto rd = [&](bf16x8 (&A)[4], bf16x8 (&B)[4], lptr pa, lptr (&pb)[4], int ks) {
      auto rb = [&](int n) {
        bf16x4 lo = __builtin_amdgcn_ds_read_tr16_b64_v4i16((LAS bf16x4*)(pb[n] + ks * 8192));
        bf16x4 hi = __builtin_amdgcn_ds_read_tr16_b64_v4i16((LAS bf16x4*)(pb[n] + ks * 8192 + 1024));
        B[n] = __builtin_shufflevector(lo, hi, 0, 1, 2, 3, 4, 5, 6, 7);
      };
      rb(0);
#pragma unroll
      for (int m = 0; m < 4; ++m) A[m] = *(const LAS bf16x8*)(pa + m * 2048);
      rb(1); rb(2); rb(3);
    };
    auto mm = [&](bf16x8 (&A)[4], bf16x8 (&B)[4], int lo, int hi) {
#pragma unroll
      for (int i = lo; i < hi; ++i) {
        const int n = i >> 2, m = (n & 1) ? 3 - (i & 3) : (i & 3);
        acc[m][n] = __builtin_amdgcn_mfma_f32_16x16x32_bf16(B[n], A[m], acc[m][n], 0, 0, 0);
      }
    };
    auto pairs = [&](int n) {
#pragma unroll
      for (int i = 0; i < n; ++i) { __builtin_amdgcn_sched_group_barrier(0x8, 1, 0); __builtin_amdgcn_sched_group_barrier(0x100, 1, 0); }
    };
    auto out_ptr = [&](const Tile& T) {
      char* o = p.out[T.dest] + (size_t)(T.src * MLOC + T.rt * 128 + wr * 64) * (NSH * 2) + (T.ct * 128 + wc * 64) * 2;
      asm volatile("" : "+s"(o));
      return o;
    };
    char* const o0 = out_ptr(T0); char* const o1 = out_ptr(T1);
    unsigned* cnt1 = (unsigned*)(p.ws[T1.dest] + WS_OUTCNT + rank * 128);
    asm volatile("" : "+s"(cnt1));
    const unsigned vo = (unsigned)(fr & 7) * (NSH * 2) + 64u * (fr >> 3) + 16u * ((fq >> 1) + 2 * (fq & 1));
    auto pack1 = [&](int m, int q) {
      const unsigned x0 = gelu2(acc[m][2 * q][0], acc[m][2 * q][1]), x1 = gelu2(acc[m][2 * q][2], acc[m][2 * q][3]);
      const unsigned y0 = gelu2(acc[m][2 * q + 1][0], acc[m][2 * q + 1][1]), y1 = gelu2(acc[m][2 * q + 1][2], acc[m][2 * q + 1][3]);
      const auto r0 = __builtin_amdgcn_permlane16_swap(x0, y0, false, false);
      const auto r1 = __builtin_amdgcn_permlane16_swap(x1, y1, false, false);
      return (u32x4){r0[0], r1[0], r0[1], r1[1]};
    };
    auto pack2 = [&](int m, u32x4& lo, u32x4& hi) {
      const u32x4 a = pack1(m, 0), c = pack1(m, 1);
#pragma unroll
      for (int e = 0; e < 4; ++e) {
        lo[e] = (unsigned)__builtin_amdgcn_update_dpp((int)a[e], (int)c[e], 0x128, 0xf, 0xc, false);
        hi[e] = (unsigned)__builtin_amdgcn_update_dpp((int)c[e], (int)a[e], 0x128, 0xf, 0x3, false);
      }
    };
    auto signal = [&](unsigned* cnt) {
      wait_vm<0>();
      if (lane == 0) __hip_atomic_fetch_add(cnt, 1u, __ATOMIC_RELAXED, __HIP_MEMORY_SCOPE_SYSTEM);
    };
    unsigned so = 0;
    lptr pB[4], pA0 = lds + baseA0, pA1 = lds + (baseA0 ^ 64u);
#pragma unroll
    for (int n = 0; n < 4; ++n) pB[n] = lds + baseB[n];
#define PART1() do { SCHED(); rd(FA[1], FB[1], pA1, pB, 1); mm(FA[0], FB[0], 0, 16); pairs(12); __builtin_amdgcn_sched_group_barrier(0x8, 4, 0); SCHED(); } while (0)
#define PART2() do { \
      so = so + SLOT == NS * SLOT ? 0 : so + SLOT; \
      pA0 = lds + baseA0 + so; pA1 = lds + (baseA0 ^ 64u) + so; \
      _Pragma("unroll") for (int n = 0; n < 4; ++n) pB[n] = lds + baseB[n] + so; \
      mm(FA[1], FB[1], 0, 4); \
      asm volatile("" : "+v"(pA0), "+v"(pA1), "+v"(pB[0]), "+v"(pB[1]), "+v"(pB[2]), "+v"(pB[3])); \
      _Pragma("unroll") for (int i = 0; i < 3; ++i) { __builtin_amdgcn_sched_group_barrier(0x8, 1, 0); __builtin_amdgcn_sched_group_barrier(0x2, 2, 0); } \
      __builtin_amdgcn_sched_group_barrier(0x8, 1, 0); \
      SCHED(); __builtin_amdgcn_s_waitcnt(0xC07F); __builtin_amdgcn_s_barrier(); SCHED(); \
      rd(FA[0], FB[0], pA0, pB, 0); mm(FA[1], FB[1], 4, 16); pairs(12); SCHED(); } while (0)
    char* part = myws + WS_PART + (size_t)pidx * PART + wid * 16384;
    if (wid == 0) TS(0);
    BAR();
    if (wid == 0) TS(1);
    rd(FA[0], FB[0], pA0, pB, 0);
    f32x4 H[16];
    int hk = 99;
#pragma unroll 1
    for (int seg = helper ? 0 : 1; seg < 3; ++seg) {
      if (seg < 2) {
#pragma unroll
        for (int m = 0; m < 4; ++m)
#pragma unroll
          for (int n = 0; n < 4; ++n) acc[m][n] = (f32x4){0.f, 0.f, 0.f, 0.f};
      }
      const int nk = seg == 0 ? HK : seg == 1 ? n1 - PRELD : PRELD;
#pragma unroll 1
      for (int kt = 0; kt < nk; ++kt) {
        PART1();
        if (hk <= HDRAIN) {
#define HST(s) case s: st16_sys<1024>(part + (14 - 2 * s) * 1024, (unsigned)lane * 16u, __builtin_bit_cast(u32x4, H[15 - 2 * s])); \
                       st16_sys<0>(part + (14 - 2 * s) * 1024, (unsigned)lane * 16u, __builtin_bit_cast(u32x4, H[14 - 2 * s])); break;
          switch (hk) { HST(0) HST(1) HST(2) HST(3) HST(4) HST(5) HST(6) HST(7) case HDRAIN: wait_vm<0>(); break; default: break; }
          ++hk;
        }
        PART2();
      }
      if (seg == 0) {
#pragma unroll
        for (int i = 0; i < 16; ++i) H[i] = acc[i >> 2][i & 3];
        hk = 0;
      } else if (seg == 1) {
        const auto srdP = __builtin_amdgcn_make_buffer_rsrc((void*)part, 0, owner ? 16384 : 0, 0x00020000);
#pragma unroll
        for (int i = 0; i < 16; ++i) H[i] = __builtin_bit_cast(f32x4, __builtin_amdgcn_raw_buffer_load_b128(srdP, lane * 16 + (i & 3) * 1024, (i >> 2) * 4096, 16));
      }
    }
    if (wid == 0) TS(2);
#pragma unroll
    for (int i = 0; i < 16; ++i) acc[i >> 2][i & 3] += H[i];
#pragma unroll
    for (int m = 0; m < 4; ++m) pack2(m, O[2 * m], O[2 * m + 1]);
#pragma unroll
    for (int m = 0; m < 4; ++m)
#pragma unroll
      for (int n = 0; n < 4; ++n) acc[m][n] = (f32x4){0.f, 0.f, 0.f, 0.f};
    auto store0 = [&](int m) {
      switch (m) {
        case 0: st16_l2(o0, vo, O[0]); st16_l2(o0 + 32768, vo, O[1]); break;
        case 1: st16_l2(o0 + 65536, vo, O[2]); st16_l2(o0 + 65536 + 32768, vo, O[3]); break;
        case 2: st16_l2(o0 + 2 * 65536, vo, O[4]); st16_l2(o0 + 2 * 65536 + 32768, vo, O[5]); break;
        default: st16_l2(o0 + 3 * 65536, vo, O[6]); st16_l2(o0 + 3 * 65536 + 32768, vo, O[7]); break;
      }
    };
    __builtin_amdgcn_s_waitcnt(0x0F70);
#pragma unroll 1
    for (int kt = -kb0; kt < n2 - kb0; ++kt) {
      PART1();
      if ((unsigned)kt <= (unsigned)ODRAIN) {
        if (kt < 4) store0(kt);
        else if (kt == ODRAIN) wait_vm<0>();
      }
      PART2();
    }
    if (wid == 0) TS(3);
#pragma unroll
    for (int m = 0; m < 4; ++m) {
      u32x4 lo, hi; pack2(m, lo, hi);
      if (T1.dest != rank) wait_vm<1>();
      st16_sys<0>(o1 + m * 65536, vo, lo);
      if (T1.dest != rank) wait_vm<1>();
      st16_sys<0>(o1 + m * 65536 + 32768, vo, hi); SCHED();
    }
    if (T1.dest != rank) signal(cnt1);
  }
}

extern "C" void kernel_launch(int rank, int nranks, ncclComm_t comm, hipStream_t stream,
                              void* const* d_in, const int* in_sizes, int n_in,
                              void* d_out, int out_size, const int* in_dtypes, int out_dtype,
                              void* const* d_out_all, void* const* d_ws_all, size_t ws_bytes) {
  if (nranks != NR || ws_bytes < WS_END || (unsigned)rank >= (unsigned)NR) { fprintf(stderr, "unsupported configuration\n"); return; }
  HIP_CHECK(hipFuncSetAttribute((const void*)gemm_a2a_kernel, hipFuncAttributeMaxDynamicSharedMemorySize, LDS_BYTES));
  Params p;
  p.x = (const char*)d_in[0]; p.w = (const char*)d_in[1]; p.rank = rank; p.pad = 0;
  for (int i = 0; i < NR; ++i) { p.out[i] = (char*)d_out_all[i]; p.ws[i] = (char*)d_ws_all[i]; }
  build_desc(p.desc);
  gemm_a2a_kernel<<<NWG, 512, LDS_BYTES, stream>>>(p);
  HIP_CHECK(hipGetLastError());
}
```

```cpp
#include <hip/hip_runtime.h>
#include <rccl/rccl.h>
#include <cstdio>

#define HIP_CHECK(expr) do { hipError_t _e = (expr); if (_e != hipSuccess) { \
    fprintf(stderr, "HIP error %s at %s:%d\n", hipGetErrorString(_e), __FILE__, __LINE__); \
    return; } } while (0)

#define LAS __attribute__((address_space(3)))
typedef short bf16x8 __attribute__((ext_vector_type(8)));
typedef short bf16x4 __attribute__((ext_vector_type(4)));
typedef float f32x4 __attribute__((ext_vector_type(4)));
typedef float f32x2 __attribute__((ext_vector_type(2)));
typedef unsigned u32x4 __attribute__((ext_vector_type(4)));
typedef __bf16 bfx2 __attribute__((ext_vector_type(2)));

#define TAU_O 68
#define TAU_X 8
#define XDELAY 100
#define HK 12
#define PACE1 60
#define PACE1H 57
#define PACE1N 61
#define PSW 64
#define PHEAD 500
#define PACE2 44
constexpr int NR = 8;
constexpr int KDIM = 4096, NDIM = 16384, MLOC = 512, NSH = 2048;
constexpr int NKT = 64, SLOT = 16384, NS = 5, NWG = 256;
constexpr int LDS_BYTES = 2 * NS * SLOT;
constexpr int HDRAIN = 12, ODRAIN = 5, PRELD = 3;
constexpr int Q4 = 80;
constexpr size_t WS_DONE = 0, WS_TMO = 128, WS_OUTCNT = 4096, WS_XCNT = 8192, WS_O0CNT = 12288, WS_XLOC = 14336, WS_CEN = 16384, WS_ARR = 20480, WS_ACK = 24576, WS_PFLAG = 32768, WS_STAMP = 65536;
constexpr size_t XTILE = 128 * KDIM * 2;
constexpr size_t WS_XBUF = 1u << 20, XPAR = NR * XTILE;
constexpr size_t WS_PART = WS_XBUF + 2 * XPAR, PART = 128 * 128 * 4, WS_END = WS_PART + 96 * PART;
constexpr unsigned long long DEADLINE = 40000000ull;

enum { ROLE_OWNER = 1, ROLE_HELPER = 2 };
struct Params { const char* x; const char* w; char* out[NR]; char* ws[NR]; int rank; int pad; unsigned desc[NWG][2]; };
struct Tile { int src; int rt; int dest; int ct; };
__device__ __forceinline__ Tile tile_of(unsigned d, int rank) {
  Tile r; r.src = (rank + (int)((d >> 13) & 7)) & 7; r.rt = d & 3; r.dest = (rank + (int)((d >> 2) & 7)) & 7; r.ct = (d >> 5) & 15; return r;
}
static inline unsigned tile_code(int so, int rt, int dof, int ct, int G, int rot) { return (unsigned)(rt | dof << 2 | ct << 5 | (G - 1) << 9 | (rot % G) << 11 | so << 13); }
static void build_desc(unsigned (*out)[2]) {
  unsigned desc[NWG][2];
  unsigned T[2][8][32]; int n[2][8] = {};
  auto put = [&](int w, int x, unsigned c) { T[w][x][n[w][x]++] = c; };
  auto code = [&](int q, int rt) { return tile_code(0, rt, 1 + q % 7, q / 7, q == Q4 ? 4 : 3, rt + q); };
  for (int k = 0; k < 80; ++k) for (int rt = 0; rt < 3; ++rt) put(0, k / 10, code(k / 10 + 8 * (k % 10), rt));
  for (int i = 0; i < 16; ++i) put(0, i / 2, code(i < 12 ? 81 + i / 3 : Q4, i < 12 ? i % 3 : i - 12));
  for (int k = 0; k < 24; ++k) for (int rt = 0; rt < 3; ++rt) put(1, k / 6, code(85 + k / 6 + 4 * (k % 6), rt));
  for (int x = 0; x < 4; ++x) for (int c = 0; c < 4; ++c) for (int r = 0; r < 3; ++r) put(1, x, tile_code(0, r, 0, x + 4 * c, 3, r + c));
  for (int k = 0; k < 3; ++k) { put(1, k, code(109 + k, 0)); put(1, k, code(109 + k, 1)); }
  put(1, 3, code(109, 2)); put(1, 3, code(110, 2));
  for (int g = 0; g < 2; ++g) for (int h = 0; h < 2; ++h) {
    for (int c = 0; c < 8; ++c) for (int r = 0; r < 4; ++r)
      if (!(4 * g + r == 8 - (1 + Q4 % 7) && 8 * h + c == Q4 / 7)) put(1, 4 + 2 * g + h, tile_code(4 * g + r, 3, 0, 8 * h + c, 4, r + c + 2 * h + g));
    if (n[1][4 + 2 * g + h] < 32) put(1, 4 + 2 * g + h, code(111, 2));
  }
  int role[NWG], own[96], nown = 0;
  for (int b = 0; b < NWG; ++b) { desc[b][0] = T[0][b & 7][b >> 3] | T[1][b & 7][b >> 3] << 16; desc[b][1] = 0; role[b] = 0; }
  for (int pass = 0; pass < 2; ++pass) for (int x = 0; x < 4; ++x) for (int i = pass ? 30 : 0; i < (pass ? 32 : 18); ++i) own[nown++] = i * 8 + x;
  for (int k = 0; HK && k < nown; ++k) {
    const int o = own[k], h = (k % 30) * 8 + 4 + k / 30;
    desc[o][1] = (unsigned)k << 16 | 1u << 14; role[o] = ROLE_OWNER;
    desc[h][1] = (desc[o][0] & 0x1fff) | (unsigned)k << 16; role[h] = ROLE_HELPER;
  }
  auto sched = [](int r, int j) { const int p1 = r == ROLE_OWNER ? PACE1 : r == ROLE_HELPER ? PACE1H : PACE1N; return j < PSW ? j * p1 : PSW * p1 + (j - PSW) * PACE2; };
  auto turn_kt = [&](int k, int d, int r) {
    const int t = sched(ROLE_OWNER, NKT - HK) + k * TAU_O + d * (TAU_O / 8), j1 = r == ROLE_OWNER ? NKT - HK : r == ROLE_HELPER ? NKT + HK : NKT;
    if (t < sched(r, j1)) return -1;
    int j = j1; while (sched(r, j) < t) ++j;
    return j - j1 < NKT - 12 ? j - j1 : NKT - 12;
  };
  bool sent[NWG] = {};
  int load[64][8] = {};
  for (int k = 0, left = NWG; left; ++k)
    for (int dd = 0; dd < 7; ++dd) {
      const int d = 1 + (dd + k) % 7;
      int best = -1, bc = 1 << 30;
      for (int b = 0; b < NWG; ++b) {
        if (sent[b] || (int)((desc[b][0] >> 2) & 7) != d) continue;
        if (turn_kt(k, d, role[b]) < 0) continue;
        const int x = b & 7, c = 8 * (2 * load[k][x] + (k > 0 ? load[k - 1][x] : 0) + (k > 1 ? load[k - 2][x] : 0)) + 12 * (role[b] == ROLE_OWNER ? 0 : role[b] == ROLE_HELPER ? 1 : 2);
        if (c < bc) { bc = c; best = b; }
      }
      if (best >= 0) { sent[best] = true; --left; load[k][best & 7]++; desc[best][1] |= (unsigned)turn_kt(k, d, role[best]) << 24; }
    }
  for (int b = 0; b < NWG; ++b) { const int l = (b & ~7) | (b & 1) << 2 | (b & 7) >> 1; out[b][0] = desc[l][0]; out[b][1] = desc[l][1]; }
}

__device__ __forceinline__ unsigned gelu2(float y0, float y1) {
  const float k0 = -2.0f * 0.7978845608028654f * 1.4426950408889634f, k1 = k0 * 0.044715f;
  const f32x2 y = {y0, y1};
  const f32x2 z = y * __builtin_elementwise_fma(y * y, (f32x2){k1, k1}, (f32x2){k0, k0});
  const f32x2 d = (f32x2){__builtin_amdgcn_exp2f(z.x), __builtin_amdgcn_exp2f(z.y)} + 1.0f;
  const f32x2 g = y * (f32x2){__builtin_amdgcn_rcpf(d.x), __builtin_amdgcn_rcpf(d.y)};
  return __builtin_bit_cast(unsigned, __builtin_convertvector(g, bfx2));
}
__device__ __forceinline__ unsigned rfl(unsigned v) { return (unsigned)__builtin_amdgcn_readfirstlane((int)v); }
__device__ __forceinline__ unsigned now32() { return (unsigned)__builtin_amdgcn_s_memrealtime(); }
template <int N> __device__ __forceinline__ void wait_vm() { asm volatile("s_waitcnt vmcnt(%0)" :: "i"(N) : "memory"); }
template <int OFF> __device__ __forceinline__ void st16_sys(char* sbase, unsigned voff, u32x4 v) {
  asm volatile("global_store_dwordx4 %0, %1, %2 offset:%3 sc0 sc1\n\ts_nop 1" :: "v"(voff), "v"(v), "s"(sbase), "i"(OFF) : "memory");
}
__device__ __forceinline__ void st16_l2(char* sbase, unsigned voff, u32x4 v) {
  asm volatile("global_store_dwordx4 %0, %1, %2\n\ts_nop 1" :: "v"(voff), "v"(v), "s"(sbase) : "memory");
}
#define BAR() do { asm volatile("" ::: "memory"); __builtin_amdgcn_s_barrier(); asm volatile("" ::: "memory"); } while (0)
#define SCHED() __builtin_amdgcn_sched_barrier(0)
#define TS(k) do {} while (0)

__device__ __forceinline__ void wait_ge(const unsigned* flag, unsigned want, unsigned* tmo, unsigned call, bool active, char* mark, int lane) {
  const unsigned long long t0 = __builtin_amdgcn_s_memrealtime();
  for (unsigned n = 1;; ++n) {
    const bool ok = !active || __hip_atomic_load(flag, __ATOMIC_RELAXED, __HIP_MEMORY_SCOPE_SYSTEM) >= want;
    if (__all(ok)) return;
    if ((n & 15) == 0 &&
        (__hip_atomic_load(tmo, __ATOMIC_RELAXED, __HIP_MEMORY_SCOPE_AGENT) == call || __builtin_amdgcn_s_memrealtime() - t0 > DEADLINE)) {
      __hip_atomic_store(tmo, call, __ATOMIC_RELAXED, __HIP_MEMORY_SCOPE_AGENT);
      if (lane == 0) __hip_atomic_store((unsigned*)mark, 0x7fc07fc0u, __ATOMIC_RELAXED, __HIP_MEMORY_SCOPE_SYSTEM);
      return;
    }
    __builtin_amdgcn_s_sleep(1);
  }
}

__global__ void __launch_bounds__(512, 2) gemm_a2a_kernel(Params p) {
  extern __shared__ __attribute__((aligned(16))) char smem[];
  LAS char* lds = (LAS char*)smem;
  const int tid = threadIdx.x, wid = (int)rfl(tid >> 6), lane = tid & 63;
  const int b = blockIdx.x, rank = p.rank;
  const unsigned desc = p.desc[b][0], aux = p.desc[b][1];
  const Tile T0 = tile_of(desc, rank), T1 = tile_of(desc >> 16, rank);
  const bool owner = (aux >> 14) & 1, helper = (aux & 0x1fffu) != 0;
  const int n0 = helper ? HK : 0, n1 = owner ? NKT - HK : NKT, n2 = NKT;
  const int NJ = n0 + n1 + n2;
  const int J0 = n0 + n1 - NS;
  char* myws = p.ws[rank];
  unsigned* tmo = (unsigned*)(myws + WS_TMO);
  char* mark = p.out[rank] + (size_t)rank * MLOC * NSH * 2;
  const unsigned pidx = (aux >> 16) & 127;
  const int kb0 = (int)(aux >> 24);

  if (wid == 7) {
    const bool staged = T1.src != rank;
    if (!staged && !owner && !helper) return;
    const unsigned call = rfl(__hip_atomic_load((unsigned*)(myws + WS_DONE), __ATOMIC_RELAXED, __HIP_MEMORY_SCOPE_AGENT)) / NWG + 1;
    unsigned* pf = (unsigned*)(myws + WS_PFLAG + pidx * 128);
    BAR();
    int it = 0;
    if (helper) {
      for (; it <= HK + HDRAIN; ++it) BAR();
      if (lane == 0) __hip_atomic_fetch_add(pf, 1u, __ATOMIC_RELAXED, __HIP_MEMORY_SCOPE_AGENT);
    }
    if (owner) {
      for (; it < n1 - PRELD - 6; ++it) BAR();
      const unsigned seen = __hip_atomic_load(pf, __ATOMIC_RELAXED, __HIP_MEMORY_SCOPE_AGENT);
      for (; it < n1 - PRELD - 1; ++it) BAR();
      if (rfl(seen) < call) wait_ge(pf, call, tmo, call, true, mark, lane);
    }
    if (staged) {
      const unsigned* fl = (const unsigned*)(myws + WS_XCNT + T1.src * 512);
#pragma unroll 1
      for (int c = 0; c < 4; ++c) {
        for (; it < J0 + 16 * c - 5; ++it) BAR();
        const unsigned seen = __hip_atomic_load(fl + c * 32, __ATOMIC_RELAXED, __HIP_MEMORY_SCOPE_SYSTEM);
        for (; it < J0 + 16 * c; ++it) BAR();
        if (rfl(seen) < call) wait_ge(fl + c * 32, call, tmo, call, true, mark, lane);
        if (c == 0) TS(6);
      }
    }
  } else if (wid == 6) {
    u32x4 xv[4];
#pragma unroll
    for (int i = 0; i < 4; ++i) {
      const int P = b * 4 + i;
      xv[i] = *(const u32x4*)(p.x + 3 * XTILE + (size_t)((P & 15) * 8 + (lane >> 3)) * 8192 + (P >> 4) * 128 + ((lane & 7) ^ ((4 * (P & 1) + (lane >> 4)) & 7)) * 16);
    }
    const unsigned done = __hip_atomic_load((unsigned*)(myws + WS_DONE), __ATOMIC_RELAXED, __HIP_MEMORY_SCOPE_AGENT);
    const unsigned xcc = (unsigned)__builtin_amdgcn_s_getreg((3 << 11) | 20) & 15u;
    if (lane == 0) __hip_atomic_fetch_add((unsigned*)(myws + WS_CEN + xcc * 128), 1u, __ATOMIC_RELAXED, __HIP_MEMORY_SCOPE_AGENT);
    const unsigned peer = (unsigned)(rank + 1 + lane) & 7;
    BAR();
    int it = 0;
    auto bars = [&](int n) { for (int e = min(it + n, NJ); it < e; ++it) BAR(); };
    const int ch = b >> 6;
    const unsigned tn = now32() + XDELAY + (unsigned)b * TAU_X;
    while ((int)(now32() - tn) < 0 && it < 36 + 16 * ch) { BAR(); ++it; }
    __builtin_amdgcn_s_waitcnt(0x0F70);
    const unsigned call = rfl(done) / NWG + 1;
    char* pws = p.ws[0];
#pragma unroll
    for (int q = 1; q < NR; ++q) pws = peer == (unsigned)q ? p.ws[q] : pws;
    const size_t so = WS_XBUF + (call & 1) * XPAR + (size_t)rank * XTILE + (size_t)b * 4096;
#pragma unroll
    for (int i = 0; i < 4; ++i)
#pragma unroll
      for (int q = 0; q < 7; ++q) st16_sys<0>(p.ws[(rank + 1 + q) & 7] + so + i * 1024, (unsigned)lane * 16u, xv[i]);
    TS(9);
    bars(5);
    wait_vm<0>();
    unsigned tick; unsigned long long sv;
#define ATOM0(OP, ADDR, VAL) asm volatile("s_mov_b64 %1, exec\n\ts_mov_b64 exec, 1\n\t" OP " %0, %2, %3, off sc0 sc1\n\ts_mov_b64 exec, %1" \
                                          : "=&v"(tick), "=&s"(sv) : "v"(ADDR), "v"(VAL) : "memory")
    ATOM0("global_atomic_add", myws + WS_XLOC + ch * 128, 1u);
    bars(3);
    asm volatile("s_waitcnt vmcnt(0)" : "+v"(tick) :: "memory");
    if (rfl(tick) == call * 64 - 1) {
      if (lane < 7) __hip_atomic_store((unsigned*)(pws + WS_XCNT + rank * 512 + ch * 128), call, __ATOMIC_RELAXED, __HIP_MEMORY_SCOPE_SYSTEM);
    }
    if (lane == 0) __hip_atomic_fetch_add((unsigned*)(myws + WS_DONE), 1u, __ATOMIC_RELAXED, __HIP_MEMORY_SCOPE_AGENT);
    TS(5);
    bars(n0 + n1 + kb0 + ODRAIN + 1 - it);
    ATOM0("global_atomic_add", myws + WS_ARR + xcc * 128, 1u);
    const unsigned cen = __hip_atomic_load((unsigned*)(myws + WS_CEN + (lane & 15) * 128), __ATOMIC_RELAXED, __HIP_MEMORY_SCOPE_AGENT);
    bars(2);
    asm volatile("s_waitcnt vmcnt(0)" : "+v"(tick) :: "memory");
    unsigned total = 0;
#pragma unroll
    for (int k = 0; k < 16; ++k) total += (unsigned)__builtin_amdgcn_readlane((int)cen, k);
    const unsigned t = rfl(tick) + 1;
    if (total != call * NWG || t == (unsigned)__builtin_amdgcn_readlane((int)cen, (int)xcc)) {
      asm volatile("buffer_wbl2 sc0 sc1" ::: "memory");
      bars(4);
      wait_vm<0>();
      ATOM0("global_atomic_umax", myws + WS_ACK + xcc * 128, t);
      bars(2);
      asm volatile("s_waitcnt vmcnt(0)" : "+v"(tick) :: "memory");
      const unsigned old = rfl(tick);
      if (t > old && lane < 7) __hip_atomic_fetch_add((unsigned*)(pws + WS_O0CNT + rank * 128), t - old, __ATOMIC_RELAXED, __HIP_MEMORY_SCOPE_SYSTEM);
    }
    TS(7);
    if (b == NWG - 1) {
      bars(NJ);
      const unsigned d = 7 - (lane & 7), n2nd = 3 * ((111 - (d - 1)) / 7 - (84 - (d - 1)) / 7);
      wait_ge((const unsigned*)(myws + (lane < 8 ? WS_OUTCNT : WS_O0CNT) + peer * 128), lane < 8 ? call * 4 * n2nd : call * NWG, tmo, call, lane < 15 && lane != 7, mark, lane);
      TS(8);
    }
  } else if (wid >= 4) {
    const int pw = wid - 4;
    unsigned done0;
    asm volatile("global_load_dword %0, %1, off sc1" : "=v"(done0) : "v"(myws + WS_DONE) : "memory");
    const auto srdB = __builtin_amdgcn_make_buffer_rsrc((void*)p.w, 0, KDIM * NDIM * 2, 0x00020000);
    unsigned voffA[2], voffB[2];
#pragma unroll
    for (int i = 0; i < 2; ++i) {
      voffA[i] = (unsigned)(lane >> 3) * 8192u + (unsigned)(((lane & 7) ^ ((4 * i + (lane >> 4)) & 7)) * 16);
      voffB[i] = (unsigned)(lane >> 4) * 32768u + (unsigned)(((lane & 15) ^ (((lane >> 4) << 2) | (i << 1))) * 16);
    }
    unsigned sA = (unsigned)(64 * pw) * 8192u, stepK = 128u, stepI = 65536u;
    unsigned sB, G, kv, kp, kb;
    auto srdA = __builtin_amdgcn_make_buffer_rsrc((void*)p.x, 0, (int)XTILE, 0x00020000);
    auto begin = [&](unsigned d, unsigned k0) {
      srdA = __builtin_amdgcn_make_buffer_rsrc((void*)(p.x + (size_t)(d & 3) * XTILE), 0, (int)XTILE, 0x00020000);
      sB = (unsigned)(32 * pw) * 32768u + ((((unsigned)rank + ((d >> 2) & 7)) & 7) * 16 + ((d >> 5) & 15)) * 256u;
      G = ((d >> 9) & 3) + 1; kv = (d >> 11) & 3; kp = 0; kb = k0;
    };
    unsigned soA, soB, ldsb = (unsigned)pw * 8192u;
    auto next = [&]() {
      const unsigned kt = min(kb + kv, 63u);
      soA = sA + kt * stepK; soB = sB + kt * (64u * 32768u);
      kv = kv + 1 == G ? 0 : kv + 1; ++kp;
      if (kp == G) { kp = 0; kb += G; }
    };
    auto issue = [&]() {
      LAS char* d = lds + ldsb;
#pragma unroll
      for (int i = 0; i < 8; ++i) __builtin_amdgcn_raw_ptr_buffer_load_lds(srdA, (LAS void*)(d + i * 1024), 16, voffA[i & 1], soA + i * stepI, 0, 17);
#pragma unroll
      for (int i = 0; i < 8; ++i) __builtin_amdgcn_raw_ptr_buffer_load_lds(srdB, (LAS void*)(d + NS * SLOT + i * 1024), 16, voffB[(i >> 1) & 1], soB + i * (4u * 32768u), 0, 0);
      ldsb = ldsb + SLOT >= NS * SLOT ? ldsb + SLOT - NS * SLOT : ldsb + SLOT;
    };
    static_assert(HK == 0 || HK > NS, "the prologue stays inside the first piece");
    begin(helper ? aux : desc, NKT - n1);
    next(); issue(); next(); issue();
    asm volatile("s_waitcnt vmcnt(16)" : "+v"(done0) :: "memory");
    BAR();
#pragma unroll
    for (int i = 2; i < NS; ++i) { next(); issue(); }
    next();
    unsigned tpace = now32() - PHEAD;
    const unsigned pace1 = owner ? PACE1 : helper ? PACE1H : PACE1N;
    for (int j = 0; j < NJ - NS; ++j) {
      tpace += j < PSW ? pace1 : PACE2;
      while ((int)(now32() - tpace) < 0) __builtin_amdgcn_s_sleep(1);
      wait_vm<48>(); BAR();
      issue();
      SCHED();
      if (j == n0 - NS - 1) begin(desc, 0);
      if (j == J0 - 1) {
        begin(desc >> 16, 0);
        if (T1.src != rank) {
          srdA = __builtin_amdgcn_make_buffer_rsrc((void*)(myws + WS_XBUF + ((rfl(done0) / NWG + 1) & 1) * XPAR + (size_t)T1.src * XTILE), 0, (int)XTILE, 0x00020000);
          sA = (unsigned)pw * 8192u; stepK = 16384u; stepI = 1024u; voffA[0] = voffA[1] = (unsigned)lane * 16u;
        }
      }
      next();
    }
    wait_vm<48>(); BAR();
    wait_vm<32>(); BAR();
    wait_vm<16>(); BAR();
    wait_vm<0>(); BAR();
    BAR();
  } else {
    const int wr = wid >> 1, wc = wid & 1, fr = lane & 15, fq = lane >> 4;
    const unsigned baseA0 = (unsigned)(wr * 64 + fr) * 128u + (unsigned)((fq ^ ((fr >> 1) & 7)) * 16);
    unsigned baseB[4];
    { const int q = (lane & 15) >> 2, pp = lane & 3; const int s = (q << 2) | ((fq & 1) << 1);
#pragma unroll
      for (int n = 0; n < 4; ++n) baseB[n] = (unsigned)(NS * SLOT) + (unsigned)(8 * fq + q) * 256u + (unsigned)(((8 * wc + 2 * n) ^ s) * 16 + 8 * pp); }
    f32x4 acc[4][4];
    bf16x8 FA[2][4], FB[2][4];
    u32x4 O[8];
    typedef LAS char* lptr;
    auto rd = [&](bf16x8 (&A)[4], bf16x8 (&B)[4], lptr pa, lptr (&pb)[4], int ks) {
      auto rb = [&](int n) {
        bf16x4 lo = __builtin_amdgcn_ds_read_tr16_b64_v4i16((LAS bf16x4*)(pb[n] + ks * 8192));
        bf16x4 hi = __builtin_amdgcn_ds_read_tr16_b64_v4i16((LAS bf16x4*)(pb[n] + ks * 8192 + 1024));
        B[n] = __builtin_shufflevector(lo, hi, 0, 1, 2, 3, 4, 5, 6, 7);
      };
      rb(0);
#pragma unroll
      for (int m = 0; m < 4; ++m) A[m] = *(const LAS bf16x8*)(pa + m * 2048);
      rb(1); rb(2); rb(3);
    };
    auto mm = [&](bf16x8 (&A)[4], bf16x8 (&B)[4], int lo, int hi) {
#pragma unroll
      for (int i = lo; i < hi; ++i) {
        const int n = i >> 2, m = (n & 1) ? 3 - (i & 3) : (i & 3);
        acc[m][n] = __builtin_amdgcn_mfma_f32_16x16x32_bf16(B[n], A[m], acc[m][n], 0, 0, 0);
      }
    };
    auto pairs = [&](int n) {
#pragma unroll
      for (int i = 0; i < n; ++i) { __builtin_amdgcn_sched_group_barrier(0x8, 1, 0); __builtin_amdgcn_sched_group_barrier(0x100, 1, 0); }
    };
    auto out_ptr = [&](const Tile& T) {
      char* o = p.out[T.dest] + (size_t)(T.src * MLOC + T.rt * 128 + wr * 64) * (NSH * 2) + (T.ct * 128 + wc * 64) * 2;
      asm volatile("" : "+s"(o));
      return o;
    };
    char* const o0 = out_ptr(T0); char* const o1 = out_ptr(T1);
    unsigned* cnt1 = (unsigned*)(p.ws[T1.dest] + WS_OUTCNT + rank * 128);
    asm volatile("" : "+s"(cnt1));
    const unsigned vo = (unsigned)(fr & 7) * (NSH * 2) + 64u * (fr >> 3) + 16u * ((fq >> 1) + 2 * (fq & 1));
    auto pack1 = [&](int m, int q) {
      const unsigned x0 = gelu2(acc[m][2 * q][0], acc[m][2 * q][1]), x1 = gelu2(acc[m][2 * q][2], acc[m][2 * q][3]);
      const unsigned y0 = gelu2(acc[m][2 * q + 1][0], acc[m][2 * q + 1][1]), y1 = gelu2(acc[m][2 * q + 1][2], acc[m][2 * q + 1][3]);
      const auto r0 = __builtin_amdgcn_permlane16_swap(x0, y0, false, false);
      const auto r1 = __builtin_amdgcn_permlane16_swap(x1, y1, false, false);
      return (u32x4){r0[0], r1[0], r0[1], r1[1]};
    };
    auto pack2 = [&](int m, u32x4& lo, u32x4& hi) {
      const u32x4 a = pack1(m, 0), c = pack1(m, 1);
#pragma unroll
      for (int e = 0; e < 4; ++e) {
        lo[e] = (unsigned)__builtin_amdgcn_update_dpp((int)a[e], (int)c[e], 0x128, 0xf, 0xc, false);
        hi[e] = (unsigned)__builtin_amdgcn_update_dpp((int)c[e], (int)a[e], 0x128, 0xf, 0x3, false);
      }
    };
    auto signal = [&](unsigned* cnt) {
      wait_vm<0>();
      if (lane == 0) __hip_atomic_fetch_add(cnt, 1u, __ATOMIC_RELAXED, __HIP_MEMORY_SCOPE_SYSTEM);
    };
    unsigned so = 0;
    lptr pB[4], pA0 = lds + baseA0, pA1 = lds + (baseA0 ^ 64u);
#pragma unroll
    for (int n = 0; n < 4; ++n) pB[n] = lds + baseB[n];
#define PART1() do { SCHED(); rd(FA[1], FB[1], pA1, pB, 1); mm(FA[0], FB[0], 0, 16); pairs(12); __builtin_amdgcn_sched_group_barrier(0x8, 4, 0); SCHED(); } while (0)
#define PART2() do { \
      so = so + SLOT == NS * SLOT ? 0 : so + SLOT; \
      pA0 = lds + baseA0 + so; pA1 = lds + (baseA0 ^ 64u) + so; \
      _Pragma("unroll") for (int n = 0; n < 4; ++n) pB[n] = lds + baseB[n] + so; \
      mm(FA[1], FB[1], 0, 4); \
      asm volatile("" : "+v"(pA0), "+v"(pA1), "+v"(pB[0]), "+v"(pB[1]), "+v"(pB[2]), "+v"(pB[3])); \
      _Pragma("unroll") for (int i = 0; i < 3; ++i) { __builtin_amdgcn_sched_group_barrier(0x8, 1, 0); __builtin_amdgcn_sched_group_barrier(0x2, 2, 0); } \
      __builtin_amdgcn_sched_group_barrier(0x8, 1, 0); \
      SCHED(); __builtin_amdgcn_s_waitcnt(0xC07F); __builtin_amdgcn_s_barrier(); SCHED(); \
      rd(FA[0], FB[0], pA0, pB, 0); mm(FA[1], FB[1], 4, 16); pairs(12); SCHED(); } while (0)
    char* part = myws + WS_PART + (size_t)pidx * PART + wid * 16384;
    if (wid == 0) TS(0);
    BAR();
    if (wid == 0) TS(1);
    rd(FA[0], FB[0], pA0, pB, 0);
    f32x4 H[16];
    int hk = 99;
#pragma unroll 1
    for (int seg = helper ? 0 : 1; seg < 3; ++seg) {
      if (seg < 2) {
#pragma unroll
        for (int m = 0; m < 4; ++m)
#pragma unroll
          for (int n = 0; n < 4; ++n) acc[m][n] = (f32x4){0.f, 0.f, 0.f, 0.f};
      }
      const int nk = seg == 0 ? HK : seg == 1 ? n1 - PRELD : PRELD;
#pragma unroll 1
      for (int kt = 0; kt < nk; ++kt) {
        PART1();
        if (hk <= HDRAIN) {
#define HST(s) case s: st16_sys<1024>(part + (14 - 2 * s) * 1024, (unsigned)lane * 16u, __builtin_bit_cast(u32x4, H[15 - 2 * s])); \
                       st16_sys<0>(part + (14 - 2 * s) * 1024, (unsigned)lane * 16u, __builtin_bit_cast(u32x4, H[14 - 2 * s])); break;
          switch (hk) { HST(0) HST(1) HST(2) HST(3) HST(4) HST(5) HST(6) HST(7) case HDRAIN: wait_vm<0>(); break; default: break; }
          ++hk;
        }
        PART2();
      }
      if (seg == 0) {
#pragma unroll
        for (int i = 0; i < 16; ++i) H[i] = acc[i >> 2][i & 3];
        hk = 0;
      } else if (seg == 1) {
        const auto srdP = __builtin_amdgcn_make_buffer_rsrc((void*)part, 0, owner ? 16384 : 0, 0x00020000);
#pragma unroll
        for (int i = 0; i < 16; ++i) H[i] = __builtin_bit_cast(f32x4, __builtin_amdgcn_raw_buffer_load_b128(srdP, lane * 16 + (i & 3) * 1024, (i >> 2) * 4096, 16));
      }
    }
    if (wid == 0) TS(2);
#pragma unroll
    for (int i = 0; i < 16; ++i) acc[i >> 2][i & 3] += H[i];
#pragma unroll
    for (int m = 0; m < 4; ++m) pack2(m, O[2 * m], O[2 * m + 1]);
#pragma unroll
    for (int m = 0; m < 4; ++m)
#pragma unroll
      for (int n = 0; n < 4; ++n) acc[m][n] = (f32x4){0.f, 0.f, 0.f, 0.f};
    auto store0 = [&](int m) {
      switch (m) {
        case 0: st16_l2(o0, vo, O[0]); st16_l2(o0 + 32768, vo, O[1]); break;
        case 1: st16_l2(o0 + 65536, vo, O[2]); st16_l2(o0 + 65536 + 32768, vo, O[3]); break;
        case 2: st16_l2(o0 + 2 * 65536, vo, O[4]); st16_l2(o0 + 2 * 65536 + 32768, vo, O[5]); break;
        default: st16_l2(o0 + 3 * 65536, vo, O[6]); st16_l2(o0 + 3 * 65536 + 32768, vo, O[7]); break;
      }
    };
    __builtin_amdgcn_s_waitcnt(0x0F70);
#pragma unroll 1
    for (int kt = -kb0; kt < n2 - kb0; ++kt) {
      PART1();
      if ((unsigned)kt <= (unsigned)ODRAIN) {
        if (kt < 4) store0(kt);
        else if (kt == ODRAIN) wait_vm<0>();
      }
      PART2();
    }
    if (wid == 0) TS(3);
#pragma unroll
    for (int m = 0; m < 4; ++m) {
      u32x4 lo, hi; pack2(m, lo, hi);
      if (T1.dest != rank) wait_vm<1>();
      st16_sys<0>(o1 + m * 65536, vo, lo);
      if (T1.dest != rank) wait_vm<1>();
      st16_sys<0>(o1 + m * 65536 + 32768, vo, hi); SCHED();
    }
    if (T1.dest != rank) signal(cnt1);
  }
}

extern "C" void kernel_launch(int rank, int nranks, ncclComm_t comm, hipStream_t stream,
                              void* const* d_in, const int* in_sizes, int n_in,
                              void* d_out, int out_size, const int* in_dtypes, int out_dtype,
                              void* const* d_out_all, void* const* d_ws_all, size_t ws_bytes) {
  if (nranks != NR || ws_bytes < WS_END || (unsigned)rank >= (unsigned)NR) { fprintf(stderr, "unsupported configuration\n"); return; }
  HIP_CHECK(hipFuncSetAttribute((const void*)gemm_a2a_kernel, hipFuncAttributeMaxDynamicSharedMemorySize, LDS_BYTES));
  Params p;
  p.x = (const char*)d_in[0]; p.w = (const char*)d_in[1]; p.rank = rank; p.pad = 0;
  for (int i = 0; i < NR; ++i) { p.out[i] = (char*)d_out_all[i]; p.ws[i] = (char*)d_ws_all[i]; }
  build_desc(p.desc);
  gemm_a2a_kernel<<<NWG, 512, LDS_BYTES, stream>>>(p);
  HIP_CHECK(hipGetLastError());
}
```

```cpp
#include <hip/hip_runtime.h>
#include <rccl/rccl.h>
#include <cstdio>

#define HIP_CHECK(expr) do { hipError_t _e = (expr); if (_e != hipSuccess) { \
    fprintf(stderr, "HIP error %s at %s:%d\n", hipGetErrorString(_e), __FILE__, __LINE__); \
    return; } } while (0)

#define LAS __attribute__((address_space(3)))
typedef short bf16x8 __attribute__((ext_vector_type(8)));
typedef short bf16x4 __attribute__((ext_vector_type(4)));
typedef float f32x4 __attribute__((ext_vector_type(4)));
typedef float f32x2 __attribute__((ext_vector_type(2)));
typedef unsigned u32x4 __attribute__((ext_vector_type(4)));
typedef __bf16 bfx2 __attribute__((ext_vector_type(2)));

#define TAU_O 66
#define TAU_X 8
#define XDELAY 100
#define HK 12
#define PACE1 60
#define PACE1H 57
#define PACE1N 61
#define PEST 30
#define PSW 64
#define PHEAD 500
#define PACE2 42
constexpr int NR = 8;
constexpr int KDIM = 4096, NDIM = 16384, MLOC = 512, NSH = 2048;
constexpr int NKT = 64, SLOT = 16384, NS = 5, NWG = 256;
constexpr int LDS_BYTES = 2 * NS * SLOT;
constexpr int HDRAIN = 12, ODRAIN = 5, PRELD = 3;
constexpr int Q4 = 80;
constexpr size_t WS_DONE = 0, WS_TMO = 128, WS_OUTCNT = 4096, WS_XCNT = 8192, WS_O0CNT = 12288, WS_XLOC = 14336, WS_CEN = 16384, WS_ARR = 20480, WS_ACK = 24576, WS_PFLAG = 32768, WS_STAMP = 65536;
constexpr size_t XTILE = 128 * KDIM * 2;
constexpr size_t WS_XBUF = 1u << 20, XPAR = NR * XTILE;
constexpr size_t WS_PART = WS_XBUF + 2 * XPAR, PART = 128 * 128 * 4, WS_END = WS_PART + 96 * PART;
constexpr unsigned long long DEADLINE = 40000000ull;

enum { ROLE_OWNER = 1, ROLE_HELPER = 2 };
struct Params { const char* x; const char* w; char* out[NR]; char* ws[NR]; int rank; int pad; unsigned desc[NWG][2]; };
struct Tile { int src; int rt; int dest; int ct; };
__device__ __forceinline__ Tile tile_of(unsigned d, int rank) {
  Tile r; r.src = (rank + (int)((d >> 13) & 7)) & 7; r.rt = d & 3; r.dest = (rank + (int)((d >> 2) & 7)) & 7; r.ct = (d >> 5) & 15; return r;
}
static inline unsigned tile_code(int so, int rt, int dof, int ct, int G, int rot) { return (unsigned)(rt | dof << 2 | ct << 5 | (G - 1) << 9 | (rot % G) << 11 | so << 13); }
static void build_desc(unsigned (*out)[2]) {
  unsigned desc[NWG][2];
  unsigned T[2][8][32]; int n[2][8] = {};
  auto put = [&](int w, int x, unsigned c) { T[w][x][n[w][x]++] = c; };
  auto code = [&](int q, int rt) { return tile_code(0, rt, 1 + q % 7, q / 7, q == Q4 ? 4 : 3, rt + q); };
  for (int k = 0; k < 80; ++k) for (int rt = 0; rt < 3; ++rt) put(0, k / 10, code(k / 10 + 8 * (k % 10), rt));
  for (int i = 0; i < 16; ++i) put(0, i / 2, code(i < 12 ? 81 + i / 3 : Q4, i < 12 ? i % 3 : i - 12));
  for (int k = 0; k < 24; ++k) for (int rt = 0; rt < 3; ++rt) put(1, k / 6, code(85 + k / 6 + 4 * (k % 6), rt));
  for (int x = 0; x < 4; ++x) for (int c = 0; c < 4; ++c) for (int r = 0; r < 3; ++r) put(1, x, tile_code(0, r, 0, x + 4 * c, 3, r + c));
  for (int k = 0; k < 3; ++k) { put(1, k, code(109 + k, 0)); put(1, k, code(109 + k, 1)); }
  put(1, 3, code(109, 2)); put(1, 3, code(110, 2));
  for (int g = 0; g < 2; ++g) for (int h = 0; h < 2; ++h) {
    for (int c = 0; c < 8; ++c) for (int r = 0; r < 4; ++r)
      if (!(4 * g + r == 8 - (1 + Q4 % 7) && 8 * h + c == Q4 / 7)) put(1, 4 + 2 * g + h, tile_code(4 * g + r, 3, 0, 8 * h + c, 4, r + c + 2 * h + g));
    if (n[1][4 + 2 * g + h] < 32) put(1, 4 + 2 * g + h, code(111, 2));
  }
  int role[NWG], own[96], nown = 0;
  for (int b = 0; b < NWG; ++b) { desc[b][0] = T[0][b & 7][b >> 3] | T[1][b & 7][b >> 3] << 16; desc[b][1] = 0; role[b] = 0; }
  for (int pass = 0; pass < 2; ++pass) for (int x = 0; x < 4; ++x) for (int i = pass ? 30 : 0; i < (pass ? 32 : 18); ++i) own[nown++] = i * 8 + x;
  for (int k = 0; HK && k < nown; ++k) {
    const int o = own[k], h = (k % 30) * 8 + 4 + k / 30;
    desc[o][1] = (unsigned)k << 16 | 1u << 14; role[o] = ROLE_OWNER;
    desc[h][1] = (desc[o][0] & 0x1fff) | (unsigned)k << 16; role[h] = ROLE_HELPER;
  }
  auto sched = [](int r, int j) { const int p1 = r == ROLE_OWNER ? PACE1 : r == ROLE_HELPER ? PACE1H : PACE1N; return j < PSW ? j * p1 : PSW * p1 + (j - PSW) * PACE2; };
  auto turn_kt = [&](int k, int d, int r) {
    const int t = sched(ROLE_OWNER, NKT - HK) + k * TAU_O + d * (TAU_O / 8), j1 = r == ROLE_OWNER ? NKT - HK : r == ROLE_HELPER ? NKT + HK : NKT;
    if (t < sched(r, j1)) return -1;
    int j = j1; while (sched(r, j) < t) ++j;
    return j - j1 < NKT - 12 ? j - j1 : NKT - 12;
  };
  bool sent[NWG] = {};
  int load[64][8] = {};
  for (int k = 0, left = NWG; left; ++k)
    for (int dd = 0; dd < 7; ++dd) {
      const int d = 1 + (dd + k) % 7;
      int best = -1, bc = 1 << 30;
      for (int b = 0; b < NWG; ++b) {
        if (sent[b] || (int)((desc[b][0] >> 2) & 7) != d) continue;
        if (turn_kt(k, d, role[b]) < 0) continue;
        const int x = b & 7, c = 8 * (2 * load[k][x] + (k > 0 ? load[k - 1][x] : 0) + (k > 1 ? load[k - 2][x] : 0)) + 12 * (role[b] == ROLE_OWNER ? 0 : role[b] == ROLE_HELPER ? 1 : 2);
        if (c < bc) { bc = c; best = b; }
      }
      if (best >= 0) { sent[best] = true; --left; load[k][best & 7]++; desc[best][1] |= (unsigned)turn_kt(k, d, role[best]) << 24; }
    }
  for (int b = 0; b < NWG; ++b) { const int l = (b & ~7) | (b & 1) << 2 | (b & 7) >> 1; out[b][0] = desc[l][0]; out[b][1] = desc[l][1]; }
}

__device__ __forceinline__ unsigned gelu2(float y0, float y1) {
  const float k0 = -2.0f * 0.7978845608028654f * 1.4426950408889634f, k1 = k0 * 0.044715f;
  const f32x2 y = {y0, y1};
  const f32x2 z = y * __builtin_elementwise_fma(y * y, (f32x2){k1, k1}, (f32x2){k0, k0});
  const f32x2 d = (f32x2){__builtin_amdgcn_exp2f(z.x), __builtin_amdgcn_exp2f(z.y)} + 1.0f;
  const f32x2 g = y * (f32x2){__builtin_amdgcn_rcpf(d.x), __builtin_amdgcn_rcpf(d.y)};
  return __builtin_bit_cast(unsigned, __builtin_convertvector(g, bfx2));
}
__device__ __forceinline__ unsigned rfl(unsigned v) { return (unsigned)__builtin_amdgcn_readfirstlane((int)v); }
__device__ __forceinline__ unsigned now32() { return (unsigned)__builtin_amdgcn_s_memrealtime(); }
template <int N> __device__ __forceinline__ void wait_vm() { asm volatile("s_waitcnt vmcnt(%0)" :: "i"(N) : "memory"); }
template <int OFF> __device__ __forceinline__ void st16_sys(char* sbase, unsigned voff, u32x4 v) {
  asm volatile("global_store_dwordx4 %0, %1, %2 offset:%3 sc0 sc1\n\ts_nop 1" :: "v"(voff), "v"(v), "s"(sbase), "i"(OFF) : "memory");
}
__device__ __forceinline__ void st16_l2(char* sbase, unsigned voff, u32x4 v) {
  asm volatile("global_store_dwordx4 %0, %1, %2\n\ts_nop 1" :: "v"(voff), "v"(v), "s"(sbase) : "memory");
}
#define BAR() do { asm volatile("" ::: "memory"); __builtin_amdgcn_s_barrier(); asm volatile("" ::: "memory"); } while (0)
#define SCHED() __builtin_amdgcn_sched_barrier(0)
#define TS(k) do {} while (0)

__device__ __forceinline__ void wait_ge(const unsigned* flag, unsigned want, unsigned* tmo, unsigned call, bool active, char* mark, int lane) {
  const unsigned long long t0 = __builtin_amdgcn_s_memrealtime();
  for (unsigned n = 1;; ++n) {
    const bool ok = !active || __hip_atomic_load(flag, __ATOMIC_RELAXED, __HIP_MEMORY_SCOPE_SYSTEM) >= want;
    if (__all(ok)) return;
    if ((n & 15) == 0 &&
        (__hip_atomic_load(tmo, __ATOMIC_RELAXED, __HIP_MEMORY_SCOPE_AGENT) == call || __builtin_amdgcn_s_memrealtime() - t0 > DEADLINE)) {
      __hip_atomic_store(tmo, call, __ATOMIC_RELAXED, __HIP_MEMORY_SCOPE_AGENT);
      if (lane == 0) __hip_atomic_store((unsigned*)mark, 0x7fc07fc0u, __ATOMIC_RELAXED, __HIP_MEMORY_SCOPE_SYSTEM);
      return;
    }
    __builtin_amdgcn_s_sleep(1);
  }
}

__global__ void __launch_bounds__(512, 2) gemm_a2a_kernel(Params p) {
  extern __shared__ __attribute__((aligned(16))) char smem[];
  LAS char* lds = (LAS char*)smem;
  const int tid = threadIdx.x, wid = (int)rfl(tid >> 6), lane = tid & 63;
  const int b = blockIdx.x, rank = p.rank;
  const unsigned desc = p.desc[b][0], aux = p.desc[b][1];
  const Tile T0 = tile_of(desc, rank), T1 = tile_of(desc >> 16, rank);
  const bool owner = (aux >> 14) & 1, helper = (aux & 0x1fffu) != 0;
  const int n0 = helper ? HK : 0, n1 = owner ? NKT - HK : NKT, n2 = NKT;
  const int NJ = n0 + n1 + n2;
  const int J0 = n0 + n1 - NS;
  char* myws = p.ws[rank];
  unsigned* tmo = (unsigned*)(myws + WS_TMO);
  char* mark = p.out[rank] + (size_t)rank * MLOC * NSH * 2;
  const unsigned pidx = (aux >> 16) & 127;
  const int kb0 = (int)(aux >> 24);

  if (wid == 7) {
    const bool staged = T1.src != rank;
    if (!staged && !owner && !helper) return;
    const unsigned call = rfl(__hip_atomic_load((unsigned*)(myws + WS_DONE), __ATOMIC_RELAXED, __HIP_MEMORY_SCOPE_AGENT)) / NWG + 1;
    unsigned* pf = (unsigned*)(myws + WS_PFLAG + pidx * 128);
    BAR();
    int it = 0;
    if (helper) {
      for (; it <= HK + HDRAIN; ++it) BAR();
      if (lane == 0) __hip_atomic_fetch_add(pf, 1u, __ATOMIC_RELAXED, __HIP_MEMORY_SCOPE_AGENT);
    }
    if (owner) {
      for (; it < n1 - PRELD - 6; ++it) BAR();
      const unsigned seen = __hip_atomic_load(pf, __ATOMIC_RELAXED, __HIP_MEMORY_SCOPE_AGENT);
      for (; it < n1 - PRELD - 1; ++it) BAR();
      if (rfl(seen) < call) wait_ge(pf, call, tmo, call, true, mark, lane);
    }
    if (staged) {
      const unsigned* fl = (const unsigned*)(myws + WS_XCNT + T1.src * 512);
#pragma unroll 1
      for (int c = 0; c < 4; ++c) {
        const int jc = J0 + (c ? 16 * c - 3 : 0);
        for (; it < jc - 5; ++it) BAR();
        const unsigned seen = __hip_atomic_load(fl + c * 32, __ATOMIC_RELAXED, __HIP_MEMORY_SCOPE_SYSTEM);
        for (; it < jc; ++it) BAR();
        if (rfl(seen) < call) wait_ge(fl + c * 32, call, tmo, call, true, mark, lane);
        if (c == 0) TS(6);
      }
    }
  } else if (wid == 6) {
    u32x4 xv[4];
#pragma unroll
    for (int i = 0; i < 4; ++i) {
      const int P = b * 4 + i;
      xv[i] = *(const u32x4*)(p.x + 3 * XTILE + (size_t)((P & 15) * 8 + (lane >> 3)) * 8192 + (P >> 4) * 128 + ((lane & 7) ^ ((4 * (P & 1) + (lane >> 4)) & 7)) * 16);
    }
    const unsigned done = __hip_atomic_load((unsigned*)(myws + WS_DONE), __ATOMIC_RELAXED, __HIP_MEMORY_SCOPE_AGENT);
    const unsigned xcc = (unsigned)__builtin_amdgcn_s_getreg((3 << 11) | 20) & 15u;
    if (lane == 0) __hip_atomic_fetch_add((unsigned*)(myws + WS_CEN + xcc * 128), 1u, __ATOMIC_RELAXED, __HIP_MEMORY_SCOPE_AGENT);
    const unsigned peer = (unsigned)(rank + 1 + lane) & 7;
    BAR();
    int it = 0;
    auto bars = [&](int n) { for (int e = min(it + n, NJ); it < e; ++it) BAR(); };
    const int ch = b >> 6;
    const unsigned tn = now32() + XDELAY + (unsigned)b * TAU_X;
    while ((int)(now32() - tn) < 0 && it < 36 + 16 * ch) { BAR(); ++it; }
    __builtin_amdgcn_s_waitcnt(0x0F70);
    const unsigned call = rfl(done) / NWG + 1;
    char* pws = p.ws[0];
#pragma unroll
    for (int q = 1; q < NR; ++q) pws = peer == (unsigned)q ? p.ws[q] : pws;
    const size_t so = WS_XBUF + (call & 1) * XPAR + (size_t)rank * XTILE + (size_t)b * 4096;
#pragma unroll
    for (int i = 0; i < 4; ++i)
#pragma unroll
      for (int q = 0; q < 7; ++q) st16_sys<0>(p.ws[(rank + 1 + q) & 7] + so + i * 1024, (unsigned)lane * 16u, xv[i]);
    TS(9);
    bars(5);
    wait_vm<0>();
    unsigned tick; unsigned long long sv;
#define ATOM0(OP, ADDR, VAL) asm volatile("s_mov_b64 %1, exec\n\ts_mov_b64 exec, 1\n\t" OP " %0, %2, %3, off sc0 sc1\n\ts_mov_b64 exec, %1" \
                                          : "=&v"(tick), "=&s"(sv) : "v"(ADDR), "v"(VAL) : "memory")
    ATOM0("global_atomic_add", myws + WS_XLOC + ch * 128, 1u);
    bars(3);
    asm volatile("s_waitcnt vmcnt(0)" : "+v"(tick) :: "memory");
    if (rfl(tick) == call * 64 - 1) {
      if (lane < 7) __hip_atomic_store((unsigned*)(pws + WS_XCNT + rank * 512 + ch * 128), call, __ATOMIC_RELAXED, __HIP_MEMORY_SCOPE_SYSTEM);
    }
    if (lane == 0) __hip_atomic_fetch_add((unsigned*)(myws + WS_DONE), 1u, __ATOMIC_RELAXED, __HIP_MEMORY_SCOPE_AGENT);
    TS(5);
    bars(n0 + n1 + kb0 + ODRAIN + 1 - it);
    ATOM0("global_atomic_add", myws + WS_ARR + xcc * 128, 1u);
    const unsigned cen = __hip_atomic_load((unsigned*)(myws + WS_CEN + (lane & 15) * 128), __ATOMIC_RELAXED, __HIP_MEMORY_SCOPE_AGENT);
    bars(2);
    asm volatile("s_waitcnt vmcnt(0)" : "+v"(tick) :: "memory");
    unsigned total = 0;
#pragma unroll
    for (int k = 0; k < 16; ++k) total += (unsigned)__builtin_amdgcn_readlane((int)cen, k);
    const unsigned t = rfl(tick) + 1;
    if (total != call * NWG || t == (unsigned)__builtin_amdgcn_readlane((int)cen, (int)xcc)) {
      asm volatile("buffer_wbl2 sc0 sc1" ::: "memory");
      bars(4);
      wait_vm<0>();
      ATOM0("global_atomic_umax", myws + WS_ACK + xcc * 128, t);
      bars(2);
      asm volatile("s_waitcnt vmcnt(0)" : "+v"(tick) :: "memory");
      const unsigned old = rfl(tick);
      if (t > old && lane < 7) __hip_atomic_fetch_add((unsigned*)(pws + WS_O0CNT + rank * 128), t - old, __ATOMIC_RELAXED, __HIP_MEMORY_SCOPE_SYSTEM);
    }
    TS(7);
    if (b == NWG - 1) {
      bars(NJ);
      const unsigned d = 7 - (lane & 7), n2nd = 3 * ((111 - (d - 1)) / 7 - (84 - (d - 1)) / 7);
      wait_ge((const unsigned*)(myws + (lane < 8 ? WS_OUTCNT : WS_O0CNT) + peer * 128), lane < 8 ? call * 4 * n2nd : call * NWG, tmo, call, lane < 15 && lane != 7, mark, lane);
      TS(8);
    }
  } else if (wid >= 4) {
    const int pw = wid - 4;
    unsigned done0;
    asm volatile("global_load_dword %0, %1, off sc1" : "=v"(done0) : "v"(myws + WS_DONE) : "memory");
    const auto srdB = __builtin_amdgcn_make_buffer_rsrc((void*)p.w, 0, KDIM * NDIM * 2, 0x00020000);
    unsigned voffA[2], voffB[2];
#pragma unroll
    for (int i = 0; i < 2; ++i) {
      voffA[i] = (unsigned)(lane >> 3) * 8192u + (unsigned)(((lane & 7) ^ ((4 * i + (lane >> 4)) & 7)) * 16);
      voffB[i] = (unsigned)(lane >> 4) * 32768u + (unsigned)(((lane & 15) ^ (((lane >> 4) << 2) | (i << 1))) * 16);
    }
    unsigned sA = (unsigned)(64 * pw) * 8192u, stepK = 128u, stepI = 65536u;
    unsigned sB, tab, pos;
    auto srdA = __builtin_amdgcn_make_buffer_rsrc((void*)p.x, 0, (int)XTILE, 0x00020000);
    auto begin = [&](unsigned d, unsigned k0, unsigned k1) {
      srdA = __builtin_amdgcn_make_buffer_rsrc((void*)(p.x + (size_t)(d & 3) * XTILE), 0, (int)XTILE, 0x00020000);
      sB = (unsigned)(32 * pw) * 32768u + ((((unsigned)rank + ((d >> 2) & 7)) & 7) * 16 + ((d >> 5) & 15)) * 256u;
      const unsigned G = ((d >> 9) & 3) + 1, rot = (d >> 11) & 3, kpre = k0 + (G == 3 ? (unsigned)(b >> 3) / 3u % 3u : (unsigned)(b >> 5) & 3u);
      const unsigned q = (unsigned)lane - kpre, g = G == 3 ? (q & 63u) * 43u >> 7 : (q & 63u) >> 2, base = kpre + g * G, r = q - g * G + rot;
      tab = (unsigned)lane < kpre || base + G > k1 ? (unsigned)lane : base + (r >= G ? r - G : r);
      pos = k0;
    };
    unsigned soA, soB, ldsb = (unsigned)pw * 8192u;
    auto next = [&]() {
      const unsigned kt = (unsigned)__builtin_amdgcn_readlane((int)tab, (int)(pos & 63u));
      soA = sA + kt * stepK; soB = sB + kt * (64u * 32768u);
      ++pos;
    };
    auto issueA = [&]() {
#pragma unroll
      for (int i = 0; i < 8; ++i) __builtin_amdgcn_raw_ptr_buffer_load_lds(srdA, (LAS void*)(lds + ldsb + i * 1024), 16, voffA[i & 1], soA + i * stepI, 0, 17);
    };
    auto issueB = [&](unsigned so, unsigned lb) {
#pragma unroll
      for (int i = 0; i < 8; ++i) __builtin_amdgcn_raw_ptr_buffer_load_lds(srdB, (LAS void*)(lds + lb + NS * SLOT + i * 1024), 16, voffB[(i >> 1) & 1], so + i * (4u * 32768u), 0, 0);
    };
    static_assert(HK > NS, "the prologue stays inside the first piece");
    begin(helper ? aux : desc, NKT - n1, helper ? HK : NKT);
    next(); issueA(); issueB(soB, ldsb); ldsb += SLOT; next(); issueA(); issueB(soB, ldsb); ldsb += SLOT;
    asm volatile("s_waitcnt vmcnt(16)" : "+v"(done0) :: "memory");
    BAR();
#pragma unroll
    for (int i = 2; i < NS; ++i) { next(); issueA(); issueB(soB, ldsb); ldsb += SLOT; }
    ldsb -= NS * SLOT;
    next();
    const unsigned pace1 = owner ? PACE1 : helper ? PACE1H : PACE1N;
    unsigned tpace = now32() - PHEAD + pace1;
    int jsw = helper ? n0 - NS - 1 : J0 - 1;
    for (int j = 0; j < NJ - NS; ++j) {
      wait_vm<48>(); BAR();
      unsigned long long t64;
      asm volatile("s_memrealtime %0" : "=s"(t64));
      issueA();
      SCHED();
      const unsigned curB = soB, curL = ldsb;
      next();
      ldsb = ldsb + SLOT >= NS * SLOT ? ldsb + SLOT - NS * SLOT : ldsb + SLOT;
      tpace += j + 1 < PSW ? pace1 : PACE2;
      asm volatile("s_waitcnt lgkmcnt(0)" : "+s"(t64) :: "memory");
      int rare = j == jsw ? -1 : (int)((unsigned)t64 + PEST - tpace);
      asm volatile("" : "+s"(rare), "+s"(soA), "+s"(soB), "+s"(ldsb), "+s"(tpace));
      SCHED();
      issueB(curB, curL);
      SCHED();
      if (rare < 0) {
        if (j == jsw) {
          if (j != J0 - 1) begin(desc, 0, NKT);
          else {
            begin(desc >> 16, 0, NKT);
            if (T1.src != rank) {
              srdA = __builtin_amdgcn_make_buffer_rsrc((void*)(myws + WS_XBUF + ((rfl(done0) / NWG + 1) & 1) * XPAR + (size_t)T1.src * XTILE), 0, (int)XTILE, 0x00020000);
              sA = (unsigned)pw * 8192u; stepK = 16384u; stepI = 1024u; voffA[0] = voffA[1] = (unsigned)lane * 16u;
            }
          }
          jsw = J0 - 1;
          next();
        }
        while ((int)(now32() - tpace) < 0) __builtin_amdgcn_s_sleep(1);
      }
    }
    wait_vm<48>(); BAR();
    wait_vm<32>(); BAR();
    wait_vm<16>(); BAR();
    wait_vm<0>(); BAR();
    BAR();
  } else {
    const int wr = wid >> 1, wc = wid & 1, fr = lane & 15, fq = lane >> 4;
    const unsigned baseA0 = (unsigned)(wr * 64 + fr) * 128u + (unsigned)((fq ^ ((fr >> 1) & 7)) * 16);
    unsigned baseB[4];
    { const int q = (lane & 15) >> 2, pp = lane & 3; const int s = (q << 2) | ((fq & 1) << 1);
#pragma unroll
      for (int n = 0; n < 4; ++n) baseB[n] = (unsigned)(NS * SLOT) + (unsigned)(8 * fq + q) * 256u + (unsigned)(((8 * wc + 2 * n) ^ s) * 16 + 8 * pp); }
    f32x4 acc[4][4];
    bf16x8 FA[2][4], FB[2][4];
    u32x4 O[8];
    typedef LAS char* lptr;
    auto rd = [&](bf16x8 (&A)[4], bf16x8 (&B)[4], lptr pa, lptr (&pb)[4], int ks) {
      auto rb = [&](int n) {
        bf16x4 lo = __builtin_amdgcn_ds_read_tr16_b64_v4i16((LAS bf16x4*)(pb[n] + ks * 8192));
        bf16x4 hi = __builtin_amdgcn_ds_read_tr16_b64_v4i16((LAS bf16x4*)(pb[n] + ks * 8192 + 1024));
        B[n] = __builtin_shufflevector(lo, hi, 0, 1, 2, 3, 4, 5, 6, 7);
      };
      rb(0);
#pragma unroll
      for (int m = 0; m < 4; ++m) A[m] = *(const LAS bf16x8*)(pa + m * 2048);
      rb(1); rb(2); rb(3);
    };
    auto mm = [&](bf16x8 (&A)[4], bf16x8 (&B)[4], int lo, int hi) {
#pragma unroll
      for (int i = lo; i < hi; ++i) {
        const int n = i >> 2, m = (n & 1) ? 3 - (i & 3) : (i & 3);
        acc[m][n] = __builtin_amdgcn_mfma_f32_16x16x32_bf16(B[n], A[m], acc[m][n], 0, 0, 0);
      }
    };
    auto pairs = [&](int n) {
#pragma unroll
      for (int i = 0; i < n; ++i) { __builtin_amdgcn_sched_group_barrier(0x8, 1, 0); __builtin_amdgcn_sched_group_barrier(0x100, 1, 0); }
    };
    auto out_ptr = [&](const Tile& T) {
      char* o = p.out[T.dest] + (size_t)(T.src * MLOC + T.rt * 128 + wr * 64) * (NSH * 2) + (T.ct * 128 + wc * 64) * 2;
      asm volatile("" : "+s"(o));
      return o;
    };
    char* const o0 = out_ptr(T0); char* const o1 = out_ptr(T1);
    unsigned* cnt1 = (unsigned*)(p.ws[T1.dest] + WS_OUTCNT + rank * 128);
    asm volatile("" : "+s"(cnt1));
    const unsigned vo = (unsigned)(fr & 7) * (NSH * 2) + 64u * (fr >> 3) + 16u * ((fq >> 1) + 2 * (fq & 1));
    auto pack1 = [&](int m, int q) {
      const unsigned x0 = gelu2(acc[m][2 * q][0], acc[m][2 * q][1]), x1 = gelu2(acc[m][2 * q][2], acc[m][2 * q][3]);
      const unsigned y0 = gelu2(acc[m][2 * q + 1][0], acc[m][2 * q + 1][1]), y1 = gelu2(acc[m][2 * q + 1][2], acc[m][2 * q + 1][3]);
      const auto r0 = __builtin_amdgcn_permlane16_swap(x0, y0, false, false);
      const auto r1 = __builtin_amdgcn_permlane16_swap(x1, y1, false, false);
      return (u32x4){r0[0], r1[0], r0[1], r1[1]};
    };
    auto pack2 = [&](int m, u32x4& lo, u32x4& hi) {
      const u32x4 a = pack1(m, 0), c = pack1(m, 1);
#pragma unroll
      for (int e = 0; e < 4; ++e) {
        lo[e] = (unsigned)__builtin_amdgcn_update_dpp((int)a[e], (int)c[e], 0x128, 0xf, 0xc, false);
        hi[e] = (unsigned)__builtin_amdgcn_update_dpp((int)c[e], (int)a[e], 0x128, 0xf, 0x3, false);
      }
    };
    auto signal = [&](unsigned* cnt) {
      wait_vm<0>();
      if (lane == 0) __hip_atomic_fetch_add(cnt, 1u, __ATOMIC_RELAXED, __HIP_MEMORY_SCOPE_SYSTEM);
    };
    unsigned so = 0;
    lptr pB[4], pA0 = lds + baseA0, pA1 = lds + (baseA0 ^ 64u);
#pragma unroll
    for (int n = 0; n < 4; ++n) pB[n] = lds + baseB[n];
#define PART1() do { SCHED(); rd(FA[1], FB[1], pA1, pB, 1); mm(FA[0], FB[0], 0, 16); pairs(12); __builtin_amdgcn_sched_group_barrier(0x8, 4, 0); SCHED(); } while (0)
#define PART2() do { \
      so = so + SLOT == NS * SLOT ? 0 : so + SLOT; \
      pA0 = lds + baseA0 + so; pA1 = lds + (baseA0 ^ 64u) + so; \
      _Pragma("unroll") for (int n = 0; n < 4; ++n) pB[n] = lds + baseB[n] + so; \
      mm(FA[1], FB[1], 0, 4); \
      asm volatile("" : "+v"(pA0), "+v"(pA1), "+v"(pB[0]), "+v"(pB[1]), "+v"(pB[2]), "+v"(pB[3])); \
      _Pragma("unroll") for (int i = 0; i < 3; ++i) { __builtin_amdgcn_sched_group_barrier(0x8, 1, 0); __builtin_amdgcn_sched_group_barrier(0x2, 2, 0); } \
      __builtin_amdgcn_sched_group_barrier(0x8, 1, 0); \
      SCHED(); __builtin_amdgcn_s_waitcnt(0xC07F); __builtin_amdgcn_s_barrier(); SCHED(); \
      rd(FA[0], FB[0], pA0, pB, 0); mm(FA[1], FB[1], 4, 16); pairs(12); SCHED(); } while (0)
    char* part = myws + WS_PART + (size_t)pidx * PART + wid * 16384;
    if (wid == 0) TS(0);
    BAR();
    if (wid == 0) TS(1);
    rd(FA[0], FB[0], pA0, pB, 0);
    f32x4 H[16];
    int hk = 99;
#pragma unroll 1
    for (int seg = helper ? 0 : 1; seg < 3; ++seg) {
      if (seg < 2) {
#pragma unroll
        for (int m = 0; m < 4; ++m)
#pragma unroll
          for (int n = 0; n < 4; ++n) acc[m][n] = (f32x4){0.f, 0.f, 0.f, 0.f};
      }
      const int nk = seg == 0 ? HK : seg == 1 ? n1 - PRELD : PRELD;
#pragma unroll 1
      for (int kt = 0; kt < nk; ++kt) {
        PART1();
        if (hk <= HDRAIN) {
#define HST(s) case s: st16_sys<1024>(part + (14 - 2 * s) * 1024, (unsigned)lane * 16u, __builtin_bit_cast(u32x4, H[15 - 2 * s])); \
                       st16_sys<0>(part + (14 - 2 * s) * 1024, (unsigned)lane * 16u, __builtin_bit_cast(u32x4, H[14 - 2 * s])); break;
          switch (hk) { HST(0) HST(1) HST(2) HST(3) HST(4) HST(5) HST(6) HST(7) case HDRAIN: wait_vm<0>(); break; default: break; }
          ++hk;
        }
        PART2();
      }
      if (seg == 0) {
#pragma unroll
        for (int i = 0; i < 16; ++i) H[i] = acc[i >> 2][i & 3];
        hk = 0;
      } else if (seg == 1) {
        const auto srdP = __builtin_amdgcn_make_buffer_rsrc((void*)part, 0, owner ? 16384 : 0, 0x00020000);
#pragma unroll
        for (int i = 0; i < 16; ++i) H[i] = __builtin_bit_cast(f32x4, __builtin_amdgcn_raw_buffer_load_b128(srdP, lane * 16 + (i & 3) * 1024, (i >> 2) * 4096, 16));
      }
    }
    if (wid == 0) TS(2);
#pragma unroll
    for (int i = 0; i < 16; ++i) acc[i >> 2][i & 3] += H[i];
#pragma unroll
    for (int m = 0; m < 4; ++m) pack2(m, O[2 * m], O[2 * m + 1]);
#pragma unroll
    for (int m = 0; m < 4; ++m)
#pragma unroll
      for (int n = 0; n < 4; ++n) acc[m][n] = (f32x4){0.f, 0.f, 0.f, 0.f};
    auto store0 = [&](int m) {
      switch (m) {
        case 0: st16_l2(o0, vo, O[0]); st16_l2(o0 + 32768, vo, O[1]); break;
        case 1: st16_l2(o0 + 65536, vo, O[2]); st16_l2(o0 + 65536 + 32768, vo, O[3]); break;
        case 2: st16_l2(o0 + 2 * 65536, vo, O[4]); st16_l2(o0 + 2 * 65536 + 32768, vo, O[5]); break;
        default: st16_l2(o0 + 3 * 65536, vo, O[6]); st16_l2(o0 + 3 * 65536 + 32768, vo, O[7]); break;
      }
    };
    __builtin_amdgcn_s_waitcnt(0x0F70);
#pragma unroll 1
    for (int kt = -kb0; kt < n2 - kb0; ++kt) {
      PART1();
      if ((unsigned)kt <= (unsigned)ODRAIN) {
        if (kt < 4) store0(kt);
        else if (kt == ODRAIN) wait_vm<0>();
      }
      PART2();
    }
    if (wid == 0) TS(3);
#pragma unroll
    for (int m = 0; m < 4; ++m) {
      u32x4 lo, hi; pack2(m, lo, hi);
      if (T1.dest != rank) wait_vm<1>();
      st16_sys<0>(o1 + m * 65536, vo, lo);
      if (T1.dest != rank) wait_vm<1>();
      st16_sys<0>(o1 + m * 65536 + 32768, vo, hi); SCHED();
    }
    if (T1.dest != rank) signal(cnt1);
  }
}

extern "C" void kernel_launch(int rank, int nranks, ncclComm_t comm, hipStream_t stream,
                              void* const* d_in, const int* in_sizes, int n_in,
                              void* d_out, int out_size, const int* in_dtypes, int out_dtype,
                              void* const* d_out_all, void* const* d_ws_all, size_t ws_bytes) {
  if (nranks != NR || ws_bytes < WS_END || (unsigned)rank >= (unsigned)NR) { fprintf(stderr, "unsupported configuration\n"); return; }
  HIP_CHECK(hipFuncSetAttribute((const void*)gemm_a2a_kernel, hipFuncAttributeMaxDynamicSharedMemorySize, LDS_BYTES));
  Params p;
  p.x = (const char*)d_in[0]; p.w = (const char*)d_in[1]; p.rank = rank; p.pad = 0;
  for (int i = 0; i < NR; ++i) { p.out[i] = (char*)d_out_all[i]; p.ws[i] = (char*)d_ws_all[i]; }
  build_desc(p.desc);
  gemm_a2a_kernel<<<NWG, 512, LDS_BYTES, stream>>>(p);
  HIP_CHECK(hipGetLastError());
}
```

```cpp
#include <hip/hip_runtime.h>
#include <rccl/rccl.h>
#include <cstdio>

#define HIP_CHECK(expr) do { hipError_t _e = (expr); if (_e != hipSuccess) { \
    fprintf(stderr, "HIP error %s at %s:%d\n", hipGetErrorString(_e), __FILE__, __LINE__); \
    return; } } while (0)

#define LAS __attribute__((address_space(3)))
typedef short bf16x8 __attribute__((ext_vector_type(8)));
typedef short bf16x4 __attribute__((ext_vector_type(4)));
typedef float f32x4 __attribute__((ext_vector_type(4)));
typedef float f32x2 __attribute__((ext_vector_type(2)));
typedef unsigned u32x4 __attribute__((ext_vector_type(4)));
typedef __bf16 bfx2 __attribute__((ext_vector_type(2)));

#define TAU_O 66
#define TAU_X 8
#define XDELAY 100
#define HK 11
#define PACE1 60
#define PACE1H 58
#define PACE1N 60
#define PEST 30
#define PSW 64
#define PHEAD 500
#define PACE2 42
constexpr int NR = 8;
constexpr int KDIM = 4096, NDIM = 16384, MLOC = 512, NSH = 2048;
constexpr int NKT = 64, SLOT = 16384, NS = 5, NWG = 256;
constexpr int LDS_BYTES = 2 * NS * SLOT;
constexpr int HDRAIN = 12, ODRAIN = 5, PRELD = 3;
constexpr int Q4 = 80;
constexpr size_t WS_DONE = 0, WS_TMO = 128, WS_OUTCNT = 4096, WS_XCNT = 8192, WS_O0CNT = 12288, WS_XLOC = 14336, WS_CEN = 16384, WS_ARR = 20480, WS_ACK = 24576, WS_PFLAG = 32768, WS_STAMP = 65536;
constexpr size_t XTILE = 128 * KDIM * 2;
constexpr size_t WS_XBUF = 1u << 20, XPAR = NR * XTILE;
constexpr size_t WS_PART = WS_XBUF + 2 * XPAR, PART = 128 * 128 * 4, WS_END = WS_PART + 96 * PART;
constexpr unsigned long long DEADLINE = 40000000ull;

enum { ROLE_OWNER = 1, ROLE_HELPER = 2 };
struct Params { const char* x; const char* w; char* out[NR]; char* ws[NR]; int rank; int pad; unsigned desc[NWG][2]; };
struct Tile { int src; int rt; int dest; int ct; };
__device__ __forceinline__ Tile tile_of(unsigned d, int rank) {
  Tile r; r.src = (rank + (int)((d >> 13) & 7)) & 7; r.rt = d & 3; r.dest = (rank + (int)((d >> 2) & 7)) & 7; r.ct = (d >> 5) & 15; return r;
}
static inline unsigned tile_code(int so, int rt, int dof, int ct, int G, int rot) { return (unsigned)(rt | dof << 2 | ct << 5 | (G - 1) << 9 | (rot % G) << 11 | so << 13); }
static void build_desc(unsigned (*out)[2]) {
  unsigned desc[NWG][2];
  unsigned T[2][8][32]; int n[2][8] = {};
  auto put = [&](int w, int x, unsigned c) { T[w][x][n[w][x]++] = c; };
  auto code = [&](int q, int rt) { return tile_code(0, rt, 1 + q % 7, q / 7, q == Q4 ? 4 : 3, rt + q); };
  for (int k = 0; k < 80; ++k) for (int rt = 0; rt < 3; ++rt) put(0, k / 10, code(k / 10 + 8 * (k % 10), rt));
  for (int i = 0; i < 16; ++i) put(0, i / 2, code(i < 12 ? 81 + i / 3 : Q4, i < 12 ? i % 3 : i - 12));
  for (int k = 0; k < 24; ++k) for (int rt = 0; rt < 3; ++rt) put(1, k / 6, code(85 + k / 6 + 4 * (k % 6), rt));
  for (int x = 0; x < 4; ++x) for (int c = 0; c < 4; ++c) for (int r = 0; r < 3; ++r) put(1, x, tile_code(0, r, 0, x + 4 * c, 3, r + c));
  for (int k = 0; k < 3; ++k) { put(1, k, code(109 + k, 0)); put(1, k, code(109 + k, 1)); }
  put(1, 3, code(109, 2)); put(1, 3, code(110, 2));
  for (int g = 0; g < 2; ++g) for (int h = 0; h < 2; ++h) {
    for (int c = 0; c < 8; ++c) for (int r = 0; r < 4; ++r)
      if (!(4 * g + r == 8 - (1 + Q4 % 7) && 8 * h + c == Q4 / 7)) put(1, 4 + 2 * g + h, tile_code(4 * g + r, 3, 0, 8 * h + c, 4, r + c + 2 * h + g));
    if (n[1][4 + 2 * g + h] < 32) put(1, 4 + 2 * g + h, code(111, 2));
  }
  int role[NWG], own[96], nown = 0;
  for (int b = 0; b < NWG; ++b) { desc[b][0] = T[0][b & 7][b >> 3] | T[1][b & 7][b >> 3] << 16; desc[b][1] = 0; role[b] = 0; }
  for (int pass = 0; pass < 2; ++pass) for (int x = 0; x < 4; ++x) for (int i = pass ? 30 : 0; i < (pass ? 32 : 18); ++i) own[nown++] = i * 8 + x;
  for (int k = 0; HK && k < nown; ++k) {
    const int o = own[k], h = (k % 30) * 8 + 4 + k / 30;
    desc[o][1] = (unsigned)k << 16 | 1u << 14; role[o] = ROLE_OWNER;
    desc[h][1] = (desc[o][0] & 0x1fff) | (unsigned)k << 16; role[h] = ROLE_HELPER;
  }
  auto sched = [](int r, int j) { const int p1 = r == ROLE_OWNER ? PACE1 : r == ROLE_HELPER ? PACE1H : PACE1N; return j < PSW ? j * p1 : PSW * p1 + (j - PSW) * PACE2; };
  auto turn_kt = [&](int k, int d, int r) {
    const int t = sched(ROLE_OWNER, NKT - HK) + k * TAU_O + d * (TAU_O / 8), j1 = r == ROLE_OWNER ? NKT - HK : r == ROLE_HELPER ? NKT + HK : NKT;
    if (t < sched(r, j1)) return -1;
    int j = j1 + 4; while (sched(r, j) < t) ++j;
    return j - j1 < NKT - 12 ? j - j1 : NKT - 12;
  };
  bool sent[NWG] = {};
  int load[64][8] = {};
  for (int k = 0, left = NWG; left; ++k)
    for (int dd = 0; dd < 7; ++dd) {
      const int d = 1 + (dd + k) % 7;
      int best = -1, bc = 1 << 30;
      for (int b = 0; b < NWG; ++b) {
        if (sent[b] || (int)((desc[b][0] >> 2) & 7) != d) continue;
        if (turn_kt(k, d, role[b]) < 0) continue;
        const int x = b & 7, c = 8 * (2 * load[k][x] + (k > 0 ? load[k - 1][x] : 0) + (k > 1 ? load[k - 2][x] : 0)) + 12 * (role[b] == ROLE_OWNER ? 0 : role[b] == ROLE_HELPER ? 1 : 2);
        if (c < bc) { bc = c; best = b; }
      }
      if (best >= 0) { sent[best] = true; --left; load[k][best & 7]++; desc[best][1] |= (unsigned)turn_kt(k, d, role[best]) << 24; }
    }
  for (int b = 0; b < NWG; ++b) { const int l = (b & ~7) | (b & 1) << 2 | (b & 7) >> 1; out[b][0] = desc[l][0]; out[b][1] = desc[l][1]; }
}

__device__ __forceinline__ unsigned rfl(unsigned v) { return (unsigned)__builtin_amdgcn_readfirstlane((int)v); }
__device__ __forceinline__ unsigned now32() { return (unsigned)__builtin_amdgcn_s_memrealtime(); }
template <int N> __device__ __forceinline__ void wait_vm() { asm volatile("s_waitcnt vmcnt(%0)" :: "i"(N) : "memory"); }
template <int OFF> __device__ __forceinline__ void st16_sys(char* sbase, unsigned voff, u32x4 v) {
  asm volatile("global_store_dwordx4 %0, %1, %2 offset:%3 sc0 sc1\n\ts_nop 1" :: "v"(voff), "v"(v), "s"(sbase), "i"(OFF) : "memory");
}
__device__ __forceinline__ void st16_l2(char* sbase, unsigned voff, u32x4 v) {
  asm volatile("global_store_dwordx4 %0, %1, %2\n\ts_nop 1" :: "v"(voff), "v"(v), "s"(sbase) : "memory");
}
#define BAR() do { asm volatile("" ::: "memory"); __builtin_amdgcn_s_barrier(); asm volatile("" ::: "memory"); } while (0)
#define SCHED() __builtin_amdgcn_sched_barrier(0)
#define TS(k) do {} while (0)

__device__ __forceinline__ void wait_ge(const unsigned* flag, unsigned want, unsigned* tmo, unsigned call, bool active, char* mark, int lane) {
  const unsigned long long t0 = __builtin_amdgcn_s_memrealtime();
  for (unsigned n = 1;; ++n) {
    const bool ok = !active || __hip_atomic_load(flag, __ATOMIC_RELAXED, __HIP_MEMORY_SCOPE_SYSTEM) >= want;
    if (__all(ok)) return;
    if ((n & 15) == 0 &&
        (__hip_atomic_load(tmo, __ATOMIC_RELAXED, __HIP_MEMORY_SCOPE_AGENT) == call || __builtin_amdgcn_s_memrealtime() - t0 > DEADLINE)) {
      __hip_atomic_store(tmo, call, __ATOMIC_RELAXED, __HIP_MEMORY_SCOPE_AGENT);
      if (lane == 0) __hip_atomic_store((unsigned*)mark, 0x7fc07fc0u, __ATOMIC_RELAXED, __HIP_MEMORY_SCOPE_SYSTEM);
      return;
    }
    __builtin_amdgcn_s_sleep(1);
  }
}

__global__ void __launch_bounds__(512, 2) gemm_a2a_kernel(Params p) {
  extern __shared__ __attribute__((aligned(16))) char smem[];
  LAS char* lds = (LAS char*)smem;
  const int tid = threadIdx.x, wid = (int)rfl(tid >> 6), lane = tid & 63;
  const int b = blockIdx.x, rank = p.rank;
  const unsigned desc = p.desc[b][0], aux = p.desc[b][1];
  const Tile T0 = tile_of(desc, rank), T1 = tile_of(desc >> 16, rank);
  const bool owner = (aux >> 14) & 1, helper = (aux & 0x1fffu) != 0;
  const int n0 = helper ? HK : 0, n1 = owner ? NKT - HK : NKT, n2 = NKT;
  const int NJ = n0 + n1 + n2;
  const int J0 = n0 + n1 - NS;
  char* myws = p.ws[rank];
  unsigned* tmo = (unsigned*)(myws + WS_TMO);
  char* mark = p.out[rank] + (size_t)rank * MLOC * NSH * 2;
  const unsigned pidx = (aux >> 16) & 127;
  const int kb0 = (int)(aux >> 24);

  if (wid == 7) {
    const bool staged = T1.src != rank;
    if (!staged && !owner && !helper) return;
    const unsigned call = rfl(__hip_atomic_load((unsigned*)(myws + WS_DONE), __ATOMIC_RELAXED, __HIP_MEMORY_SCOPE_AGENT)) / NWG + 1;
    unsigned* pf = (unsigned*)(myws + WS_PFLAG + pidx * 128);
    BAR();
    int it = 0;
    if (helper) {
      for (; it <= HK + HDRAIN; ++it) BAR();
      if (lane == 0) __hip_atomic_fetch_add(pf, 1u, __ATOMIC_RELAXED, __HIP_MEMORY_SCOPE_AGENT);
    }
    if (owner) {
      for (; it < n1 - PRELD - 6; ++it) BAR();
      const unsigned seen = __hip_atomic_load(pf, __ATOMIC_RELAXED, __HIP_MEMORY_SCOPE_AGENT);
      for (; it < n1 - PRELD - 1; ++it) BAR();
      if (rfl(seen) < call) wait_ge(pf, call, tmo, call, true, mark, lane);
    }
    if (staged) {
      const unsigned* fl = (const unsigned*)(myws + WS_XCNT + T1.src * 512);
#pragma unroll 1
      for (int c = 0; c < 4; ++c) {
        const int jc = J0 + (c ? 16 * c - 3 : 0);
        for (; it < jc - 5; ++it) BAR();
        const unsigned seen = __hip_atomic_load(fl + c * 32, __ATOMIC_RELAXED, __HIP_MEMORY_SCOPE_SYSTEM);
        for (; it < jc; ++it) BAR();
        if (rfl(seen) < call) wait_ge(fl + c * 32, call, tmo, call, true, mark, lane);
        if (c == 0) TS(6);
      }
    }
  } else if (wid == 6) {
    u32x4 xv[4];
#pragma unroll
    for (int i = 0; i < 4; ++i) {
      const int P = b * 4 + i;
      xv[i] = *(const u32x4*)(p.x + 3 * XTILE + (size_t)((P & 15) * 8 + (lane >> 3)) * 8192 + (P >> 4) * 128 + ((lane & 7) ^ ((4 * (P & 1) + (lane >> 4)) & 7)) * 16);
    }
    const unsigned done = __hip_atomic_load((unsigned*)(myws + WS_DONE), __ATOMIC_RELAXED, __HIP_MEMORY_SCOPE_AGENT);
    const unsigned xcc = (unsigned)__builtin_amdgcn_s_getreg((3 << 11) | 20) & 15u;
    if (lane == 0) __hip_atomic_fetch_add((unsigned*)(myws + WS_CEN + xcc * 128), 1u, __ATOMIC_RELAXED, __HIP_MEMORY_SCOPE_AGENT);
    const unsigned peer = (unsigned)(rank + 1 + lane) & 7;
    BAR();
    int it = 0;
    auto bars = [&](int n) { for (int e = min(it + n, NJ); it < e; ++it) BAR(); };
    const int ch = b >> 6;
    const unsigned tn = now32() + XDELAY + (unsigned)b * TAU_X;
    while ((int)(now32() - tn) < 0 && it < 36 + 16 * ch) { BAR(); ++it; }
    __builtin_amdgcn_s_waitcnt(0x0F70);
    const unsigned call = rfl(done) / NWG + 1;
    char* pws = p.ws[0];
#pragma unroll
    for (int q = 1; q < NR; ++q) pws = peer == (unsigned)q ? p.ws[q] : pws;
    const size_t so = WS_XBUF + (call & 1) * XPAR + (size_t)rank * XTILE + (size_t)b * 4096;
#pragma unroll
    for (int i = 0; i < 4; ++i)
#pragma unroll
      for (int q = 0; q < 7; ++q) st16_sys<0>(p.ws[(rank + 1 + q) & 7] + so + i * 1024, (unsigned)lane * 16u, xv[i]);
    TS(9);
    bars(5);
    wait_vm<0>();
    unsigned tick; unsigned long long sv;
#define ATOM0(OP, ADDR, VAL) asm volatile("s_mov_b64 %1, exec\n\ts_mov_b64 exec, 1\n\t" OP " %0, %2, %3, off sc0 sc1\n\ts_mov_b64 exec, %1" \
                                          : "=&v"(tick), "=&s"(sv) : "v"(ADDR), "v"(VAL) : "memory")
    ATOM0("global_atomic_add", myws + WS_XLOC + ch * 128, 1u);
    bars(3);
    asm volatile("s_waitcnt vmcnt(0)" : "+v"(tick) :: "memory");
    if (rfl(tick) == call * 64 - 1) {
      if (lane < 7) __hip_atomic_store((unsigned*)(pws + WS_XCNT + rank * 512 + ch * 128), call, __ATOMIC_RELAXED, __HIP_MEMORY_SCOPE_SYSTEM);
    }
    if (lane == 0) __hip_atomic_fetch_add((unsigned*)(myws + WS_DONE), 1u, __ATOMIC_RELAXED, __HIP_MEMORY_SCOPE_AGENT);
    TS(5);
    bars(n0 + n1 + kb0 + ODRAIN + 1 - it);
    ATOM0("global_atomic_add", myws + WS_ARR + xcc * 128, 1u);
    const unsigned cen = __hip_atomic_load((unsigned*)(myws + WS_CEN + (lane & 15) * 128), __ATOMIC_RELAXED, __HIP_MEMORY_SCOPE_AGENT);
    bars(2);
    asm volatile("s_waitcnt vmcnt(0)" : "+v"(tick) :: "memory");
    unsigned total = 0;
#pragma unroll
    for (int k = 0; k < 16; ++k) total += (unsigned)__builtin_amdgcn_readlane((int)cen, k);
    const unsigned t = rfl(tick) + 1;
    if (total != call * NWG || t == (unsigned)__builtin_amdgcn_readlane((int)cen, (int)xcc)) {
      asm volatile("buffer_wbl2 sc0 sc1" ::: "memory");
      bars(4);
      wait_vm<0>();
      ATOM0("global_atomic_umax", myws + WS_ACK + xcc * 128, t);
      bars(2);
      asm volatile("s_waitcnt vmcnt(0)" : "+v"(tick) :: "memory");
      const unsigned old = rfl(tick);
      if (t > old && lane < 7) __hip_atomic_fetch_add((unsigned*)(pws + WS_O0CNT + rank * 128), t - old, __ATOMIC_RELAXED, __HIP_MEMORY_SCOPE_SYSTEM);
    }
    TS(7);
    if (b == NWG - 1) {
      bars(NJ);
      const unsigned d = 7 - (lane & 7), n2nd = 3 * ((111 - (d - 1)) / 7 - (84 - (d - 1)) / 7);
      wait_ge((const unsigned*)(myws + (lane < 8 ? WS_OUTCNT : WS_O0CNT) + peer * 128), lane < 8 ? call * 4 * n2nd : call * NWG, tmo, call, lane < 15 && lane != 7, mark, lane);
      TS(8);
    }
  } else if (wid >= 4) {
    const int pw = wid - 4;
    unsigned done0;
    asm volatile("global_load_dword %0, %1, off sc1" : "=v"(done0) : "v"(myws + WS_DONE) : "memory");
    const auto srdB = __builtin_amdgcn_make_buffer_rsrc((void*)p.w, 0, KDIM * NDIM * 2, 0x00020000);
    unsigned voffA[2], voffB[2];
#pragma unroll
    for (int i = 0; i < 2; ++i) {
      voffA[i] = (unsigned)(lane >> 3) * 8192u + (unsigned)(((lane & 7) ^ ((4 * i + (lane >> 4)) & 7)) * 16);
      voffB[i] = (unsigned)(lane >> 4) * 32768u + (unsigned)(((lane & 15) ^ (((lane >> 4) << 2) | (i << 1))) * 16);
    }
    unsigned sA = (unsigned)(64 * pw) * 8192u, stepK = 128u, stepI = 65536u;
    unsigned sB, tab, pos;
    auto srdA = __builtin_amdgcn_make_buffer_rsrc((void*)p.x, 0, (int)XTILE, 0x00020000);
    auto begin = [&](unsigned d, unsigned k0, unsigned k1) {
      srdA = __builtin_amdgcn_make_buffer_rsrc((void*)(p.x + (size_t)(d & 3) * XTILE), 0, (int)XTILE, 0x00020000);
      sB = (unsigned)(32 * pw) * 32768u + ((((unsigned)rank + ((d >> 2) & 7)) & 7) * 16 + ((d >> 5) & 15)) * 256u;
      const unsigned G = ((d >> 9) & 3) + 1, rot = (d >> 11) & 3, kpre = k0 + (G == 3 ? (unsigned)(b >> 3) / 3u % 3u : (unsigned)(b >> 5) & 3u);
      const unsigned q = (unsigned)lane - kpre, g = G == 3 ? (q & 63u) * 43u >> 7 : (q & 63u) >> 2, base = kpre + g * G, r = q - g * G + rot;
      tab = (unsigned)lane < kpre || base + G > k1 ? (unsigned)lane : base + (r >= G ? r - G : r);
      pos = k0;
    };
    unsigned soA, soB, ldsb = (unsigned)pw * 8192u;
    auto next = [&]() {
      const unsigned kt = (unsigned)__builtin_amdgcn_readlane((int)tab, (int)(pos & 63u));
      soA = sA + kt * stepK; soB = sB + kt * (64u * 32768u);
      ++pos;
    };
    auto issueA = [&]() {
#pragma unroll
      for (int i = 0; i < 8; ++i) __builtin_amdgcn_raw_ptr_buffer_load_lds(srdA, (LAS void*)(lds + ldsb + i * 1024), 16, voffA[i & 1], soA + i * stepI, 0, 17);
    };
    auto issueB = [&](unsigned so, unsigned lb) {
#pragma unroll
      for (int i = 0; i < 8; ++i) __builtin_amdgcn_raw_ptr_buffer_load_lds(srdB, (LAS void*)(lds + lb + NS * SLOT + i * 1024), 16, voffB[(i >> 1) & 1], so + i * (4u * 32768u), 0, 0);
    };
    static_assert(HK > NS, "the prologue stays inside the first piece");
    begin(helper ? aux : desc, NKT - n1, helper ? HK : NKT);
    next(); issueA(); issueB(soB, ldsb); ldsb += SLOT; next(); issueA(); issueB(soB, ldsb); ldsb += SLOT;
    asm volatile("s_waitcnt vmcnt(16)" : "+v"(done0) :: "memory");
    BAR();
#pragma unroll
    for (int i = 2; i < NS; ++i) { next(); issueA(); issueB(soB, ldsb); ldsb += SLOT; }
    ldsb -= NS * SLOT;
    next();
    const unsigned pace1 = owner ? PACE1 : helper ? PACE1H : PACE1N;
    unsigned tpace = now32() - PHEAD + pace1;
    int jsw = helper ? n0 - NS - 1 : J0 - 1;
    for (int j = 0; j < NJ - NS; ++j) {
      wait_vm<48>(); BAR();
      unsigned long long t64;
      asm volatile("s_memrealtime %0" : "=s"(t64));
      issueA();
      SCHED();
      const unsigned curB = soB, curL = ldsb;
      next();
      ldsb = ldsb + SLOT >= NS * SLOT ? ldsb + SLOT - NS * SLOT : ldsb + SLOT;
      tpace += j + 1 < PSW ? pace1 : PACE2;
      asm volatile("s_waitcnt lgkmcnt(0)" : "+s"(t64) :: "memory");
      int rare = j == jsw ? -1 : (int)((unsigned)t64 + PEST - tpace);
      asm volatile("" : "+s"(rare), "+s"(soA), "+s"(soB), "+s"(ldsb), "+s"(tpace));
      SCHED();
      issueB(curB, curL);
      SCHED();
      if (rare < 0) {
        if (j == jsw) {
          if (j != J0 - 1) begin(desc, 0, NKT);
          else {
            begin(desc >> 16, 0, NKT);
            if (T1.src != rank) {
              srdA = __builtin_amdgcn_make_buffer_rsrc((void*)(myws + WS_XBUF + ((rfl(done0) / NWG + 1) & 1) * XPAR + (size_t)T1.src * XTILE), 0, (int)XTILE, 0x00020000);
              sA = (unsigned)pw * 8192u; stepK = 16384u; stepI = 1024u; voffA[0] = voffA[1] = (unsigned)lane * 16u;
            }
          }
          jsw = J0 - 1;
          next();
        }
        while ((int)(now32() - tpace) < 0) __builtin_amdgcn_s_sleep(1);
      }
    }
    wait_vm<48>(); BAR();
    wait_vm<32>(); BAR();
    wait_vm<16>(); BAR();
    wait_vm<0>(); BAR();
    BAR();
  } else {
    const int wr = wid >> 1, wc = wid & 1, fr = lane & 15, fq = lane >> 4;
    const unsigned baseA0 = (unsigned)(wr * 64 + fr) * 128u + (unsigned)((fq ^ ((fr >> 1) & 7)) * 16);
    unsigned baseB[4];
    { const int q = (lane & 15) >> 2, pp = lane & 3; const int s = (q << 2) | ((fq & 1) << 1);
#pragma unroll
      for (int n = 0; n < 4; ++n) baseB[n] = (unsigned)(NS * SLOT) + (unsigned)(8 * fq + q) * 256u + (unsigned)(((8 * wc + 2 * n) ^ s) * 16 + 8 * pp); }
    f32x4 acc[4][4];
    bf16x8 FA[2][4], FB[2][4];
    typedef LAS char* lptr;
    auto rd = [&](bf16x8 (&A)[4], bf16x8 (&B)[4], lptr pa, lptr (&pb)[4], int ks) {
      auto rb = [&](int n) {
        bf16x4 lo = __builtin_amdgcn_ds_read_tr16_b64_v4i16((LAS bf16x4*)(pb[n] + ks * 8192));
        bf16x4 hi = __builtin_amdgcn_ds_read_tr16_b64_v4i16((LAS bf16x4*)(pb[n] + ks * 8192 + 1024));
        B[n] = __builtin_shufflevector(lo, hi, 0, 1, 2, 3, 4, 5, 6, 7);
      };
      rb(0);
#pragma unroll
      for (int m = 0; m < 4; ++m) A[m] = *(const LAS bf16x8*)(pa + m * 2048);
      rb(1); rb(2); rb(3);
    };
    auto mm = [&](bf16x8 (&A)[4], bf16x8 (&B)[4], int lo, int hi) {
#pragma unroll
      for (int i = lo; i < hi; ++i) {
        const int n = i >> 2, m = (n & 1) ? 3 - (i & 3) : (i & 3);
        acc[m][n] = __builtin_amdgcn_mfma_f32_16x16x32_bf16(B[n], A[m], acc[m][n], 0, 0, 0);
      }
    };
    auto pairs = [&](int n) {
#pragma unroll
      for (int i = 0; i < n; ++i) { __builtin_amdgcn_sched_group_barrier(0x8, 1, 0); __builtin_amdgcn_sched_group_barrier(0x100, 1, 0); }
    };
    auto out_ptr = [&](const Tile& T) {
      char* o = p.out[T.dest] + (size_t)(T.src * MLOC + T.rt * 128 + wr * 64) * (NSH * 2) + (T.ct * 128 + wc * 64) * 2;
      asm volatile("" : "+s"(o));
      return o;
    };
    char* const o0 = out_ptr(T0); char* const o1 = out_ptr(T1);
    unsigned* cnt1 = (unsigned*)(p.ws[T1.dest] + WS_OUTCNT + rank * 128);
    asm volatile("" : "+s"(cnt1));
    const unsigned vo = (unsigned)(fr & 7) * (NSH * 2) + 64u * (fr >> 3) + 16u * ((fq >> 1) + 2 * (fq & 1));
    auto shuffle = [&](unsigned a0, unsigned a1, unsigned b0, unsigned b1, unsigned c0, unsigned c1, unsigned d0, unsigned d1, u32x4& lo, u32x4& hi) {
      const auto r0 = __builtin_amdgcn_permlane16_swap(a0, b0, false, false), r1 = __builtin_amdgcn_permlane16_swap(a1, b1, false, false);
      const auto r2 = __builtin_amdgcn_permlane16_swap(c0, d0, false, false), r3 = __builtin_amdgcn_permlane16_swap(c1, d1, false, false);
      const u32x4 a = {r0[0], r1[0], r0[1], r1[1]}, c = {r2[0], r3[0], r2[1], r3[1]};
#pragma unroll
      for (int e = 0; e < 4; ++e) {
        lo[e] = (unsigned)__builtin_amdgcn_update_dpp((int)a[e], (int)c[e], 0x128, 0xf, 0xc, false);
        hi[e] = (unsigned)__builtin_amdgcn_update_dpp((int)c[e], (int)a[e], 0x128, 0xf, 0x3, false);
      }
    };
    auto gelu_step = [&](int g, const f32x2 (&y)[8], f32x2 (&t)[8], unsigned (&u)[8]) {
      const float k0 = -2.0f * 0.7978845608028654f * 1.4426950408889634f, k1 = k0 * 0.044715f;
      const int q = 4 * (g & 1);
      if (g < 2) { _Pragma("unroll") for (int e = q; e < q + 4; ++e) t[e] = y[e] * y[e]; }
      else if (g < 4) { _Pragma("unroll") for (int e = q; e < q + 4; ++e) t[e] = __builtin_elementwise_fma(t[e], (f32x2){k1, k1}, (f32x2){k0, k0}); }
      else if (g < 6) { _Pragma("unroll") for (int e = q; e < q + 4; ++e) t[e] = y[e] * t[e]; }
      else if (g < 14) t[g - 6] = (f32x2){__builtin_amdgcn_exp2f(t[g - 6].x), __builtin_amdgcn_exp2f(t[g - 6].y)};
      else if (g < 16) { _Pragma("unroll") for (int e = q; e < q + 4; ++e) t[e] = t[e] + 1.0f; }
      else if (g < 24) t[g - 16] = (f32x2){__builtin_amdgcn_rcpf(t[g - 16].x), __builtin_amdgcn_rcpf(t[g - 16].y)};
      else if (g < 26) { _Pragma("unroll") for (int e = q; e < q + 4; ++e) t[e] = y[e] * t[e]; }
      else if (g < 28) { _Pragma("unroll") for (int e = q; e < q + 4; ++e) u[e] = __builtin_bit_cast(unsigned, __builtin_convertvector(t[e], bfx2)); }
    };
    auto pack2 = [&](int m, u32x4& lo, u32x4& hi) {
      f32x2 y[8], t[8]; unsigned u[8];
#pragma unroll
      for (int e = 0; e < 8; ++e) y[e] = (f32x2){acc[m][e >> 1][2 * (e & 1)], acc[m][e >> 1][2 * (e & 1) + 1]};
#pragma unroll
      for (int g = 0; g < 28; ++g) { gelu_step(g, y, t, u); SCHED(); }
      shuffle(u[0], u[1], u[2], u[3], u[4], u[5], u[6], u[7], lo, hi);
    };
    auto signal = [&](unsigned* cnt) {
      wait_vm<0>();
      if (lane == 0) __hip_atomic_fetch_add(cnt, 1u, __ATOMIC_RELAXED, __HIP_MEMORY_SCOPE_SYSTEM);
    };
    unsigned so = 0;
    lptr pB[4], pA0 = lds + baseA0, pA1 = lds + (baseA0 ^ 64u);
#pragma unroll
    for (int n = 0; n < 4; ++n) pB[n] = lds + baseB[n];
#define PART1() do { SCHED(); rd(FA[1], FB[1], pA1, pB, 1); mm(FA[0], FB[0], 0, 16); pairs(12); __builtin_amdgcn_sched_group_barrier(0x8, 4, 0); SCHED(); } while (0)
#define PART2A() do { \
      so = so + SLOT == NS * SLOT ? 0 : so + SLOT; \
      pA0 = lds + baseA0 + so; pA1 = lds + (baseA0 ^ 64u) + so; \
      _Pragma("unroll") for (int n = 0; n < 4; ++n) pB[n] = lds + baseB[n] + so; \
      mm(FA[1], FB[1], 0, 4); \
      asm volatile("" : "+v"(pA0), "+v"(pA1), "+v"(pB[0]), "+v"(pB[1]), "+v"(pB[2]), "+v"(pB[3])); \
      _Pragma("unroll") for (int i = 0; i < 3; ++i) { __builtin_amdgcn_sched_group_barrier(0x8, 1, 0); __builtin_amdgcn_sched_group_barrier(0x2, 2, 0); } \
      __builtin_amdgcn_sched_group_barrier(0x8, 1, 0); \
      SCHED(); __builtin_amdgcn_s_waitcnt(0xC07F); __builtin_amdgcn_s_barrier(); SCHED(); } while (0)
#define PART2() do { PART2A(); rd(FA[0], FB[0], pA0, pB, 0); mm(FA[1], FB[1], 4, 16); pairs(12); SCHED(); } while (0)
    char* part = myws + WS_PART + (size_t)pidx * PART + wid * 16384;
    if (wid == 0) TS(0);
    BAR();
    if (wid == 0) TS(1);
    rd(FA[0], FB[0], pA0, pB, 0);
    f32x4 H[16];
    int hk = 99;
#pragma unroll 1
    for (int seg = helper ? 0 : 1; seg < 3; ++seg) {
      if (seg < 2) {
#pragma unroll
        for (int m = 0; m < 4; ++m)
#pragma unroll
          for (int n = 0; n < 4; ++n) acc[m][n] = (f32x4){0.f, 0.f, 0.f, 0.f};
      }
      const int nk = seg == 0 ? HK : seg == 1 ? n1 - PRELD : PRELD;
#pragma unroll 1
      for (int kt = 0; kt < nk; ++kt) {
        PART1();
        if (hk <= HDRAIN) {
#define HST(s) case s: st16_sys<1024>(part + (14 - 2 * s) * 1024, (unsigned)lane * 16u, __builtin_bit_cast(u32x4, H[15 - 2 * s])); \
                       st16_sys<0>(part + (14 - 2 * s) * 1024, (unsigned)lane * 16u, __builtin_bit_cast(u32x4, H[14 - 2 * s])); break;
          switch (hk) { HST(0) HST(1) HST(2) HST(3) HST(4) HST(5) HST(6) HST(7) case HDRAIN: wait_vm<0>(); break; default: break; }
          ++hk;
        }
        PART2();
      }
      if (seg == 0) {
#pragma unroll
        for (int i = 0; i < 16; ++i) H[i] = acc[i >> 2][i & 3];
        hk = 0;
      } else if (seg == 1) {
        const auto srdP = __builtin_amdgcn_make_buffer_rsrc((void*)part, 0, owner ? 16384 : 0, 0x00020000);
#pragma unroll
        for (int i = 0; i < 16; ++i) H[i] = __builtin_bit_cast(f32x4, __builtin_amdgcn_raw_buffer_load_b128(srdP, lane * 16 + (i & 3) * 1024, (i >> 2) * 4096, 16));
      }
    }
    if (wid == 0) TS(2);
#pragma unroll
    for (int i = 0; i < 16; ++i) { H[i] += acc[i >> 2][i & 3]; acc[i >> 2][i & 3] = (f32x4){0.f, 0.f, 0.f, 0.f}; }
    bf16x4 bl;
    auto rd1 = [&](int k, bf16x8 (&A)[4], bf16x8 (&B)[4], lptr pa, int ks) {
      if (k >= 2 && k < 6) { A[k - 2] = *(const LAS bf16x8*)(pa + (k - 2) * 2048); return; }
      const int n = k < 2 ? 0 : (k - 4) >> 1;
      const bf16x4 v = __builtin_amdgcn_ds_read_tr16_b64_v4i16((LAS bf16x4*)(pB[n] + ks * 8192 + (k & 1) * 1024));
      if (k & 1) B[n] = __builtin_shufflevector(bl, v, 0, 1, 2, 3, 4, 5, 6, 7); else bl = v;
    };
#pragma unroll
    for (int m = 0; m < 4; ++m) {
      f32x2 y[8], t[8]; unsigned u[8]; u32x4 lo, hi;
#pragma unroll
      for (int e = 0; e < 8; ++e) y[e] = (f32x2){H[4 * m + (e >> 1)][2 * (e & 1)], H[4 * m + (e >> 1)][2 * (e & 1) + 1]};
      auto ops = [&](int g) { gelu_step(g, y, t, u); };
      SCHED();
#pragma unroll
      for (int g = 0; g < 16; ++g) { mm(FA[0], FB[0], g, g + 1); if (g < 12) rd1(g, FA[1], FB[1], pA1, 1); ops(g); SCHED(); }
      so = so + SLOT == NS * SLOT ? 0 : so + SLOT;
#pragma unroll
      for (int g = 16; g < 20; ++g) {
        mm(FA[1], FB[1], g - 16, g - 15);
        if (g == 16) { pA0 = lds + baseA0 + so; pA1 = lds + (baseA0 ^ 64u) + so; asm volatile("" : "+v"(pA0), "+v"(pA1)); }
        else if (g == 17) { pB[0] = lds + baseB[0] + so; pB[1] = lds + baseB[1] + so; asm volatile("" : "+v"(pB[0]), "+v"(pB[1])); }
        else if (g == 18) { pB[2] = lds + baseB[2] + so; pB[3] = lds + baseB[3] + so; asm volatile("" : "+v"(pB[2]), "+v"(pB[3])); }
        ops(g); SCHED();
      }
      __builtin_amdgcn_s_waitcnt(0xC07F); __builtin_amdgcn_s_barrier(); SCHED();
#pragma unroll
      for (int g = 20; g < 28; ++g) { mm(FA[1], FB[1], g - 16, g - 15); rd1(g - 20, FA[0], FB[0], pA0, 0); ops(g); SCHED(); }
      mm(FA[1], FB[1], 12, 16);
#pragma unroll
      for (int k = 8; k < 12; ++k) rd1(k, FA[0], FB[0], pA0, 0);
      shuffle(u[0], u[1], u[2], u[3], u[4], u[5], u[6], u[7], lo, hi);
      H[4 * m] = __builtin_bit_cast(f32x4, lo); H[4 * m + 1] = __builtin_bit_cast(f32x4, hi);
#pragma unroll
      for (int i = 0; i < 4; ++i) { __builtin_amdgcn_sched_group_barrier(0x8, 1, 0); __builtin_amdgcn_sched_group_barrier(0x100, 1, 0); __builtin_amdgcn_sched_group_barrier(0x2, 5, 0); }
      SCHED();
    }
    auto store0 = [&](int m) {
#define S0(m) st16_l2(o0 + m * 65536, vo, __builtin_bit_cast(u32x4, H[4 * m])); st16_l2(o0 + m * 65536 + 32768, vo, __builtin_bit_cast(u32x4, H[4 * m + 1])); break;
      switch (m) { case 0: S0(0) case 1: S0(1) case 2: S0(2) default: S0(3) }
    };
#pragma unroll 1
    for (int kt = 4 - kb0; kt < n2 - kb0; ++kt) {
      PART1();
      if ((unsigned)kt <= (unsigned)ODRAIN) {
        if (kt < 4) store0(kt);
        else if (kt == ODRAIN) wait_vm<0>();
      }
      PART2();
    }
    if (wid == 0) TS(3);
#pragma unroll
    for (int m = 0; m < 4; ++m) {
      u32x4 lo, hi; pack2(m, lo, hi);
      if (T1.dest != rank) wait_vm<1>();
      st16_sys<0>(o1 + m * 65536, vo, lo);
      if (T1.dest != rank) wait_vm<1>();
      st16_sys<0>(o1 + m * 65536 + 32768, vo, hi); SCHED();
    }
    if (T1.dest != rank) signal(cnt1);
  }
}

extern "C" void kernel_launch(int rank, int nranks, ncclComm_t comm, hipStream_t stream,
                              void* const* d_in, const int* in_sizes, int n_in,
                              void* d_out, int out_size, const int* in_dtypes, int out_dtype,
                              void* const* d_out_all, void* const* d_ws_all, size_t ws_bytes) {
  if (nranks != NR || ws_bytes < WS_END || (unsigned)rank >= (unsigned)NR) { fprintf(stderr, "unsupported configuration\n"); return; }
  HIP_CHECK(hipFuncSetAttribute((const void*)gemm_a2a_kernel, hipFuncAttributeMaxDynamicSharedMemorySize, LDS_BYTES));
  Params p;
  p.x = (const char*)d_in[0]; p.w = (const char*)d_in[1]; p.rank = rank; p.pad = 0;
  for (int i = 0; i < NR; ++i) { p.out[i] = (char*)d_out_all[i]; p.ws[i] = (char*)d_ws_all[i]; }
  build_desc(p.desc);
  gemm_a2a_kernel<<<NWG, 512, LDS_BYTES, stream>>>(p);
  HIP_CHECK(hipGetLastError());
}
```
